```python
import numpy as np

kernel_name = "gpu_ag_gemm_m4096_k4096_n4096_e4m3e4m3_dequant"


ABSMAX_RTOL = 0.03

COMPUTE_DTYPE = "float32"
EPILOGUE = "dequant"
IN_QUANT = None
OUT_QUANT = None
IN_FP8 = "e4m3e4m3"
IN_DTYPES = {"x": "e4m3", "w_mat": "e4m3",
             "scale_x": "float32", "scale_w": "float32"}
WORLD_SIZE = 8

def _snap_fp8(x, mant_bits, max_abs, min_normal):
    a = np.abs(x.astype(np.float64))
    _m, _e = np.frexp(a)
    q = np.where(a >= min_normal, np.ldexp(1.0, _e - 1 - mant_bits),
                 min_normal * 2.0 ** -mant_bits)
    snapped = np.minimum(np.round(a / q) * q, max_abs)
    return (np.sign(x) * snapped).astype(np.float32)

def setup_inputs():

    x = _snap_fp8(np.random.randn(4096, 4096).astype(np.float32), 3, 448.0, 2.0 ** -6)
    w_mat = _snap_fp8(np.random.randn(4096, 4096).astype(np.float32), 3, 448.0, 2.0 ** -6)
    scale_x = (0.02 * (1.0 + np.random.rand(1))).astype(np.float32)
    scale_w = (0.02 * (1.0 + np.random.rand(1))).astype(np.float32)
    return {"x": x, "w_mat": w_mat, "scale_x": scale_x, "scale_w": scale_w}

def reference(x, w_mat, scale_x, scale_w):
    acc = (x @ w_mat).astype(np.float32)
    y = (acc * (scale_x * scale_w)).astype(np.float32)
    return y
```

```cpp
#include <hip/hip_runtime.h>
#include <rccl/rccl.h>
#include <cstdio>
#include <cstdint>

#define HIP_CHECK(expr) do { hipError_t _e = (expr); if (_e != hipSuccess) { \
    fprintf(stderr, "HIP error %s at %s:%d\n", hipGetErrorString(_e), __FILE__, __LINE__); \
    return; } } while (0)

typedef unsigned u32x4 __attribute__((ext_vector_type(4)));
typedef int v2i __attribute__((ext_vector_type(2)));
typedef int v4i __attribute__((ext_vector_type(4)));
typedef int v8i __attribute__((ext_vector_type(8)));
typedef float v16f __attribute__((ext_vector_type(16)));
typedef unsigned long long u64;
#define LAS __attribute__((address_space(3)))
#define GAS __attribute__((address_space(1)))

constexpr int NR = 8;
constexpr int KTOT = 4096, NL = 512, MS = 512;
constexpr int BM = 64, BN = 128, KS = 128, NSL = KTOT / KS;
constexpr int UNIT = BM * KS;
constexpr int NWG = 256, NTHR = 512;
constexpr size_t WS_SLOT = (size_t)NWG * UNIT;
constexpr size_t WS_FLAG = 2 * WS_SLOT;
constexpr size_t WS_CNT  = WS_FLAG + NR * 8 * 128;
constexpr size_t WS_TMO  = WS_CNT + NWG * 128;
constexpr size_t WS_DBG  = WS_TMO + 128;
constexpr size_t WS_END  = WS_DBG + NWG * 64 * 8;
constexpr u64 DEADLINE = 30000000ull;
constexpr int LDS_X = 0;
constexpr int LDS_W = 2 * UNIT;
constexpr int WSL = KS * BN;
constexpr int LDS_BAD = LDS_W + 3 * WSL;
constexpr int LDS_TOT = 96 * 1024;

struct Args {
  char* ws[NR];
  const unsigned char* x; const unsigned char* w;
  const float* sx; const float* sw; float* out;
  int rank; int pad;
};

#define STAMP(i) do {} while (0)

template <int N> __device__ __forceinline__ void waitv() { asm volatile("s_waitcnt vmcnt(%0)" :: "n"(N) : "memory"); }
__device__ __forceinline__ void bar() { asm volatile("s_waitcnt lgkmcnt(0)" ::: "memory"); __builtin_amdgcn_s_barrier(); asm volatile("" ::: "memory"); }

__device__ __forceinline__ void glds16(const void* g, unsigned lds_addr) {
  unsigned keep;
  asm volatile("s_mov_b32 %0, m0\n\ts_mov_b32 m0, %2\n\ts_nop 0\n\tglobal_load_lds_dwordx4 %1, off\n\ts_mov_b32 m0, %0"
               : "=&s"(keep) : "v"(g), "s"(lds_addr) : "memory");
}

__global__ void __launch_bounds__(NTHR, 2) ag_gemm(Args a) {
  extern __shared__ __attribute__((aligned(16))) char lds[];
  const int tid = threadIdx.x, lane = tid & 63;
  const int wid = __builtin_amdgcn_readfirstlane(tid >> 6);
  const int b = blockIdx.x;
  const int rank = a.rank;
  char* const me = a.ws[rank];
  if (wid == 0) STAMP(0);

  unsigned call;
  {
    const int j = b & 7, s = b >> 3;
    const int row = tid >> 3, cp = tid & 7;
    const int c = cp ^ ((row >> 1) & 7);
    const u32x4 v = *(const u32x4*)(a.x + (size_t)(j * BM + row) * KTOT + s * KS + c * 16);
    unsigned* f = (unsigned*)(a.ws[tid & (NR - 1)] + WS_FLAG + (size_t)(rank * 8 + j) * 128) + s;
    unsigned* cnt = (unsigned*)(me + WS_CNT + (size_t)b * 128);
    call = __hip_atomic_load(cnt, __ATOMIC_RELAXED, __HIP_MEMORY_SCOPE_AGENT) + 1u;
    call = __builtin_amdgcn_readfirstlane(call);
    char* dst = me + (size_t)(call & 1) * WS_SLOT + (size_t)(j * NSL + s) * UNIT + tid * 16;
    asm volatile("global_store_dwordx4 %0, %1, off sc0 sc1\n\ts_nop 1" :: "v"(dst), "v"(v), "v"(f) : "memory");
    waitv<0>();
    __builtin_amdgcn_s_barrier();
    if (tid < NR) __hip_atomic_store(f, call, __ATOMIC_RELAXED, __HIP_MEMORY_SCOPE_SYSTEM);
    if (tid == 64) __hip_atomic_store(cnt, call, __ATOMIC_RELAXED, __HIP_MEMORY_SCOPE_AGENT);
  }
  if (wid == 0) STAMP(1);

  const int j = b & 7, idx = b >> 3, q = idx >> 2, ct = idx & 3;
  const int src = (rank + q) & (NR - 1);
  unsigned* const tmo = (unsigned*)(me + WS_TMO);

  if (wid >= 4) {
    const int lw = wid - 4;
    if (lw == 0) {
      const unsigned* f = (const unsigned*)(me + WS_FLAG + (size_t)(src * 8 + j) * 128) + (lane & 31);
      const u64 t0 = __builtin_amdgcn_s_memrealtime();
      while (true) {
        unsigned v = __hip_atomic_load(f, __ATOMIC_RELAXED, __HIP_MEMORY_SCOPE_SYSTEM);
        if (__all(v >= call)) break;
        if (__hip_atomic_load(tmo, __ATOMIC_RELAXED, __HIP_MEMORY_SCOPE_AGENT) == call ||
            __builtin_amdgcn_s_memrealtime() - t0 > DEADLINE) {
          __hip_atomic_store(tmo, call, __ATOMIC_RELAXED, __HIP_MEMORY_SCOPE_AGENT);
          break;
        }
        __builtin_amdgcn_s_sleep(1);
      }
      STAMP(2);
      __builtin_amdgcn_fence(__ATOMIC_ACQUIRE, "");
      const unsigned bad0 = __hip_atomic_load(tmo, __ATOMIC_RELAXED, __HIP_MEMORY_SCOPE_AGENT) == call;
      waitv<0>();
      *(volatile LAS unsigned*)(lds + LDS_BAD) = bad0;
      STAMP(3);
    }
    bar();
    if (__builtin_amdgcn_readfirstlane(*(volatile LAS unsigned*)(lds + LDS_BAD))) return;
    const char* base = a.ws[src] + (size_t)(call & 1) * WS_SLOT + (size_t)(j * NSL) * UNIT + lw * 2048 + lane * 16;
    char* xl = lds + LDS_X + lw * 2048 + lane * 16;
    u32x4 r[3][2];
#define LDU(set, s) do { const char* _p = base + (size_t)(s) * UNIT; \
      asm volatile("global_load_dwordx4 %0, %2, off\n\tglobal_load_dwordx4 %1, %2, off offset:1024" \
                   : "=&v"(r[set][0]), "=&v"(r[set][1]) : "v"(_p) : "memory"); } while (0)
#define STEP(set, nset, s, VM) do { \
      waitv<VM>(); asm volatile("" : "+v"(r[set][0]), "+v"(r[set][1])); \
      if ((s) + 2 < NSL) LDU(nset, (s) + 2); \
      *(u32x4*)(xl + ((s) & 1) * UNIT) = r[set][0]; *(u32x4*)(xl + ((s) & 1) * UNIT + 1024) = r[set][1]; \
      bar(); } while (0)
    LDU(0, 0); LDU(1, 1);
    for (int s = 0; s < 30; s += 3) {
      STEP(0, 2, s, 2); STEP(1, 0, s + 1, 2); STEP(2, 1, s + 2, 2);
    }
    STEP(0, 2, 30, 2); STEP(1, 0, 31, 0);
    if (lw == 0) STAMP(4);
  } else {
    const int kr = lane >> 3, cp = lane & 7;
    const int cw = ((cp >> 1) ^ ((kr >> 1) & 3)) * 32 + (cp & 1) * 16;
    const unsigned char* wsrc = a.w + (size_t)(wid * 32 + kr) * NL + ct * BN + cw;
#define LDW(s) do { _Pragma("unroll") for (int i = 0; i < 4; ++i) \
      glds16(wsrc + (size_t)((s) * KS + i * 8) * NL, lds0 + LDS_W + ((s) % 3) * WSL + (wid * 4 + i) * 1024); } while (0)
    const unsigned lds0 = (unsigned)(size_t)(LAS char*)lds;
    float sc = a.sx[0] * a.sw[0];
    asm volatile("" : "+v"(sc));
    LDW(0); LDW(1);
    bar();
    const bool bad = __builtin_amdgcn_readfirstlane(*(volatile LAS unsigned*)(lds + LDS_BAD));
    v16f acc[2] = {};
    const int r31 = lane & 31, h = lane >> 5;
    const int asw = (r31 >> 1) & 7;
    const int q8 = (lane & 15) >> 1, p8 = lane & 1, g1 = (lane >> 4) & 1;
    const int boff = (32 * h + q8) * 128 + ((wid ^ ((q8 >> 1) & 3)) << 5) + g1 * 16 + p8 * 8;
    if (bad) { waitv<0>(); sc = __builtin_nanf(""); }
    else
    for (int s = 0; s < NSL; ++s) {
      if (s + 1 < NSL) waitv<4>(); else waitv<0>();
      bar();
      if (s + 2 < NSL) LDW(s + 2);
      const char* xb = lds + LDS_X + (s & 1) * UNIT;
      const char* wb = lds + LDS_W + (s % 3) * WSL + boff;
#pragma unroll
      for (int kk = 0; kk < 2; ++kk) {
        v8i bf;
#pragma unroll
        for (int r = 0; r < 4; ++r) {
          v2i t = __builtin_amdgcn_ds_read_tr8_b64_v2i32((LAS v2i*)(wb + (kk * 64 + r * 8) * 128));
          bf[2 * r] = t[0]; bf[2 * r + 1] = t[1];
        }
#pragma unroll
        for (int m = 0; m < 2; ++m) {
          const char* ap = xb + (m * 32 + r31) * 128;
          v4i a0 = *(const v4i*)(ap + (((kk * 4 + h * 2 + 0) ^ asw) << 4));
          v4i a1 = *(const v4i*)(ap + (((kk * 4 + h * 2 + 1) ^ asw) << 4));
          v8i af = {a0[0], a0[1], a0[2], a0[3], a1[0], a1[1], a1[2], a1[3]};
          acc[m] = __builtin_amdgcn_mfma_scale_f32_32x32x64_f8f6f4(af, bf, acc[m], 0, 0, 0, 0, 0, 0);
        }
      }
    }
    if (wid == 0) STAMP(5);
    float* o = a.out + (size_t)(src * MS + j * BM + 4 * h) * NL + ct * BN + wid * 32 + r31;
#pragma unroll
    for (int m = 0; m < 2; ++m)
#pragma unroll
      for (int i = 0; i < 16; ++i)
        o[(size_t)(m * 32 + (i & 3) + 8 * (i >> 2)) * NL] = acc[m][i] * sc;
  }
}

extern "C" void kernel_launch(int rank, int nranks, ncclComm_t comm,
                              hipStream_t stream,
                              void* const* d_in, const int* in_sizes, int n_in,
                              void* d_out, int out_size,
                              const int* in_dtypes, int out_dtype,
                              void* const* d_out_all,
                              void* const* d_ws_all, size_t ws_bytes) {
  if (nranks != NR || (unsigned)rank >= (unsigned)NR || ws_bytes < WS_END || n_in < 4) return;
  static bool init = false;
  if (!init) {
    HIP_CHECK(hipFuncSetAttribute((const void*)ag_gemm, hipFuncAttributeMaxDynamicSharedMemorySize, LDS_TOT));
    init = true;
  }
  Args a;
  for (int p = 0; p < NR; ++p) a.ws[p] = (char*)d_ws_all[p];
  a.x = (const unsigned char*)d_in[0]; a.w = (const unsigned char*)d_in[1];
  a.sx = (const float*)d_in[2]; a.sw = (const float*)d_in[3];
  a.out = (float*)d_out; a.rank = rank; a.pad = 0;
  hipLaunchKernelGGL(ag_gemm, dim3(NWG), dim3(NTHR), LDS_TOT, stream, a);
  HIP_CHECK(hipGetLastError());
}
```

```cpp
#include <hip/hip_runtime.h>
#include <rccl/rccl.h>
#include <cstdio>
#include <cstdint>

#pragma clang diagnostic ignored "-Wint-to-pointer-cast"

#define HIP_CHECK(expr) do { hipError_t _e = (expr); if (_e != hipSuccess) { \
    fprintf(stderr, "HIP error %s at %s:%d\n", hipGetErrorString(_e), __FILE__, __LINE__); \
    return; } } while (0)

typedef unsigned u32;
typedef unsigned u32x2 __attribute__((ext_vector_type(2)));
typedef unsigned u32x4 __attribute__((ext_vector_type(4)));
typedef int v2i __attribute__((ext_vector_type(2)));
typedef int v8i __attribute__((ext_vector_type(8)));
typedef float v16f __attribute__((ext_vector_type(16)));
typedef unsigned long long u64;
#define LAS __attribute__((address_space(3)))
#define LP(T, a) ((LAS T*)(u32)(a))

#define TAU 226
#define CREDIT 4u
#define WB 5
#define WA 2
#define W3 W2
#define KEND 8
#define PRE 5
#define W2 6
#define PACE 60
#define PUBT 150
#define PSLEEP 8
#define NPUSH 4

constexpr int NR = 8;
constexpr int KTOT = 4096, NL = 512, MS = 512;
constexpr int BM = 64, BN = 128, KS = 128, NSL = KTOT / KS;
constexpr int NWG = 256, NTHR = 640;
constexpr int PL = 4096;
constexpr int U_C = 4096, U_H = 6144, U_V = 6160, USTRIDE = 9216;
static_assert(NPUSH >= 0 && NPUSH <= 5, "");
constexpr size_t WS_SLOT  = (size_t)NWG * USTRIDE;
constexpr size_t WS_PUSH  = 2 * WS_SLOT;
constexpr size_t PUSH_PAR = (size_t)NR * 8 * 8 * PL;
constexpr size_t WS_E     = WS_PUSH + 2 * PUSH_PAR;
constexpr size_t WS_FLAG  = WS_E + (size_t)64 * NSL * PL;
constexpr size_t FLAG_PAR = (size_t)NR * 8 * NSL * 128;
constexpr size_t WS_PFLAG = WS_FLAG + 2 * FLAG_PAR;
constexpr size_t WS_EFLAG = WS_PFLAG + NR * 8 * 128;
constexpr size_t WS_CNT   = WS_EFLAG + 64 * 512;
constexpr size_t WS_INV   = WS_CNT + NWG * 128;
constexpr size_t WS_TMO   = WS_INV + 16 * 128;
constexpr size_t WS_DBG   = WS_TMO + 128;
constexpr size_t WS_END   = WS_DBG + NWG * 64 * 8;
constexpr u64 DEADLINE = 30000000ull;
constexpr u64 SLOW = 20000ull;
constexpr int WSL = KS * BN;
constexpr int LDS_LUT = 0;
constexpr int LDS_CLUT = 128;
constexpr int LDS_MISC = 256;
constexpr int L_BAD = LDS_MISC, L_VAL = LDS_MISC + 4, L_NGO = LDS_MISC + 8, L_STOP = LDS_MISC + 12;
constexpr int L_ARR = LDS_MISC + 16, L_NL = LDS_MISC + 20, L_RDY = LDS_MISC + 24, L_CVL = LDS_MISC + 28, L_DUMMY = LDS_MISC + 64;
constexpr int SCR = 2048;
constexpr int A3O = 1040, A4O = 1312, VBO = 1536;
constexpr int LDS_SCR = 512;
constexpr int LDS_E = 9 * 1024;
constexpr int LDS_N = LDS_E + 4 * PL;
constexpr int LDS_W = LDS_N + 8 * PL;
constexpr int CVS = 4352;
constexpr int LDS_CV = LDS_W + 4 * WSL;
constexpr int LDS_TOT = LDS_CV + 8 * CVS;
static_assert(LDS_SCR + 4 * SCR <= LDS_E && LDS_TOT <= 160 * 1024, "lds");

struct Args {
  char* ws[NR];
  const unsigned char* x; const unsigned char* w;
  const float* sx; const float* sw; float* out;
  int rank; int pad;
};

#define STAMP(i) do {} while (0)
#define TDECL
#define TSTART() do {} while (0)
#define TACC(v) do {} while (0)
#define TOUT(i, v) do {} while (0)
#define RMARK() do {} while (0)

#define SCHED() __builtin_amdgcn_sched_barrier(0)
template <int N> __device__ __forceinline__ void waitv() { asm volatile("s_waitcnt vmcnt(%0)" :: "n"(N) : "memory"); }
__device__ __forceinline__ void cfence() { asm volatile("" ::: "memory"); }
__device__ __forceinline__ void bar() {
  cfence(); SCHED();
  __builtin_amdgcn_s_waitcnt(0xC07F);
  __builtin_amdgcn_s_barrier();
  SCHED(); cfence();
}
__device__ __forceinline__ void set_m0(u32 lds_addr) { asm volatile("s_mov_b32 m0, %0\n\ts_nop 0" :: "s"(lds_addr) : "memory"); }
template <int OFF, bool SYS> __device__ __forceinline__ void glds(const void* sbase, u32 voff) {
  if (SYS) asm volatile("global_load_lds_dwordx4 %0, %1 offset:%2 sc0 sc1" :: "v"(voff), "s"(sbase), "n"(OFF) : "memory");
  else asm volatile("global_load_lds_dwordx4 %0, %1 offset:%2" :: "v"(voff), "s"(sbase), "n"(OFF) : "memory");
}
__device__ __forceinline__ void st16(void* p, u32x4 v) { asm volatile("global_store_dwordx4 %0, %1, off sc0 sc1\n\ts_nop 1" :: "v"(p), "v"(v) : "memory"); }
__device__ __forceinline__ void st8(void* p, u32x2 v) { asm volatile("global_store_dwordx2 %0, %1, off sc0 sc1\n\ts_nop 1" :: "v"(p), "v"(v) : "memory"); }
__device__ __forceinline__ void st4(void* p, u32 v) { asm volatile("global_store_dword %0, %1, off sc0 sc1\n\ts_nop 1" :: "v"(p), "v"(v) : "memory"); }
__device__ __forceinline__ u32 ld4(const void* p) {
  u32 v;
  asm volatile("global_load_dword %0, %1, off sc0 sc1\n\ts_waitcnt vmcnt(0)" : "=&v"(v) : "v"(p) : "memory");
  return v;
}

__device__ __forceinline__ bool expired(u64 t0, u32* tmo, u32 call) {
  const u64 dt = __builtin_amdgcn_s_memrealtime() - t0;
  if (dt <= SLOW) return false;
  if (dt > DEADLINE) { st4(tmo, call); return true; }
  __builtin_amdgcn_s_sleep(64);
  return __builtin_amdgcn_readfirstlane(ld4(tmo)) == call;
}

__device__ __forceinline__ u32 wave_scan(u32 x) {
  x += __builtin_amdgcn_update_dpp(0u, x, 0x111, 0xf, 0xf, true);
  x += __builtin_amdgcn_update_dpp(0u, x, 0x112, 0xf, 0xf, true);
  x += __builtin_amdgcn_update_dpp(0u, x, 0x114, 0xf, 0xf, true);
  x += __builtin_amdgcn_update_dpp(0u, x, 0x118, 0xf, 0xf, true);
  x += __builtin_amdgcn_update_dpp(0u, x, 0x142, 0xa, 0xf, false);
  x += __builtin_amdgcn_update_dpp(0u, x, 0x143, 0xc, 0xf, false);
  return x;
}
constexpr u32 B1 = 0x01010101u, M0F = 0x0F0F0F0Fu, MSM = 0x87878787u, MEX = 0x78787878u, M03 = 0x03030303u;
__device__ __forceinline__ u32 rotl4(u32 x) { return __builtin_amdgcn_alignbit(x, x, 28); }
__device__ __forceinline__ u32 rotr4(u32 x) { return __builtin_amdgcn_alignbit(x, x, 4); }
__device__ __forceinline__ u32x2 centry(u32 p) {
  return *LP(const u32x2, __builtin_amdgcn_udot4(p & 0x04040404u, 0x10080402u, (u32)LDS_CLUT, false));
}
__device__ __forceinline__ u32x2 dentry(u32 c, bool with_count = true) {
  const u32 p = __builtin_amdgcn_udot4(__builtin_amdgcn_perm(0u, 0x01000000u, c), 0x40201008u, (u32)LDS_LUT, false);
  if (with_count) return *LP(const u32x2, p);
  return u32x2{*LP(const u32, p), 0u};
}
__device__ __forceinline__ void put4(u32 o, u32 y) {
  const u64 w = (u64)y << ((o & 3u) * 8u);
  LAS u32* p = LP(u32, o & ~3u);
  __hip_atomic_fetch_or(p, (u32)w, __ATOMIC_RELAXED, __HIP_MEMORY_SCOPE_WAVEFRONT);
  __hip_atomic_fetch_or(p + 1, (u32)(w >> 32), __ATOMIC_RELAXED, __HIP_MEMORY_SCOPE_WAVEFRONT);
}
__device__ __forceinline__ u32x2 pair(u32 o) {
  const LAS u32* p = LP(const u32, o & ~3u);
  return u32x2{p[0], p[1]};
}
__device__ __forceinline__ u32 expand(u32x2 w, u32 o, u32 table, u32 sel) {
  return __builtin_amdgcn_perm(__builtin_amdgcn_alignbyte(w[1], w[0], o), table, sel);
}

__global__ void __launch_bounds__(NTHR, 3) ag_gemm(Args a) {
  const int tid = threadIdx.x, lane = tid & 63;
  const int wid = __builtin_amdgcn_readfirstlane(tid >> 6);
  const int b = blockIdx.x;
  const int rank = a.rank;
  char* const me = a.ws[rank];
  if (wid == 0) STAMP(0);
  const int j = b & 7, idx = b >> 3, q = idx >> 2, ct = idx & 3;
  const int src = (rank + q) & (NR - 1);
  u32* const tmo = (u32*)(me + WS_TMO);
  const u32 dummy = L_DUMMY + (lane & 31) * 4;
  const u32 l16 = lane * 16u;
  u32* const cnt = (u32*)(me + WS_CNT + (size_t)b * 128);
  const int brow = (wid & 3) * 16 + (lane >> 2);
  const unsigned char* const xlane = a.x + (size_t)(j * BM + brow) * KTOT + (((lane & 3) ^ ((brow >> 2) & 3)) << 5);

  if (wid < 4) {
    constexpr int NT = (NPUSH + 3) / 4;
    u32x4 pv[NT ? NT : 1][2];
#pragma unroll
    for (int t = 0; t < NT; ++t) {
      const int p = min((idx + 32 * t) >> 3, NPUSH - 1);
      pv[t][0] = *(const u32x4*)(xlane + p * KS); pv[t][1] = *(const u32x4*)(xlane + p * KS + 16);
    }
    const u32 callv = __hip_atomic_load(cnt, __ATOMIC_RELAXED, __HIP_MEMORY_SCOPE_AGENT);
    if (tid < 16) {
      u32 add = 0, cs = 0x0C0C0C0Cu, n = 0;
#pragma unroll
      for (int i = 0; i < 4; ++i)
        if ((tid >> i) & 1) { add |= (1 + n) << (8 * i); cs = (cs & ~(0xFFu << (8 * n))) | (i << (8 * n)); ++n; }
      *LP(u32x2, LDS_LUT + tid * 8) = u32x2{add, n};
      *LP(u32x2, LDS_CLUT + tid * 8) = u32x2{cs, n};
    }
    const int kr = lane >> 3, cp = lane & 7;
    u32 wvo[4];
#pragma unroll
    for (int i = 0; i < 4; ++i) wvo[i] = (kr + i * 8) * NL + ((cp >> 1) ^ ((kr >> 1) & 3)) * 32 + (cp & 1) * 16 - i * 1024;
    const unsigned char* const wsrc = a.w + (size_t)(wid * 32) * NL + ct * BN;
    auto stage_w = [&](int s) {
      set_m0(LDS_W + (s & 3) * WSL + wid * 4096);
      const unsigned char* sb = wsrc + (size_t)s * KS * NL;
      glds<0, false>(sb, wvo[0]); glds<1024, false>(sb, wvo[1]); glds<2048, false>(sb, wvo[2]); glds<3072, false>(sb, wvo[3]);
    };
    char* const pdst = a.ws[(rank + idx) & (NR - 1)];
    bar();
    const u32 call = __builtin_amdgcn_readfirstlane(callv) + 1u;
    const size_t par = call & 1;
#pragma unroll
    for (int t = 0; t < NT; ++t) asm volatile("" :: "v"(pv[t][0]), "v"(pv[t][1]));
    const u64 tpush = __builtin_amdgcn_s_memrealtime();
    if (idx >= 8) {
#pragma unroll 1
      for (int s = 0; s < 4; ++s) stage_w(s);
    }
    bool b1 = false;
#pragma unroll
    for (int t = 0; t < NT; ++t) {
      const int p = (idx + 32 * t) >> 3;
      if (p < NPUSH) {
        u32x4 np;
#pragma unroll
        for (int k = 0; k < 4; ++k) np[k] = (pv[t][0][k] & MSM) | rotl4(pv[t][1][k] & MSM);
        if (p * PACE > PUBT && !b1) { __builtin_amdgcn_s_barrier(); b1 = true; }
        while (__builtin_amdgcn_s_memrealtime() - tpush < (u64)(p * PACE)) __builtin_amdgcn_s_sleep(1);
        st16(pdst + WS_PUSH + par * PUSH_PAR + (size_t)((rank * 8 + j) * 8 + p) * PL + wid * 1024 + l16, np);
      }
    }
    if (idx < 8) {
#pragma unroll 1
      for (int s = 0; s < 4; ++s) stage_w(s);
    }
    if (!b1) __builtin_amdgcn_s_barrier();
    bar();
    float sc = a.sx[0] * a.sw[0];
    waitv<0>();
    asm volatile("" : "+v"(sc));
    if (lane < NT && ((idx + 32 * lane) >> 3) < NPUSH)
      st4(pdst + WS_PFLAG + (size_t)(rank * 8 + j) * 128 + (((idx + 32 * lane) >> 3) * 4 + wid) * 4, call);
    if (wid == 0) STAMP(11);
    const char* const esrc = me + WS_E + (size_t)((q * 8 + j) * NSL) * PL + wid * 1024;
    const char* const psrc = me + WS_PUSH + par * PUSH_PAR + (size_t)((src * 8 + j) * 8) * PL + wid * 1024;
    auto stage_e = [&](int s, bool push) {
      if (push) { set_m0(LDS_N + (s & 7) * PL + wid * 1024); glds<0, true>(psrc + s * PL, l16); }
      set_m0(LDS_E + (s & 3) * PL + wid * 1024); glds<0, true>(esrc + s * PL, l16);
    };
    const bool bad0 = __builtin_amdgcn_readfirstlane(*LP(volatile u32, L_STOP));
    v16f acc[2] = {};
    const int r31 = lane & 31, h = lane >> 5, wm = wid >> 1, wn = wid & 1;
    const int q8 = (lane & 15) >> 1, p8 = lane & 1, g1 = (lane >> 4) & 1;
    u32 boff[2], aoff[2];
#pragma unroll
    for (int i = 0; i < 2; ++i) {
      boff[i] = LDS_W + (32 * h + q8) * 128 + (((wn * 2 + i) ^ ((q8 >> 1) & 3)) << 5) + g1 * 16 + p8 * 8;
      aoff[i] = (wm * 32 + r31) * 64 + (((i * 2 + h) ^ ((r31 >> 2) & 3)) << 4);
    }
    TDECL;
    struct Frag { u32x4 ep[2]; v8i bf[2][2]; };
    auto ldfrag = [&](Frag& f, int kk, int s) {
      f.ep[kk] = *LP(const u32x4, aoff[kk] + (s & 3) * PL + LDS_E);
#pragma unroll
      for (int n = 0; n < 2; ++n) {
        v2i t[4];
#pragma unroll
        for (int r = 0; r < 4; ++r) t[r] = __builtin_amdgcn_ds_read_tr8_b64_v2i32(LP(v2i, boff[n] + (s & 3) * WSL) + (kk * 64 + r * 8) * 16);
        f.bf[kk][n] = v8i{t[0][0], t[0][1], t[1][0], t[1][1], t[2][0], t[2][1], t[3][0], t[3][1]};
      }
    };
    auto afrag = [&](const u32x4 np, const u32x4 ep) {
      v8i af;
#pragma unroll
      for (int k = 0; k < 4; ++k) {
        af[k] = (np[k] & MSM) | (ep[k] & MEX);
        af[4 + k] = rotr4((np[k] & MEX) | (ep[k] & MSM));
      }
      return af;
    };
#define MFMA(n, af, bf) acc[n] = __builtin_amdgcn_mfma_scale_f32_32x32x64_f8f6f4(af, bf, acc[n], 0, 0, 0, 0, 0, 0)
    auto step = [&](int s, const Frag& cur, Frag& nxt, bool more, bool stg, bool push) {
      const u32x4 np0 = *LP(const u32x4, aoff[0] + (s & 7) * PL + LDS_N);
      const u32x4 np1 = *LP(const u32x4, aoff[1] + (s & 7) * PL + LDS_N);
      SCHED();
      if (stg) { stage_e(s + 4, push); stage_w(s + 4); }
      SCHED();
      const v8i a0 = afrag(np0, cur.ep[0]);
      MFMA(0, a0, cur.bf[0][0]);
      SCHED();
      const v8i a1 = afrag(np1, cur.ep[1]);
      SCHED();
      MFMA(1, a0, cur.bf[0][1]);
      SCHED();
      if (more) ldfrag(nxt, 0, s + 1);
      SCHED();
      MFMA(0, a1, cur.bf[1][0]);
      SCHED();
      if (more) ldfrag(nxt, 1, s + 1);
      SCHED();
      MFMA(1, a1, cur.bf[1][1]);
      SCHED();
      asm volatile("" :: "v"(a0), "v"(cur.bf[0][0]), "v"(cur.bf[0][1]), "v"(cur.bf[1][0]));
    };
    const u32 arr = lane ? dummy : (u32)L_ARR;
    auto spin = [&](int na, int nl, int nr) {
      while (true) {
        u32x4 w;
        asm volatile("ds_read_b128 %0, %1\n\ts_waitcnt lgkmcnt(0)" : "=&v"(w) : "v"((u32)L_ARR) : "memory");
        if (__all((int)w[0] >= na && (int)w[1] >= nl && (int)w[2] >= nr)) break;
      }
    };
    auto meet = [&](int s) {
      SCHED();
      asm volatile("s_waitcnt lgkmcnt(0)\n\tds_add_u32 %0, %1" :: "v"(arr), "v"(1u) : "memory");
      spin(4 * s + 8, s + 1, s == NSL - 1 ? NSL + 1 : min(s + 5, NSL));
      __builtin_amdgcn_s_waitcnt(0xC07F);
      SCHED();
    };
    if (!bad0) {
      Frag fa, fb;
      spin(0, 0, 4);
#pragma unroll 1
      for (int s = 0; s < 4; ++s) stage_e(s, s < NPUSH);
      waitv<0>(); meet(-1);
      ldfrag(fa, 0, 0); ldfrag(fa, 1, 0);
      meet(0); step(0, fa, fb, true, true, 4 < NPUSH);
      TSTART();
#pragma unroll 1
      for (int s = 1; s < NSL - 5; s += 2) {
        if (s > 9) TACC(_ta); else TACC(_tb);
        waitv<10>(); meet(s);
        if (s >= 9) TACC(_tc); else TACC(_tb);
        step(s, fb, fa, true, true, false);
        if (s >= 9) TACC(_ta); else TACC(_tb);
        waitv<10>(); meet(s + 1);
        if (s >= 9) TACC(_tc); else TACC(_tb);
        step(s + 1, fa, fb, true, true, false);
      }
      waitv<10>(); meet(NSL - 5); step(NSL - 5, fb, fa, true, true, false);
      waitv<10>(); meet(NSL - 4); step(NSL - 4, fa, fb, true, false, false);
      waitv<5>(); meet(NSL - 3); step(NSL - 3, fb, fa, true, false, false);
      waitv<0>(); meet(NSL - 2); step(NSL - 2, fa, fb, true, false, false);
      meet(NSL - 1); step(NSL - 1, fb, fa, false, false, false);
    }
    if (*LP(volatile u32, L_BAD)) sc = __builtin_nanf("");
    if (wid == 0) { STAMP(5); TOUT(35, _ta / 17); TOUT(37, _tc / 18); }
    float* o = a.out + (size_t)(src * MS + j * BM + wm * 32 + 4 * h) * NL + ct * BN + wn * 64 + r31;
#pragma unroll
    for (int i = 0; i < 16; ++i)
#pragma unroll
      for (int n = 0; n < 2; ++n)
        o[(size_t)((i & 3) + 8 * (i >> 2)) * NL + n * 32] = acc[n][i] * sc;
  } else if (wid < 8) {
    const int lw = wid - 4;
    const u32 scr = LDS_SCR + lw * SCR;
    u32 call;
    {
      const u32x4 v0 = *(const u32x4*)(xlane + idx * KS);
      const u32x4 v1 = *(const u32x4*)(xlane + idx * KS + 16);
      const u32 callv = __hip_atomic_load(cnt, __ATOMIC_RELAXED, __HIP_MEMORY_SCOPE_AGENT);
      char* fdst = a.ws[lane & (NR - 1)] + WS_FLAG + (size_t)((rank * 8 + j) * NSL + idx) * 128;
      *LP(u32x4, scr + l16) = u32x4{0, 0, 0, 0};
      *LP(u32x2, scr + 1024 + lane * 8) = u32x2{0, 0};
      bar();
      call = __builtin_amdgcn_readfirstlane(callv) + 1u;
      asm volatile("" : "+v"(fdst));
      if (lw == 0) STAMP(9);
      char* const dst = me + (size_t)(call & 1) * WS_SLOT + (size_t)(j * NSL + idx) * USTRIDE;
      const u32 vbuf = scr + VBO;
      const u32 v[8] = {v0[0], v0[1], v0[2], v0[3], v1[0], v1[1], v1[2], v1[3]};
      u32 z[8], sum = 0;
#pragma unroll
      for (int k = 0; k < 8; ++k) { z[k] = v[k] & MEX; if (k & 1) sum = __builtin_amdgcn_udot4(z[k], B1, sum, false); }
      sum = __builtin_amdgcn_readlane(wave_scan(sum), 63);
      const u32 bb = ((sum + 7946u) >> 13) & 15u;
      const u32 kz = ((7u - bb) & 15u) * 0x08080808u;
      u32x4 np;
#pragma unroll
      for (int k = 0; k < 4; ++k) np[k] = (v[k] & MSM) | rotl4(v[4 + k] & MSM);
      st16(dst + lw * 1024 + l16, np);
      u32x2 e1[8], cp2 = {0, 0};
#pragma unroll
      for (int k = 0; k < 8; ++k) {
        z[k] = (((z[k] + kz) >> 3) & M0F) ^ 0x04040404u;
        const u32 p = __builtin_amdgcn_perm(0xFFFFFFFFu, 0x8201FFFFu, z[k]);
        cp2[k >> 2] |= (p & M03) << (2 * (k & 3));
        e1[k] = centry(p);
      }
      const u32 t1 = (e1[0][1] + e1[1][1] + e1[2][1] + e1[3][1]) | ((e1[4][1] + e1[5][1] + e1[6][1] + e1[7][1]) << 16);
      const u32 sc1 = wave_scan(t1);
      const u32 tot1 = __builtin_amdgcn_readlane(sc1, 63);
      const u32 n2 = (tot1 & 0xFFFFu) + (tot1 >> 16);
      bool raw = n2 > 1024u;
      u32 n2d = 0, n3d = 0, n4d = 0;
      if (!raw) {
        const u32 ex = sc1 - t1 + (scr | ((scr + (tot1 & 0xFFFFu)) << 16));
        u32 o = ex & 0xFFFFu;
#pragma unroll
        for (int k = 0; k < 8; ++k) {
          if (k == 4) o = ex >> 16;
          put4(o, __builtin_amdgcn_perm(0u, z[k], e1[k][0])); o += e1[k][1];
        }
        cfence();
        const u32x4 q2 = *LP(const u32x4, scr + l16);
        u32 l2 = 0; u32x2 e2[4];
#pragma unroll
        for (int k = 0; k < 4; ++k) {
          const u32 p = __builtin_amdgcn_perm(0xFFFFFFFFu, 0xFFFF8201u, q2[k] ^ B1);
          l2 |= (p & M03) << (2 * k); e2[k] = centry(p);
        }
        *LP(u32, vbuf + lane * 4) = l2;
        const u32 t2 = e2[0][1] + e2[1][1] + e2[2][1] + e2[3][1];
        const u32 sc2 = wave_scan(t2);
        const u32 n3 = __builtin_amdgcn_readlane(sc2, 63);
        raw = n3 > 256u;
        if (!raw) {
          n2d = (n2 + 15u) >> 4; n3d = (n3 + 15u) >> 4;
          if (t2) {
            o = scr + A3O + sc2 - t2;
#pragma unroll
            for (int k = 0; k < 4; ++k) { put4(o, __builtin_amdgcn_perm(0u, q2[k] ^ 0x05050505u, e2[k][0])); o += e2[k][1]; }
          }
          cfence();
          const u32 q3 = *LP(const u32, scr + A3O + lane * 4) ^ 0x05050505u;
          const u32 p3 = __builtin_amdgcn_perm(0x800182FFu, 0xFFFFFFFFu, q3);
          u32 l3 = (p3 & M03) << (2 * (lane & 3));
          l3 |= __builtin_amdgcn_update_dpp(0u, l3, 0xB1, 0xf, 0xf, false);
          l3 |= __builtin_amdgcn_update_dpp(0u, l3, 0x4E, 0xf, 0xf, false);
          *LP(u32, vbuf + (n2d + (lane >> 2)) * 4) = l3;
          if (__any(p3 & 0x04040404u)) {
            const u32x2 e3 = centry(p3);
            const u32 sc3 = wave_scan(e3[1]);
            const u32 n4 = __builtin_amdgcn_readlane(sc3, 63);
            raw = n4 > 128u;
            if (!raw) {
              if (e3[1]) put4(scr + A4O + sc3 - e3[1], __builtin_amdgcn_perm(0u, q3, e3[0]));
              cfence();
              const u32x2 q4 = *LP(const u32x2, scr + A4O + (lane & 15) * 8);
              n4d = (n4 + 7u) >> 3;
              if (lane < 16) *LP(u32, vbuf + (n2d + n3d + lane) * 4) = q4[0] | (q4[1] << 4);
            }
          }
        }
      }
      cfence();
      u32 vd = n2d + n3d + n4d;
      if (raw) {
        cp2 = u32x2{(v[0] & MEX) | rotl4(v[4] & MEX), (v[1] & MEX) | rotl4(v[5] & MEX)}; vd = 128;
        *LP(u32, vbuf + lane * 4) = (v[2] & MEX) | rotl4(v[6] & MEX);
        *LP(u32, vbuf + 256 + lane * 4) = (v[3] & MEX) | rotl4(v[7] & MEX);
      }
      cfence();
      u32 voff;
      asm volatile("ds_add_rtn_u32 %0, %1, %2\n\ts_waitcnt lgkmcnt(0)" : "=&v"(voff) : "v"(lane ? dummy : (u32)L_VAL), "v"(vd) : "memory");
      const u32 va = *LP(const u32, vbuf + lane * 4), vb = *LP(const u32, vbuf + 256 + lane * 4);
      st8(dst + U_C + lw * 512 + lane * 8, cp2);
      voff = __builtin_amdgcn_readfirstlane(voff);
      if ((u32)lane < vd) st4(dst + U_V + (voff + lane) * 4, va);
      if ((u32)lane + 64u < vd) st4(dst + U_V + (voff + 64 + lane) * 4, vb);
      if (lane == 0) st4(dst + U_H + lw * 4, voff | (n2d << 10) | (n3d << 17) | (n4d << 22) | (bb << 27) | (raw ? 0x80000000u : 0u));
      if (lw == 0) STAMP(10);
      waitv<0>();
      bar();
      if (lw == 0) {
        const u32 vtot = *LP(volatile u32, L_VAL);
        if (lane < NR) st4(fdst + (size_t)(call & 1) * FLAG_PAR, (call << 6) | ((16u + vtot * 4u + 63u) >> 6));
        if (lane == 0) st4(cnt, call);
        STAMP(1);
      }
    }
    bar();
    if (__builtin_amdgcn_readfirstlane(*LP(volatile u32, L_STOP))) return;
    char* const edst = me + WS_E + (size_t)((q * 8 + j) * NSL + ct) * PL + lw * 1024 + l16;
    char* const eflag = me + WS_EFLAG + (size_t)(q * 8 + j) * 512 + ct * 16 + lw * 4;
    int pend = -1;
    TDECL;
#pragma unroll 1
    for (int t = 0; t < 8; ++t) {
      if ((int)__builtin_amdgcn_readfirstlane(*LP(volatile u32, L_CVL)) <= t) {
        if (pend >= 0) { waitv<0>(); if (lane == 0) st4(eflag + pend * 64, call); pend = -1; }
        while ((int)__builtin_amdgcn_readfirstlane(*LP(volatile u32, L_CVL)) <= t) __builtin_amdgcn_s_sleep(2);
      }
      cfence();
      TSTART();
      const u32 S = LDS_CV + t * CVS;
      const u32 info = __builtin_amdgcn_readfirstlane(*LP(const u32, S + 2048 + lw * 4));
      const u32x2 cc = *LP(const u32x2, S + lw * 512 + lane * 8);
      const u32 V = S + 2064 + (info & 0x1FFu) * 4;
      const u32 n2d = (info >> 10) & 0x7Fu, n3d = (info >> 17) & 0x1Fu, bb = (info >> 27) & 15u;
      u32 l2 = *LP(const u32, V + lane * 4);
      u32x4 ep;
      if (info >> 31) {
        ep = u32x4{cc[0], cc[1], l2, *LP(const u32, V + 256 + lane * 4)};
      } else {
        const u32 l3 = *LP(const u32, V + n2d * 4 + (lane >> 2) * 4);
        u32 c1[8], c2[4]; u32x2 e1[8], e2[4];
#pragma unroll
        for (int k = 0; k < 8; ++k) { c1[k] = (cc[k >> 2] >> (2 * (k & 3))) & M03; e1[k] = dentry(c1[k], (k & 3) != 3); }
        SCHED();
        l2 = (u32)lane < n2d ? l2 : 0u;
        u32 w4 = 0, c3 = (l3 >> (2 * (lane & 3))) & M03;
#pragma unroll
        for (int k = 0; k < 4; ++k) { c2[k] = (l2 >> (2 * k)) & M03; e2[k] = dentry(c2[k], k != 3); }
        SCHED();
        const u32 bbv = bb * B1;
        if (info & (0x1Fu << 22)) {
          c3 = (u32)lane < 4 * n3d ? c3 : 0u;
          const u32x2 e3 = dentry(c3);
          const u32 l4 = *LP(const u32, V + (n2d + n3d + (lane & 15)) * 4) ^ 0x44444444u;
          const u32 K4 = ((bb + 9u) & 15u) * B1;
          *LP(u32x2, scr + A4O + (lane & 15) * 8) = u32x2{(((l4 & M0F) + K4) & M0F) << 3, ((((l4 >> 4) & M0F) + K4) & M0F) << 3};
          cfence();
          const u32 o4 = scr + A4O + min(wave_scan(e3[1]) - e3[1], 128u);
          const u32x2 ww = pair(o4);
          w4 = __builtin_amdgcn_alignbyte(ww[1], ww[0], o4);
          c3 += e3[0];
        }
        *LP(u32, scr + A3O + lane * 4) = __builtin_amdgcn_perm(w4, ((bbv + 0x000A0B0Cu) & M0F) << 3, c3);
        const u32 ta = __builtin_popcount(cc[0] & (cc[0] >> 1) & 0x55555555u), tb = __builtin_popcount(cc[1] & (cc[1] >> 1) & 0x55555555u);
        const u32 t2 = __builtin_popcount(l2 & (l2 >> 1) & 0x55555555u);
        const u32 tt = ta | (tb << 11) | (t2 << 22);
        const u32 sc = wave_scan(tt);
        const u32 na = __builtin_amdgcn_readlane(sc, 63) & 0x7FFu;
        const u32 ex = sc - tt;
        cfence();
        u32 o2[4], o1[8]; u32x2 w[8];
        o2[0] = (ex >> 22) + (scr + A3O);
#pragma unroll
        for (int k = 1; k < 4; ++k) o2[k] = o2[k - 1] + e2[k - 1][1];
#pragma unroll
        for (int k = 0; k < 4; ++k) w[k] = pair(o2[k]);
        SCHED();
        o1[0] = (ex & 0x7FFu) + scr; o1[4] = ((ex >> 11) & 0x7FFu) + (scr + na);
#pragma unroll
        for (int k = 1; k < 8; ++k) if (k != 4) o1[k] = o1[k - 1] + e1[k - 1][1];
#pragma unroll
        for (int k = 0; k < 4; ++k) c2[k] += e2[k][0];
        SCHED();
        u32x4 y;
#pragma unroll
        for (int k = 0; k < 4; ++k) y[k] = expand(w[k], o2[k], ((bbv + 0x000D0E02u) & M0F) << 3, c2[k]);
        *LP(u32x4, scr + l16) = y;
        cfence();
#pragma unroll
        for (int k = 0; k < 8; ++k) w[k] = pair(o1[k]);
        SCHED();
#pragma unroll
        for (int k = 0; k < 8; ++k) c1[k] += e1[k][0];
        SCHED();
        const u32 T1 = ((bbv + 0x00000F01u) & M0F) << 3;
#pragma unroll
        for (int k = 0; k < 4; ++k) ep[k] = expand(w[k], o1[k], T1, c1[k]) | rotl4(expand(w[4 + k], o1[4 + k], T1, c1[4 + k]));
      }
      st16(edst + (size_t)t * 4 * PL, ep);
      TACC(_ta);
      if (pend >= 0) { waitv<1>(); if (lane == 0) st4(eflag + pend * 64, call); }
      pend = t;
      TACC(_tb);
    }
    waitv<0>();
    if (lane == 0) st4(eflag + 7 * 64, call);
    if (lw == 0) { STAMP(8); TOUT(32, _ta / 8); TOUT(33, _tb / 8); }
  } else if (wid == 9) {
    char* const inv = me + WS_INV + (size_t)(__builtin_amdgcn_s_getreg((3 << 11) | 20) & 15u) * 128;
    if (b < 8) asm volatile("buffer_inv sc0 sc1" ::: "memory");
    bar();
    const u32 call = __builtin_amdgcn_readfirstlane(ld4(cnt)) + 1u;
    if (b < 8 && lane == 0) st4(inv, call << 6);
    bar(); bar();
    if (__builtin_amdgcn_readfirstlane(*LP(volatile u32, L_STOP))) return;
    const u64 t0 = __builtin_amdgcn_s_memrealtime();
    {
      const char* f = lane < 32 ? me + WS_FLAG + (size_t)(call & 1) * FLAG_PAR + (size_t)((src * 8 + j) * NSL + lane) * 128 : inv;
      u32 go = 1;
      while (true) {
        const u64 ok = __ballot((ld4(f) >> 6) >= (call & 0x3FFFFFFu));
        if (ok == ~0ull) break;
        if ((u32)ok == ~0u && __builtin_amdgcn_s_memrealtime() - t0 > 300) {
          asm volatile("buffer_inv sc0 sc1\n\ts_waitcnt vmcnt(0)" ::: "memory");
          break;
        }
        if (expired(t0, tmo, call)) { go = 2; *LP(volatile u32, L_BAD) = 1; break; }
        __builtin_amdgcn_s_sleep(PSLEEP);
      }
      *LP(volatile u32, L_NGO) = go;
    }
    const char* const f = lane < 32 ? me + WS_EFLAG + (size_t)(q * 8 + j) * 512 + l16
                                    : me + WS_PFLAG + (size_t)(src * 8 + j) * 128 + (lane & 7) * 16;
    while (true) {
      u32x4 v;
      asm volatile("global_load_dwordx4 %0, %1, off sc0 sc1\n\ts_waitcnt vmcnt(0)" : "=&v"(v) : "v"(f) : "memory");
      const u64 known = __ballot(min(min(v[0], v[1]), min(v[2], v[3])) >= call);
      const int nready = __builtin_ctzll(~(u64)((u32)known & ((u32)(known >> 32) | (~0u << NPUSH))));
      if (nready == NSL) break;
      *LP(volatile u32, L_RDY) = nready;
      if (expired(t0, tmo, call)) break;
      __builtin_amdgcn_s_sleep(PSLEEP);
    }
    *LP(volatile u32, L_RDY) = NSL;
    if (__builtin_amdgcn_readfirstlane(ld4(tmo)) == call) *LP(volatile u32, L_BAD) = 1;
    *LP(volatile u32, L_RDY) = NSL + 1;
    STAMP(13);
  } else {
    __builtin_amdgcn_s_setprio(3);
    *LP(u32x4, LDS_MISC) = u32x4{0, 0, 0, 0};
    *LP(u32x4, LDS_MISC + 16) = u32x4{0, NPUSH, 0, 0};
    bar();
    const u32 call = __builtin_amdgcn_readfirstlane(ld4(cnt)) + 1u;
    bar();
    u32 fv;
    {
      const char* f = me + WS_FLAG + (size_t)(call & 1) * FLAG_PAR + (size_t)((src * 8 + j) * NSL + (lane & 7) * 4 + ct) * 128;
      const u64 t0 = __builtin_amdgcn_s_memrealtime();
      const u32 want = call & 0x3FFFFFFu;
      bool bad0 = false;
      while (true) {
        fv = ld4(f);
        if (__all((fv >> 6) >= want)) break;
        if (expired(t0, tmo, call)) { bad0 = true; break; }
        __builtin_amdgcn_s_sleep(1);
      }
      STAMP(2);
      waitv<0>();
      if (bad0) { *LP(volatile u32, L_BAD) = 1; *LP(volatile u32, L_STOP) = 1; bar(); return; }
    }
    const u32 ub = min(fv & 63u, 33u) * 64u;
    const char* const base = a.ws[src] + (size_t)(call & 1) * WS_SLOT + (size_t)(j * NSL) * USTRIDE;
    const u32 l3x = (lane & 3) * 16u - 64u;
    const u32 tau = q ? TAU : TAU * 3 / 4;
    u32 due = ((u32)__builtin_amdgcn_s_memrealtime() << 4) - CREDIT * TAU;
    auto pace = [&](u32 bytes) {
      while ((int)(((u32)__builtin_amdgcn_s_memrealtime() << 4) - due) < 0) {}
      due += bytes * tau >> 10;
    };
    int ct_ = 0, cp_ = 0;
    auto issue_c = [&]() {
      const char* sb = base + (size_t)(4 * ct_ + ct) * USTRIDE + U_C;
      const u32 la = LDS_CV + ct_ * CVS;
      const u32 lim = __builtin_amdgcn_readlane(ub, ct_) + 2048u;
      pace(cp_ == 2 ? 4u * (lim - 2048u) : 4096u);
      if (cp_ == 2 && lim > 3072u) {
        set_m0(la + 3072); glds<0, true>(sb, min(l16 + 3072u, lim + l3x));
        set_m0(la + 4096); if (lane < 16) glds<0, true>(sb, min(l16 + 4096u, lim + l3x));
      }
      set_m0(la + cp_ * 1024); glds<0, true>(sb, min(l16 + cp_ * 1024u, lim + l3x));
      if (++cp_ == 3) { cp_ = 0; ++ct_; }
    };
    TDECL;
    int arrived = 0;
    auto issue_n = [&](int s, int p) {
      const char* sb = base + (size_t)s * USTRIDE;
      pace(1024u);
      if (p == 0) {
        TACC(_ta);
        while (arrived < 4 * (s - 5)) arrived = __builtin_amdgcn_readfirstlane(*LP(volatile u32, L_ARR));
        TACC(_tb);
        set_m0(LDS_N + (s & 7) * PL); glds<0, false>(sb, l16);
      }
      if (p == 1) glds<1024, false>(sb, l16);
      if (p == 2) glds<2048, false>(sb, l16);
      if (p == 3) glds<3072, false>(sb, l16);
    };
    auto post = [&](u32 addr, u32 v) { *LP(volatile u32, addr) = v; };
    constexpr int NB = 24 - 3 * PRE;
    static_assert(PRE >= 2 && PRE <= 7 && W2 >= 4 && W2 <= 16 && NPUSH + NB + 4 <= NSL, "");
    int cdone = 0;
    auto landed_c = [&]() { ++cdone; if (cdone % 3 == 0) post(L_CVL, cdone / 3); };
    issue_c(); issue_c();
    bar();
    TSTART();
#pragma unroll 1
    for (int n = 0; n < 3 * PRE - 2; ++n) { waitv<1>(); issue_c(); landed_c(); }
    waitv<1>(); landed_c();
    u32 go;
    do go = __builtin_amdgcn_readfirstlane(*LP(volatile u32, L_NGO)); while (!go);
    if (go == 2) { waitv<0>(); post(L_CVL, 8); post(L_NL, NSL); return; }
#pragma unroll
    for (int p = 0; p < 4; ++p) issue_n(NPUSH, p);
    waitv<4>(); issue_c(); landed_c();
#pragma unroll 1
    for (int s = NPUSH; s < NPUSH + NB; ++s) {
#pragma unroll
      for (int p = 0; p < 4; ++p) { waitv<4>(); if (p == 3) post(L_NL, s + 1); issue_n(s + 1, p); }
      waitv<4>();
      if (s + 1 < NPUSH + NB) issue_c();
      landed_c();
    }
#pragma unroll
    for (int n = 4; n < W2; ++n) issue_n(NPUSH + NB + n / 4, n % 4);
    RMARK();
    constexpr int S0 = NSL - KEND, S1 = NSL - (W3 + 3) / 4;
    static_assert(S0 >= NPUSH + NB && S0 <= S1 && W3 >= W2 && W3 <= 16, "");
#pragma unroll 1
    for (int s = NPUSH + NB; s < S0; ++s) {
#pragma unroll
      for (int p = 0; p < 4; ++p) { waitv<W2 - 1>(); if (p == 3) post(L_NL, s + 1); issue_n(s + (p + W2) / 4, (p + W2) % 4); }
    }
#pragma unroll
    for (int n = W2; n < W3; ++n) issue_n(S0 + n / 4, n % 4);
#pragma unroll 1
    for (int s = S0; s < S1; ++s) {
#pragma unroll
      for (int p = 0; p < 4; ++p) { waitv<W3 - 1>(); if (p == 3) post(L_NL, s + 1); issue_n(s + (p + W3) / 4, (p + W3) % 4); }
    }
#pragma unroll
    for (int n = 0; n < 4 * (NSL - S1) - W3; ++n) { waitv<W3 - 1>(); issue_n(S1 + (n + W3) / 4, (n + W3) % 4); }
    if (NSL - S1 >= 4) { waitv<12>(); post(L_NL, NSL - 3); }
    if (NSL - S1 >= 3) { waitv<8>(); post(L_NL, NSL - 2); }
    if (NSL - S1 >= 2) { waitv<4>(); post(L_NL, NSL - 1); }
    waitv<0>(); post(L_NL, NSL);
    STAMP(4); TOUT(39, _tb / (NSL - NPUSH)); TOUT(12, _r1);
  }
}

extern "C" void kernel_launch(int rank, int nranks, ncclComm_t comm,
                              hipStream_t stream,
                              void* const* d_in, const int* in_sizes, int n_in,
                              void* d_out, int out_size,
                              const int* in_dtypes, int out_dtype,
                              void* const* d_out_all,
                              void* const* d_ws_all, size_t ws_bytes) {
  if (nranks != NR || (unsigned)rank >= (unsigned)NR || ws_bytes < WS_END || n_in < 4) return;
  static bool init = false;
  if (!init) {
    HIP_CHECK(hipFuncSetAttribute((const void*)ag_gemm, hipFuncAttributeMaxDynamicSharedMemorySize, LDS_TOT));
    init = true;
  }
  Args a;
  for (int p = 0; p < NR; ++p) a.ws[p] = (char*)d_ws_all[p];
  a.x = (const unsigned char*)d_in[0]; a.w = (const unsigned char*)d_in[1];
  a.sx = (const float*)d_in[2]; a.sw = (const float*)d_in[3];
  a.out = (float*)d_out; a.rank = rank; a.pad = 0;
  hipLaunchKernelGGL(ag_gemm, dim3(NWG), dim3(NTHR), LDS_TOT, stream, a);
  HIP_CHECK(hipGetLastError());
}
```

```cpp
#include <hip/hip_runtime.h>
#include <rccl/rccl.h>
#include <cstdio>
#include <cstdint>

#pragma clang diagnostic ignored "-Wint-to-pointer-cast"

#define HIP_CHECK(expr) do { hipError_t _e = (expr); if (_e != hipSuccess) { \
    fprintf(stderr, "HIP error %s at %s:%d\n", hipGetErrorString(_e), __FILE__, __LINE__); \
    return; } } while (0)

typedef unsigned u32;
typedef unsigned u32x2 __attribute__((ext_vector_type(2)));
typedef unsigned u32x4 __attribute__((ext_vector_type(4)));
typedef int v2i __attribute__((ext_vector_type(2)));
typedef int v8i __attribute__((ext_vector_type(8)));
typedef float v16f __attribute__((ext_vector_type(16)));
typedef unsigned long long u64;
#define LAS __attribute__((address_space(3)))
#define LP(T, a) ((LAS T*)(u32)(a))

#define TAU 222
#define CREDIT 4u
#define WB 5
#define WA 2
#define W3 W2
#define KEND 8
#define PRE 5
#define W2 6
#define PACE 60
#define NE 3
#define TREF 250
#define CMAX 60
#define PSLEEP 8
#define CSYS true
#define LOCF 2
#define CLIQUE 0x4D
#define NPUSH 3

constexpr int NR = 8;
constexpr int KTOT = 4096, NL = 512, MS = 512;
constexpr int BM = 64, BN = 128, KS = 128, NSL = KTOT / KS;
constexpr int NWG = 256, NTHR = 640;
constexpr int PL = 4096;
constexpr int U_C = 4096, U_H = 6144, U_V = 6160, USTRIDE = 9216;
constexpr int N0 = NPUSH + NE;
static_assert(NPUSH >= 1 && NPUSH <= 3 && NE >= 1 && N0 <= 8, "");
constexpr size_t WS_SLOT  = (size_t)NWG * USTRIDE;
constexpr size_t WS_PUSH  = 2 * WS_SLOT;
constexpr size_t PUSH_PAR = (size_t)NR * 8 * 8 * PL;
constexpr size_t WS_E     = WS_PUSH + 2 * PUSH_PAR;
constexpr size_t WS_FLAG  = WS_E + (size_t)64 * NSL * PL;
constexpr size_t FLAG_PAR = (size_t)NR * 8 * NSL * 128;
constexpr size_t WS_PFLAG = WS_FLAG + 2 * FLAG_PAR;
constexpr size_t WS_NFLAG = WS_PFLAG + NR * 8 * 128;
constexpr size_t WS_EFLAG = WS_NFLAG + NR * 8 * 8 * 128;
constexpr size_t WS_CNT   = WS_EFLAG + 64 * 512;
constexpr size_t WS_INV   = WS_CNT + NWG * 128;
constexpr size_t WS_TMO   = WS_INV + 16 * 128;
constexpr size_t WS_DBG   = WS_TMO + 128;
constexpr size_t WS_TR = WS_DBG + NWG * 64 * 8;
constexpr size_t WS_END   = WS_TR + NWG * 1024;
constexpr u64 DEADLINE = 30000000ull;
constexpr u64 SLOW = 20000ull;
constexpr int WSL = KS * BN;
constexpr int LDS_LUT = 0;
constexpr int LDS_CLUT = 128;
constexpr int LDS_MISC = 256;
constexpr int L_BAD = LDS_MISC, L_VAL = LDS_MISC + 4, L_NGO = LDS_MISC + 8, L_PUB = LDS_MISC + 12, L_LEN = LDS_MISC + 32;
constexpr int L_ARR = LDS_MISC + 16, L_NL = LDS_MISC + 20, L_RDY = LDS_MISC + 24, L_CVL = LDS_MISC + 28, L_DUMMY = LDS_MISC + 64;
constexpr int SCR = 2048;
constexpr int A3O = 1040, A4O = 1312, VBO = 1536;
constexpr int LDS_SCR = 512;
constexpr int LDS_E = 9 * 1024;
constexpr int LDS_N = LDS_E + 4 * PL;
constexpr int LDS_W = LDS_N + 8 * PL;
constexpr int CVS = 4352;
constexpr int LDS_CV = LDS_W + 4 * WSL;
constexpr int LDS_TR = LDS_CV + 8 * CVS;
constexpr int LDS_TOT = LDS_TR + 1024;
static_assert(LDS_SCR + 4 * SCR <= LDS_E && LDS_TOT <= 160 * 1024, "lds");

struct Args {
  char* ws[NR];
  const unsigned char* x; const unsigned char* w;
  const float* sx; const float* sw; float* out;
  int rank; u32 order;
};

#define STAMP(i) do {} while (0)
#define TDECL
#define TSTART() do {} while (0)
#define TACC(v) do {} while (0)
#define TOUT(i, v) do {} while (0)
#define RMARK() do {} while (0)

#define OKIND 3
#define OST(p, v) __hip_atomic_store((p), (v), __ATOMIC_RELAXED, __HIP_MEMORY_SCOPE_SYSTEM)
#define LNWG NWG
#define LNTHR NTHR
#define LLDS LDS_TOT
#define TR(i) do {} while (0)
#define SCHED() __builtin_amdgcn_sched_barrier(0)
template <int N> __device__ __forceinline__ void waitv() { asm volatile("s_waitcnt vmcnt(%0)" :: "n"(N) : "memory"); }
__device__ __forceinline__ void cfence() { asm volatile("" ::: "memory"); }
__device__ __forceinline__ void bar() {
  cfence(); SCHED();
  __builtin_amdgcn_s_waitcnt(0xC07F);
  __builtin_amdgcn_s_barrier();
  SCHED(); cfence();
}
__device__ __forceinline__ void set_m0(u32 lds_addr) { asm volatile("s_mov_b32 m0, %0\n\ts_nop 0" :: "s"(lds_addr) : "memory"); }
template <int OFF, bool SYS> __device__ __forceinline__ void glds(const void* sbase, u32 voff) {
  if (SYS) asm volatile("global_load_lds_dwordx4 %0, %1 offset:%2 sc0 sc1" :: "v"(voff), "s"(sbase), "n"(OFF) : "memory");
  else asm volatile("global_load_lds_dwordx4 %0, %1 offset:%2" :: "v"(voff), "s"(sbase), "n"(OFF) : "memory");
}
__device__ __forceinline__ void st16(void* p, u32x4 v) { asm volatile("global_store_dwordx4 %0, %1, off sc0 sc1\n\ts_nop 1" :: "v"(p), "v"(v) : "memory"); }
__device__ __forceinline__ void st8(void* p, u32x2 v) { asm volatile("global_store_dwordx2 %0, %1, off sc0 sc1\n\ts_nop 1" :: "v"(p), "v"(v) : "memory"); }
__device__ __forceinline__ void st4(void* p, u32 v) { asm volatile("global_store_dword %0, %1, off sc0 sc1\n\ts_nop 1" :: "v"(p), "v"(v) : "memory"); }
__device__ __forceinline__ u32 ld4(const void* p) {
  u32 v;
  asm volatile("global_load_dword %0, %1, off sc0 sc1\n\ts_waitcnt vmcnt(0)" : "=&v"(v) : "v"(p) : "memory");
  return v;
}

__device__ __forceinline__ bool expired(u64 t0, u32* tmo, u32 call) {
  const u64 dt = __builtin_amdgcn_s_memrealtime() - t0;
  if (dt <= SLOW) return false;
  if (dt > DEADLINE) { st4(tmo, call); return true; }
  __builtin_amdgcn_s_sleep(64);
  return __builtin_amdgcn_readfirstlane(ld4(tmo)) == call;
}

__device__ __forceinline__ u32 wave_scan(u32 x) {
  x += __builtin_amdgcn_update_dpp(0u, x, 0x111, 0xf, 0xf, true);
  x += __builtin_amdgcn_update_dpp(0u, x, 0x112, 0xf, 0xf, true);
  x += __builtin_amdgcn_update_dpp(0u, x, 0x114, 0xf, 0xf, true);
  x += __builtin_amdgcn_update_dpp(0u, x, 0x118, 0xf, 0xf, true);
  x += __builtin_amdgcn_update_dpp(0u, x, 0x142, 0xa, 0xf, false);
  x += __builtin_amdgcn_update_dpp(0u, x, 0x143, 0xc, 0xf, false);
  return x;
}
constexpr u32 B1 = 0x01010101u, M0F = 0x0F0F0F0Fu, MSM = 0x87878787u, MEX = 0x78787878u, M03 = 0x03030303u;
__device__ __forceinline__ u32 rotl4(u32 x) { return __builtin_amdgcn_alignbit(x, x, 28); }
__device__ __forceinline__ u32 rotr4(u32 x) { return __builtin_amdgcn_alignbit(x, x, 4); }
__device__ __forceinline__ u32x2 centry(u32 p) {
  return *LP(const u32x2, __builtin_amdgcn_udot4(p & 0x04040404u, 0x10080402u, (u32)LDS_CLUT, false));
}
__device__ __forceinline__ u32x2 dentry(u32 c, bool with_count = true) {
  const u32 p = __builtin_amdgcn_udot4(__builtin_amdgcn_perm(0u, 0x01000000u, c), 0x40201008u, (u32)LDS_LUT, false);
  if (with_count) return *LP(const u32x2, p);
  return u32x2{*LP(const u32, p), 0u};
}
__device__ __forceinline__ void put4(u32 o, u32 y) {
  const u64 w = (u64)y << ((o & 3u) * 8u);
  LAS u32* p = LP(u32, o & ~3u);
  __hip_atomic_fetch_or(p, (u32)w, __ATOMIC_RELAXED, __HIP_MEMORY_SCOPE_WAVEFRONT);
  __hip_atomic_fetch_or(p + 1, (u32)(w >> 32), __ATOMIC_RELAXED, __HIP_MEMORY_SCOPE_WAVEFRONT);
}
__device__ __forceinline__ u32x2 pair(u32 o) {
  const LAS u32* p = LP(const u32, o & ~3u);
  return u32x2{p[0], p[1]};
}
__device__ __forceinline__ u32 expand(u32x2 w, u32 o, u32 table, u32 sel) {
  return __builtin_amdgcn_perm(__builtin_amdgcn_alignbyte(w[1], w[0], o), table, sel);
}

__global__ void __launch_bounds__(NTHR, 3) ag_gemm(Args a) {
  const int tid = threadIdx.x, lane = tid & 63;
  const int wid = __builtin_amdgcn_readfirstlane(tid >> 6);
  const int b = blockIdx.x;
  const int rank = a.rank;
  char* const me = a.ws[rank];
  if (wid == 0) STAMP(0);
  if (wid == 0) TR(255);
  const int jp = b & 7, idx = b >> 3, ct = idx & 3;
  const int j = (jp & 3) * 2 + ((idx >> 2) & 1), q = (jp & 4) | (idx >> 3);
  const int src = (a.order >> (4 * q)) & (NR - 1);
  u32* const tmo = (u32*)(me + WS_TMO);
  const u32 dummy = L_DUMMY + (lane & 31) * 4;
  const u32 l16 = lane * 16u;
  char* const inv = me + WS_INV + (size_t)(__builtin_amdgcn_s_getreg((3 << 11) | 20) & 15u) * 128;
  u32* const cnt = (u32*)(me + WS_CNT + (size_t)b * 128);
  const int brow = (wid & 3) * 16 + (lane >> 2);
  const unsigned char* const xlane = a.x + (size_t)(jp * BM + brow) * KTOT + (((lane & 3) ^ ((brow >> 2) & 3)) << 5);

  if (wid < 4) {
    const int pe = idx >> 3;
    const bool early = pe == 3 && (idx & 7) < NE;
    const int ps = pe == 3 ? NPUSH + (idx & 7) : pe;
    auto nplane = [&](const u32x4 v0, const u32x4 v1) {
      u32x4 np;
#pragma unroll
      for (int k = 0; k < 4; ++k) np[k] = (v0[k] & MSM) | rotl4(v1[k] & MSM);
      return np;
    };
    const u32x4 pv0 = *(const u32x4*)(xlane + ps * KS), pv1 = *(const u32x4*)(xlane + ps * KS + 16);
    char* nflag = a.ws[lane & (NR - 1)] + WS_NFLAG + (size_t)((rank * 8 + jp) * 8 + (idx & 7)) * 128 + wid * 4;
    const u32 callv = __hip_atomic_load(cnt, __ATOMIC_RELAXED, __HIP_MEMORY_SCOPE_AGENT);
    if (tid < 16) {
      u32 add = 0, cs = 0x0C0C0C0Cu, n = 0;
#pragma unroll
      for (int i = 0; i < 4; ++i)
        if ((tid >> i) & 1) { add |= (1 + n) << (8 * i); cs = (cs & ~(0xFFu << (8 * n))) | (i << (8 * n)); ++n; }
      *LP(u32x2, LDS_LUT + tid * 8) = u32x2{add, n};
      *LP(u32x2, LDS_CLUT + tid * 8) = u32x2{cs, n};
    }
    const int kr = lane >> 3, cp = lane & 7;
    u32 wvo[4];
#pragma unroll
    for (int i = 0; i < 4; ++i) wvo[i] = (kr + i * 8) * NL + ((cp >> 1) ^ ((kr >> 1) & 3)) * 32 + (cp & 1) * 16 - i * 1024;
    const unsigned char* const wsrc = a.w + (size_t)(wid * 32) * NL + ct * BN;
    auto stage_w = [&](int s) {
      set_m0(LDS_W + (s & 3) * WSL + wid * 4096);
      const unsigned char* sb = wsrc + (size_t)s * KS * NL;
      glds<0, false>(sb, wvo[0]); glds<1024, false>(sb, wvo[1]); glds<2048, false>(sb, wvo[2]); glds<3072, false>(sb, wvo[3]);
    };
    char* const pdst = a.ws[(rank + idx) & (NR - 1)];
    bar();
    const u32 call = __builtin_amdgcn_readfirstlane(callv) + 1u;
    const size_t par = call & 1;
    const size_t poff = WS_PUSH + par * PUSH_PAR + (size_t)((rank * 8 + jp) * 8) * PL + wid * 1024 + l16;
    const size_t pfoff = WS_PFLAG + (size_t)(rank * 8 + jp) * 128 + wid * 4;
    const u32x4 np0 = nplane(pv0, pv1);
    asm volatile("" :: "v"(np0));
    const u64 tpush = __builtin_amdgcn_s_memrealtime();
    auto until = [&](int p) { while (__builtin_amdgcn_s_memrealtime() - tpush < (u64)(p * PACE)) __builtin_amdgcn_s_sleep(1); };
    asm volatile("" : "+v"(nflag));
    if (pe < NPUSH) { until(pe); st16(pdst + poff + pe * PL, np0); }
    if (early) {
      st16(me + par * WS_SLOT + (size_t)(jp * NSL + ps) * USTRIDE + wid * 1024 + l16, np0);
      waitv<0>();
      if (lane < NR) st4(nflag, call);
    }
    if (wid == 0) TR(244);
    float sc = a.sx[0] * a.sw[0];
    while (!__builtin_amdgcn_readfirstlane(*LP(volatile u32, L_NGO))) __builtin_amdgcn_s_sleep(4);
    waitv<0>();
    if (lane == 0 && pe < NPUSH) st4(pdst + pfoff + pe * 16, call);
    asm volatile("" : "+v"(sc));
    if (wid == 0) { STAMP(11); TR(245); }
    const char* const esrc = me + WS_E + (size_t)((q * 8 + j) * NSL) * PL + wid * 1024;
    const char* const psrc = me + WS_PUSH + par * PUSH_PAR + (size_t)((src * 8 + j) * 8) * PL + wid * 1024;
    auto stage_e = [&](int s, bool push) {
      if (push) { set_m0(LDS_N + (s & 7) * PL + wid * 1024); glds<0, true>(psrc + s * PL, l16); }
      set_m0(LDS_E + (s & 3) * PL + wid * 1024); glds<0, true>(esrc + s * PL, l16);
    };
    v16f acc[2] = {};
    const int r31 = lane & 31, h = lane >> 5, wm = wid >> 1, wn = wid & 1;
    const int q8 = (lane & 15) >> 1, p8 = lane & 1, g1 = (lane >> 4) & 1;
    u32 boff[2], aoff[2];
#pragma unroll
    for (int i = 0; i < 2; ++i) {
      boff[i] = LDS_W + (32 * h + q8) * 128 + (((wn * 2 + i) ^ ((q8 >> 1) & 3)) << 5) + g1 * 16 + p8 * 8;
      aoff[i] = (wm * 32 + r31) * 64 + (((i * 2 + h) ^ ((r31 >> 2) & 3)) << 4);
    }
    TDECL;
    struct Frag { u32x4 ep[2]; v8i bf[2][2]; };
    auto ldfrag = [&](Frag& f, int kk, int s) {
      f.ep[kk] = *LP(const u32x4, aoff[kk] + (s & 3) * PL + LDS_E);
#pragma unroll
      for (int n = 0; n < 2; ++n) {
        v2i t[4];
#pragma unroll
        for (int r = 0; r < 4; ++r) t[r] = __builtin_amdgcn_ds_read_tr8_b64_v2i32(LP(v2i, boff[n] + (s & 3) * WSL) + (kk * 64 + r * 8) * 16);
        f.bf[kk][n] = v8i{t[0][0], t[0][1], t[1][0], t[1][1], t[2][0], t[2][1], t[3][0], t[3][1]};
      }
    };
    auto afrag = [&](const u32x4 np, const u32x4 ep) {
      v8i af;
#pragma unroll
      for (int k = 0; k < 4; ++k) {
        af[k] = (np[k] & MSM) | (ep[k] & MEX);
        af[4 + k] = rotr4((np[k] & MEX) | (ep[k] & MSM));
      }
      return af;
    };
#define MFMA(n, af, bf) acc[n] = __builtin_amdgcn_mfma_scale_f32_32x32x64_f8f6f4(af, bf, acc[n], 0, 0, 0, 0, 0, 0)
    auto step = [&](int s, const Frag& cur, Frag& nxt, bool more, bool stg) {
      const u32x4 np0 = *LP(const u32x4, aoff[0] + (s & 7) * PL + LDS_N);
      const u32x4 np1 = *LP(const u32x4, aoff[1] + (s & 7) * PL + LDS_N);
      SCHED();
      if (stg) { stage_e(s + 4, s + 4 < NPUSH); stage_w(s + 4); }
      SCHED();
      const v8i a0 = afrag(np0, cur.ep[0]);
      MFMA(0, a0, cur.bf[0][0]);
      SCHED();
      const v8i a1 = afrag(np1, cur.ep[1]);
      SCHED();
      MFMA(1, a0, cur.bf[0][1]);
      SCHED();
      if (more) ldfrag(nxt, 0, s + 1);
      SCHED();
      MFMA(0, a1, cur.bf[1][0]);
      SCHED();
      if (more) ldfrag(nxt, 1, s + 1);
      SCHED();
      MFMA(1, a1, cur.bf[1][1]);
      SCHED();
      asm volatile("" :: "v"(a0), "v"(cur.bf[0][0]), "v"(cur.bf[0][1]), "v"(cur.bf[1][0]));
    };
    const u32 arr = lane ? dummy : (u32)L_ARR;
    auto spin = [&](int na, int nl, int nr) {
      while (true) {
        u32x4 w;
        asm volatile("ds_read_b128 %0, %1\n\ts_waitcnt lgkmcnt(0)" : "=&v"(w) : "v"((u32)L_ARR) : "memory");
        if (__all((int)w[0] >= na && (int)w[1] >= nl && (int)w[2] >= nr)) break;
      }
    };
    auto meet = [&](int s) {
      SCHED();
      asm volatile("s_waitcnt lgkmcnt(0)\n\tds_add_u32 %0, %1" :: "v"(arr), "v"(1u) : "memory");
      if (wid == 0 && s >= 0) TR(96 + s);
      spin(4 * s + 8, s + 1, s == NSL - 1 ? NSL + 1 : min(s + 5, NSL));
      if (wid == 0 && s >= 0) TR(32 + s);
      __builtin_amdgcn_s_waitcnt(0xC07F);
      SCHED();
    };
    {
      Frag fa, fb;
#pragma unroll 1
      for (int s = 0; s < 4; ++s) { stage_w(s); __builtin_amdgcn_s_sleep(8); }
      spin(0, 0, 4);
#pragma unroll 1
      for (int s = 0; s < 4; ++s) stage_e(s, s < NPUSH);
      waitv<0>(); meet(-1);
      ldfrag(fa, 0, 0); ldfrag(fa, 1, 0);
      meet(0); step(0, fa, fb, true, true);
      TSTART();
#pragma unroll 1
      for (int s = 1; s < NSL - 5; s += 2) {
        if (s > 9) TACC(_ta); else TACC(_tb);
        waitv<10>(); meet(s);
        if (s >= 9) TACC(_tc); else TACC(_tb);
        step(s, fb, fa, true, true);
        if (s >= 9) TACC(_ta); else TACC(_tb);
        waitv<10>(); meet(s + 1);
        if (s >= 9) TACC(_tc); else TACC(_tb);
        step(s + 1, fa, fb, true, true);
      }
      waitv<10>(); meet(NSL - 5); step(NSL - 5, fb, fa, true, true);
      waitv<10>(); meet(NSL - 4); step(NSL - 4, fa, fb, true, false);
      waitv<5>(); meet(NSL - 3); step(NSL - 3, fb, fa, true, false);
      waitv<0>(); meet(NSL - 2); step(NSL - 2, fa, fb, true, false);
      meet(NSL - 1); step(NSL - 1, fb, fa, false, false);
    }
    if (*LP(volatile u32, L_BAD)) sc = __builtin_nanf("");
    if (tid == 0) st4(cnt, call);
    if (wid == 0) { STAMP(5); TOUT(35, _ta / 17); TOUT(37, _tc / 18); }
    float* o = a.out + (size_t)(src * MS + j * BM + wm * 32 + 4 * h) * NL + ct * BN + wn * 64 + r31;
#pragma unroll
    for (int i = 0; i < 16; ++i)
#pragma unroll
      for (int n = 0; n < 2; ++n)
        OST(o + (size_t)((i & 3) + 8 * (i >> 2)) * NL + n * 32, acc[n][i] * sc);
  } else if (wid < 8) {
    const int lw = wid - 4;
    const u32 scr = LDS_SCR + lw * SCR;
    u32 call;
    {
      const u32x4 v0 = *(const u32x4*)(xlane + idx * KS);
      const u32x4 v1 = *(const u32x4*)(xlane + idx * KS + 16);
      const u32 callv = __hip_atomic_load(cnt, __ATOMIC_RELAXED, __HIP_MEMORY_SCOPE_AGENT);
      char* fdst = a.ws[lane & (NR - 1)] + WS_FLAG + (size_t)((rank * 8 + jp) * NSL + idx) * 128;
      *LP(u32x4, scr + l16) = u32x4{0, 0, 0, 0};
      *LP(u32x2, scr + 1024 + lane * 8) = u32x2{0, 0};
      bar();
      call = __builtin_amdgcn_readfirstlane(callv) + 1u;
      asm volatile("" : "+v"(fdst));
      if (lw == 0) { STAMP(9); TR(242); }
      char* const dst = me + (size_t)(call & 1) * WS_SLOT + (size_t)(jp * NSL + idx) * USTRIDE;
      const u32 vbuf = scr + VBO;
      const u32 v[8] = {v0[0], v0[1], v0[2], v0[3], v1[0], v1[1], v1[2], v1[3]};
      u32 z[8], sum = 0;
#pragma unroll
      for (int k = 0; k < 8; ++k) { z[k] = v[k] & MEX; if (k & 1) sum = __builtin_amdgcn_udot4(z[k], B1, sum, false); }
      sum = __builtin_amdgcn_readlane(wave_scan(sum), 63);
      const u32 bb = ((sum + 7946u) >> 13) & 15u;
      const u32 kz = ((7u - bb) & 15u) * 0x08080808u;
      u32x4 np;
#pragma unroll
      for (int k = 0; k < 4; ++k) np[k] = (v[k] & MSM) | rotl4(v[4 + k] & MSM);
      if (idx >= N0) st16(dst + lw * 1024 + l16, np);
      u32x2 e1[8], cp2 = {0, 0};
#pragma unroll
      for (int k = 0; k < 8; ++k) {
        z[k] = (((z[k] + kz) >> 3) & M0F) ^ 0x04040404u;
        const u32 p = __builtin_amdgcn_perm(0xFFFFFFFFu, 0x8201FFFFu, z[k]);
        cp2[k >> 2] |= (p & M03) << (2 * (k & 3));
        e1[k] = centry(p);
      }
      const u32 t1 = (e1[0][1] + e1[1][1] + e1[2][1] + e1[3][1]) | ((e1[4][1] + e1[5][1] + e1[6][1] + e1[7][1]) << 16);
      const u32 sc1 = wave_scan(t1);
      const u32 tot1 = __builtin_amdgcn_readlane(sc1, 63);
      const u32 n2 = (tot1 & 0xFFFFu) + (tot1 >> 16);
      bool raw = n2 > 1024u;
      u32 n2d = 0, n3d = 0, n4d = 0;
      if (!raw) {
        const u32 ex = sc1 - t1 + (scr | ((scr + (tot1 & 0xFFFFu)) << 16));
        u32 o = ex & 0xFFFFu;
#pragma unroll
        for (int k = 0; k < 8; ++k) {
          if (k == 4) o = ex >> 16;
          put4(o, __builtin_amdgcn_perm(0u, z[k], e1[k][0])); o += e1[k][1];
        }
        cfence();
        const u32x4 q2 = *LP(const u32x4, scr + l16);
        u32 l2 = 0; u32x2 e2[4];
#pragma unroll
        for (int k = 0; k < 4; ++k) {
          const u32 p = __builtin_amdgcn_perm(0xFFFFFFFFu, 0xFFFF8201u, q2[k] ^ B1);
          l2 |= (p & M03) << (2 * k); e2[k] = centry(p);
        }
        *LP(u32, vbuf + lane * 4) = l2;
        const u32 t2 = e2[0][1] + e2[1][1] + e2[2][1] + e2[3][1];
        const u32 sc2 = wave_scan(t2);
        const u32 n3 = __builtin_amdgcn_readlane(sc2, 63);
        raw = n3 > 256u;
        if (!raw) {
          n2d = (n2 + 15u) >> 4; n3d = (n3 + 15u) >> 4;
          if (t2) {
            o = scr + A3O + sc2 - t2;
#pragma unroll
            for (int k = 0; k < 4; ++k) { put4(o, __builtin_amdgcn_perm(0u, q2[k] ^ 0x05050505u, e2[k][0])); o += e2[k][1]; }
          }
          cfence();
          const u32 q3 = *LP(const u32, scr + A3O + lane * 4) ^ 0x05050505u;
          const u32 p3 = __builtin_amdgcn_perm(0x800182FFu, 0xFFFFFFFFu, q3);
          u32 l3 = (p3 & M03) << (2 * (lane & 3));
          l3 |= __builtin_amdgcn_update_dpp(0u, l3, 0xB1, 0xf, 0xf, false);
          l3 |= __builtin_amdgcn_update_dpp(0u, l3, 0x4E, 0xf, 0xf, false);
          *LP(u32, vbuf + (n2d + (lane >> 2)) * 4) = l3;
          if (__any(p3 & 0x04040404u)) {
            const u32x2 e3 = centry(p3);
            const u32 sc3 = wave_scan(e3[1]);
            const u32 n4 = __builtin_amdgcn_readlane(sc3, 63);
            raw = n4 > 128u;
            if (!raw) {
              if (e3[1]) put4(scr + A4O + sc3 - e3[1], __builtin_amdgcn_perm(0u, q3, e3[0]));
              cfence();
              const u32x2 q4 = *LP(const u32x2, scr + A4O + (lane & 15) * 8);
              n4d = (n4 + 7u) >> 3;
              if (lane < 16) *LP(u32, vbuf + (n2d + n3d + lane) * 4) = q4[0] | (q4[1] << 4);
            }
          }
        }
      }
      cfence();
      u32 vd = n2d + n3d + n4d;
      if (raw) {
        cp2 = u32x2{(v[0] & MEX) | rotl4(v[4] & MEX), (v[1] & MEX) | rotl4(v[5] & MEX)}; vd = 128;
        *LP(u32, vbuf + lane * 4) = (v[2] & MEX) | rotl4(v[6] & MEX);
        *LP(u32, vbuf + 256 + lane * 4) = (v[3] & MEX) | rotl4(v[7] & MEX);
      }
      cfence();
      u32 voff;
      asm volatile("ds_add_rtn_u32 %0, %1, %2\n\ts_waitcnt lgkmcnt(0)" : "=&v"(voff) : "v"(lane ? dummy : (u32)L_VAL), "v"(vd) : "memory");
      const u32 va = *LP(const u32, vbuf + lane * 4), vb = *LP(const u32, vbuf + 256 + lane * 4);
      st8(dst + U_C + lw * 512 + lane * 8, cp2);
      voff = __builtin_amdgcn_readfirstlane(voff);
      if ((u32)lane < vd) st4(dst + U_V + (voff + lane) * 4, va);
      if ((u32)lane + 64u < vd) st4(dst + U_V + (voff + 64 + lane) * 4, vb);
      if (lane == 0) st4(dst + U_H + lw * 4, voff | (n2d << 10) | (n3d << 17) | (n4d << 22) | (bb << 27) | (raw ? 0x80000000u : 0u));
      if (lw == 0) { STAMP(10); TR(243); }
      waitv<0>();
      u32 nth;
      asm volatile("ds_add_rtn_u32 %0, %1, %2\n\ts_waitcnt lgkmcnt(0)" : "=&v"(nth) : "v"(lane ? dummy : (u32)L_PUB), "v"(1u) : "memory");
      if (__builtin_amdgcn_readfirstlane(nth) == 3) {
        const u32 vtot = *LP(volatile u32, L_VAL);
        if (lane < NR) st4(fdst + (size_t)(call & 1) * FLAG_PAR, (call << 6) | ((16u + vtot * 4u + 63u) >> 6));
        STAMP(1); TR(241);
      }
    }
    char* const edst = me + WS_E + (size_t)((q * 8 + j) * NSL + ct) * PL + lw * 1024 + l16;
    char* const eflag = me + WS_EFLAG + (size_t)(q * 8 + j) * 512 + ct * 16 + lw * 4;
    int pend = -1;
    TDECL;
#pragma unroll 1
    for (int t = 0; t < 8; ++t) {
      if ((int)__builtin_amdgcn_readfirstlane(*LP(volatile u32, L_CVL)) <= t) {
        if (pend >= 0) { waitv<0>(); if (lane == 0) st4(eflag + pend * 64, call); pend = -1; }
        while ((int)__builtin_amdgcn_readfirstlane(*LP(volatile u32, L_CVL)) <= t) __builtin_amdgcn_s_sleep(2);
      }
      cfence();
      TSTART();
      const u32 S = LDS_CV + t * CVS;
      const u32 info = __builtin_amdgcn_readfirstlane(*LP(const u32, S + 2048 + lw * 4));
      const u32x2 cc = *LP(const u32x2, S + lw * 512 + lane * 8);
      const u32 V = S + 2064 + (info & 0x1FFu) * 4;
      const u32 n2d = (info >> 10) & 0x7Fu, n3d = (info >> 17) & 0x1Fu, bb = (info >> 27) & 15u;
      u32 l2 = *LP(const u32, V + lane * 4);
      u32x4 ep;
      if (info >> 31) {
        ep = u32x4{cc[0], cc[1], l2, *LP(const u32, V + 256 + lane * 4)};
      } else {
        const u32 l3 = *LP(const u32, V + n2d * 4 + (lane >> 2) * 4);
        u32 c1[8], c2[4]; u32x2 e1[8], e2[4];
#pragma unroll
        for (int k = 0; k < 8; ++k) { c1[k] = (cc[k >> 2] >> (2 * (k & 3))) & M03; e1[k] = dentry(c1[k], (k & 3) != 3); }
        SCHED();
        l2 = (u32)lane < n2d ? l2 : 0u;
        u32 w4 = 0, c3 = (l3 >> (2 * (lane & 3))) & M03;
#pragma unroll
        for (int k = 0; k < 4; ++k) { c2[k] = (l2 >> (2 * k)) & M03; e2[k] = dentry(c2[k], k != 3); }
        SCHED();
        const u32 bbv = bb * B1;
        if (info & (0x1Fu << 22)) {
          c3 = (u32)lane < 4 * n3d ? c3 : 0u;
          const u32x2 e3 = dentry(c3);
          const u32 l4 = *LP(const u32, V + (n2d + n3d + (lane & 15)) * 4) ^ 0x44444444u;
          const u32 K4 = ((bb + 9u) & 15u) * B1;
          *LP(u32x2, scr + A4O + (lane & 15) * 8) = u32x2{(((l4 & M0F) + K4) & M0F) << 3, ((((l4 >> 4) & M0F) + K4) & M0F) << 3};
          cfence();
          const u32 o4 = scr + A4O + min(wave_scan(e3[1]) - e3[1], 128u);
          const u32x2 ww = pair(o4);
          w4 = __builtin_amdgcn_alignbyte(ww[1], ww[0], o4);
          c3 += e3[0];
        }
        *LP(u32, scr + A3O + lane * 4) = __builtin_amdgcn_perm(w4, ((bbv + 0x000A0B0Cu) & M0F) << 3, c3);
        const u32 ta = __builtin_popcount(cc[0] & (cc[0] >> 1) & 0x55555555u), tb = __builtin_popcount(cc[1] & (cc[1] >> 1) & 0x55555555u);
        const u32 t2 = __builtin_popcount(l2 & (l2 >> 1) & 0x55555555u);
        const u32 tt = ta | (tb << 11) | (t2 << 22);
        const u32 sc = wave_scan(tt);
        const u32 na = __builtin_amdgcn_readlane(sc, 63) & 0x7FFu;
        const u32 ex = sc - tt;
        cfence();
        u32 o2[4], o1[8]; u32x2 w[8];
        o2[0] = (ex >> 22) + (scr + A3O);
#pragma unroll
        for (int k = 1; k < 4; ++k) o2[k] = o2[k - 1] + e2[k - 1][1];
#pragma unroll
        for (int k = 0; k < 4; ++k) w[k] = pair(o2[k]);
        SCHED();
        o1[0] = (ex & 0x7FFu) + scr; o1[4] = ((ex >> 11) & 0x7FFu) + (scr + na);
#pragma unroll
        for (int k = 1; k < 8; ++k) if (k != 4) o1[k] = o1[k - 1] + e1[k - 1][1];
#pragma unroll
        for (int k = 0; k < 4; ++k) c2[k] += e2[k][0];
        SCHED();
        u32x4 y;
#pragma unroll
        for (int k = 0; k < 4; ++k) y[k] = expand(w[k], o2[k], ((bbv + 0x000D0E02u) & M0F) << 3, c2[k]);
        *LP(u32x4, scr + l16) = y;
        cfence();
#pragma unroll
        for (int k = 0; k < 8; ++k) w[k] = pair(o1[k]);
        SCHED();
#pragma unroll
        for (int k = 0; k < 8; ++k) c1[k] += e1[k][0];
        SCHED();
        const u32 T1 = ((bbv + 0x00000F01u) & M0F) << 3;
#pragma unroll
        for (int k = 0; k < 4; ++k) ep[k] = expand(w[k], o1[k], T1, c1[k]) | rotl4(expand(w[4 + k], o1[4 + k], T1, c1[4 + k]));
      }
      st16(edst + (size_t)t * 4 * PL, ep);
      if (lw == 0) TR(72 + t);
      TACC(_ta);
      if (pend >= 0) { waitv<1>(); if (lane == 0) st4(eflag + pend * 64, call); }
      pend = t;
      TACC(_tb);
    }
    waitv<0>();
    if (lane == 0) st4(eflag + 7 * 64, call);
    if (lw == 0) { STAMP(8); TOUT(32, _ta / 8); TOUT(33, _tb / 8); }
  } else if (wid == 9) {
    if (b < 8) asm volatile("buffer_inv sc0 sc1" ::: "memory");
    bar();
    const u32 call = __builtin_amdgcn_readfirstlane(ld4(cnt)) + 1u;
    if (b < 8 && lane < 4) st4(inv + lane * 4, call);
    while (__builtin_amdgcn_readfirstlane(*LP(volatile u32, L_PUB)) < 4) __builtin_amdgcn_s_sleep(4);
    const u64 t0 = __builtin_amdgcn_s_memrealtime();
    {
      const char* f = me + WS_FLAG + (size_t)(call & 1) * FLAG_PAR + (size_t)((src * 8 + j) * NSL + (lane & 31)) * 128;
      u32 go = 1, fv;
      while (true) {
        fv = ld4(f);
        const bool ok = (fv >> 6) >= (call & 0x3FFFFFFu);
        if ((lane & 35) == ct && ok) *LP(volatile u32, L_LEN + (lane >> 2) * 4) = max(min(fv & 63u, 33u), 1u) * 64u + 2048u;
        if (__all(ok)) break;
        if (expired(t0, tmo, call)) { go = 2; *LP(volatile u32, L_BAD) = 1; break; }
        __builtin_amdgcn_s_sleep(2);
      }
      *LP(volatile u32, L_NGO) = go;
      TR(246);
    }
    const char* const f = lane < 32 ? me + WS_EFLAG + (size_t)(q * 8 + j) * 512 + l16
                                    : me + WS_PFLAG + (size_t)(src * 8 + j) * 128 + (lane & 7) * 16;
    while (true) {
      u32x4 v;
      asm volatile("global_load_dwordx4 %0, %1, off sc0 sc1\n\ts_waitcnt vmcnt(0)" : "=&v"(v) : "v"(f) : "memory");
      const u64 known = __ballot(min(min(v[0], v[1]), min(v[2], v[3])) >= call);
      const int nready = __builtin_ctzll(~(u64)((u32)known & ((u32)(known >> 32) | (~0u << NPUSH))));
      if (nready == NSL) break;
      *LP(volatile u32, L_RDY) = nready;
      if (expired(t0, tmo, call)) break;
      __builtin_amdgcn_s_sleep(PSLEEP);
    }
    *LP(volatile u32, L_RDY) = NSL;
    if (__builtin_amdgcn_readfirstlane(ld4(tmo)) == call) *LP(volatile u32, L_BAD) = 1;
    *LP(volatile u32, L_RDY) = NSL + 1;
    STAMP(13);
  } else {
    __builtin_amdgcn_s_setprio(3);
    const u32 tstart = (u32)__builtin_amdgcn_s_memrealtime();
    *LP(u32x4, LDS_MISC) = u32x4{0, 0, 0, 0};
    *LP(u32x4, LDS_MISC + 16) = u32x4{0, NPUSH, 0, 0};
    *LP(u32x4, L_LEN) = u32x4{0, 0, 0, 0}; *LP(u32x4, L_LEN + 16) = u32x4{0, 0, 0, 0};
    bar();
    const u32 call = __builtin_amdgcn_readfirstlane(ld4(cnt)) + 1u;
    {
      const char* f = lane < NE ? me + WS_NFLAG + (size_t)((src * 8 + j) * 8 + lane) * 128 : inv;
      const u64 t0 = __builtin_amdgcn_s_memrealtime();
      while (true) {
        u32x4 v;
        asm volatile("global_load_dwordx4 %0, %1, off sc0 sc1\n\ts_waitcnt vmcnt(0)" : "=&v"(v) : "v"(f) : "memory");
        const u64 ok = __ballot(min(min(v[0], v[1]), min(v[2], v[3])) >= call);
        if (ok == ~0ull) break;
        if ((~ok & ((1u << NE) - 1)) == 0 && __builtin_amdgcn_s_memrealtime() - t0 > 500) {
          asm volatile("buffer_inv sc0 sc1\n\ts_waitcnt vmcnt(0)" ::: "memory");
          break;
        }
        if (expired(t0, tmo, call)) {
          *LP(volatile u32, L_BAD) = 1; *LP(volatile u32, L_CVL) = 8; *LP(volatile u32, L_NL) = NSL;
          return;
        }
        __builtin_amdgcn_s_sleep(2);
      }
      STAMP(2); TR(240);
      waitv<0>();
    }
    u32 lim = 0;
    bool dead = false;
    const char* const base = a.ws[src] + (size_t)(call & 1) * WS_SLOT + (size_t)(j * NSL) * USTRIDE;
    const u32 tau = q ? TAU : TAU * LOCF / 8;
    u32 due = (tstart + (u32)max((int)((u32)__builtin_amdgcn_s_memrealtime() - tstart) - CMAX, TREF)) << 4;
    auto pace = [&](u32 bytes) {
      while ((int)(((u32)__builtin_amdgcn_s_memrealtime() << 4) - due) < 0) {}
      due += bytes * tau >> 10;
    };
    int ct_ = 0, cp_ = 0;
    auto issue_c = [&]() {
      const char* sb = base + (size_t)(4 * ct_ + ct) * USTRIDE + U_C;
      const u32 la = LDS_CV + ct_ * CVS;
      if (cp_ == 0) {
        while (!(lim = __builtin_amdgcn_readfirstlane(*LP(volatile u32, L_LEN + ct_ * 4))))
          if (__builtin_amdgcn_readfirstlane(*LP(volatile u32, L_NGO)) == 2) { dead = true; break; }
        if (ct_ == 0) { const u32 now = ((u32)__builtin_amdgcn_s_memrealtime() - CMAX) << 4; if ((int)(now - due) > 0) due = now; TR(247); }
      }
      if (dead) return;
      pace(cp_ == 2 ? 4u * (lim - 2048u) : 4096u);
      if (cp_ == 2 && lim > 3072u) {
        set_m0(la + 3072); if (l16 + 3072u < lim) glds<0, CSYS>(sb, l16 + 3072u);
        set_m0(la + 4096); if (l16 + 4096u < lim) glds<0, CSYS>(sb, l16 + 4096u);
      }
      set_m0(la + cp_ * 1024);
      if (l16 + cp_ * 1024u < lim) glds<0, CSYS>(sb, l16 + cp_ * 1024u);
      if (++cp_ == 3) { cp_ = 0; ++ct_; }
    };
    TDECL;
    int arrived = 0;
    auto issue_n = [&](int s, int p) {
      const char* sb = base + (size_t)s * USTRIDE;
      pace(1024u);
      if (p == 0) {
        TACC(_ta);
        while (s >= 8 && arrived < 4 * (s - 5)) arrived = __builtin_amdgcn_readfirstlane(*LP(volatile u32, L_ARR));
        TACC(_tb);
        set_m0(LDS_N + (s & 7) * PL); glds<0, false>(sb, l16);
      }
      if (p == 1) glds<1024, false>(sb, l16);
      if (p == 2) glds<2048, false>(sb, l16);
      if (p == 3) glds<3072, false>(sb, l16);
    };
    auto post = [&](u32 addr, u32 v) { *LP(volatile u32, addr) = v; TR((addr == L_NL ? 0 : 64) + v - 1); };
    constexpr int NB = 24 - 3 * PRE;
    static_assert(PRE >= 2 && PRE <= 7 && W2 >= 4 && W2 <= 16 && N0 + NB + 4 <= NSL, "");
    int cdone = 0;
    auto landed_c = [&]() { if (cdone < 16) TR(224 + cdone); ++cdone; if (cdone % 3 == 0) post(L_CVL, cdone / 3); };
#pragma unroll 1
    for (int s = NPUSH; s < N0; ++s) {
#pragma unroll
      for (int p = 0; p < 4; ++p) { waitv<W2 - 1>(); issue_n(s, p); }
    }
    waitv<3>(); issue_c();
    waitv<1>(); post(L_NL, N0); issue_c();
    TSTART();
#pragma unroll 1
    for (int n = 0; n < 3 * PRE - 2; ++n) { waitv<1>(); issue_c(); landed_c(); }
    waitv<1>(); landed_c();
    u32 go;
    do go = __builtin_amdgcn_readfirstlane(*LP(volatile u32, L_NGO)); while (!go);
    if (go == 2) { waitv<0>(); post(L_CVL, 8); post(L_NL, NSL); return; }
#pragma unroll
    for (int p = 0; p < 4; ++p) issue_n(N0, p);
    waitv<4>(); issue_c(); landed_c();
#pragma unroll 1
    for (int s = N0; s < N0 + NB; ++s) {
#pragma unroll
      for (int p = 0; p < 4; ++p) { waitv<4>(); if (p == 3) post(L_NL, s + 1); issue_n(s + 1, p); }
      waitv<4>();
      if (s + 1 < N0 + NB) issue_c();
      landed_c();
    }
#pragma unroll
    for (int n = 4; n < W2; ++n) issue_n(N0 + NB + n / 4, n % 4);
    RMARK();
    constexpr int S0 = NSL - KEND, S1 = NSL - (W3 + 3) / 4;
    static_assert(S0 >= N0 + NB && S0 <= S1 && W3 >= W2 && W3 <= 16, "");
#pragma unroll 1
    for (int s = N0 + NB; s < S0; ++s) {
#pragma unroll
      for (int p = 0; p < 4; ++p) { waitv<W2 - 1>(); if (p == 3) post(L_NL, s + 1); issue_n(s + (p + W2) / 4, (p + W2) % 4); }
    }
#pragma unroll
    for (int n = W2; n < W3; ++n) issue_n(S0 + n / 4, n % 4);
#pragma unroll 1
    for (int s = S0; s < S1; ++s) {
#pragma unroll
      for (int p = 0; p < 4; ++p) { waitv<W3 - 1>(); if (p == 3) post(L_NL, s + 1); issue_n(s + (p + W3) / 4, (p + W3) % 4); }
    }
#pragma unroll
    for (int n = 0; n < 4 * (NSL - S1) - W3; ++n) { waitv<W3 - 1>(); issue_n(S1 + (n + W3) / 4, (n + W3) % 4); }
    if (NSL - S1 >= 4) { waitv<12>(); post(L_NL, NSL - 3); }
    if (NSL - S1 >= 3) { waitv<8>(); post(L_NL, NSL - 2); }
    if (NSL - S1 >= 2) { waitv<4>(); post(L_NL, NSL - 1); }
    waitv<0>(); post(L_NL, NSL);
    STAMP(4); TOUT(39, _tb / (NSL - NPUSH)); TOUT(12, _r1);
  }
}

extern "C" void kernel_launch(int rank, int nranks, ncclComm_t comm,
                              hipStream_t stream,
                              void* const* d_in, const int* in_sizes, int n_in,
                              void* d_out, int out_size,
                              const int* in_dtypes, int out_dtype,
                              void* const* d_out_all,
                              void* const* d_ws_all, size_t ws_bytes) {
  if (nranks != NR || (unsigned)rank >= (unsigned)NR || ws_bytes < WS_END || n_in < 4) return;
  static bool init = false;
  if (!init) {
    HIP_CHECK(hipFuncSetAttribute((const void*)ag_gemm, hipFuncAttributeMaxDynamicSharedMemorySize, LDS_TOT));
    init = true;
  }
  Args a;
  for (int p = 0; p < NR; ++p) a.ws[p] = (char*)d_ws_all[p];
  a.x = (const unsigned char*)d_in[0]; a.w = (const unsigned char*)d_in[1];
  a.sx = (const float*)d_in[2]; a.sw = (const float*)d_in[3];
  a.out = (float*)d_out; a.rank = rank;
  {
    const unsigned cl = CLIQUE;
    unsigned o = 0; int n = 0;
    for (int pass = 0; pass < 2; ++pass)
      for (int i = 0; i < NR; ++i) { const int p = (rank + i) & 7; if ((int)(((cl >> p) ^ (cl >> rank)) & 1) == pass) o |= (unsigned)p << (4 * n++); }
    a.order = o;
  }
  hipLaunchKernelGGL(ag_gemm, dim3(LNWG), dim3(LNTHR), LLDS, stream, a);
  HIP_CHECK(hipGetLastError());
}
```

```cpp
#include <hip/hip_runtime.h>
#include <rccl/rccl.h>
#include <cstdio>
#include <cstdint>

#pragma clang diagnostic ignored "-Wint-to-pointer-cast"

#define HIP_CHECK(expr) do { hipError_t _e = (expr); if (_e != hipSuccess) { \
    fprintf(stderr, "HIP error %s at %s:%d\n", hipGetErrorString(_e), __FILE__, __LINE__); \
    return; } } while (0)

typedef unsigned u32;
typedef unsigned u32x2 __attribute__((ext_vector_type(2)));
typedef unsigned u32x4 __attribute__((ext_vector_type(4)));
typedef int v2i __attribute__((ext_vector_type(2)));
typedef int v8i __attribute__((ext_vector_type(8)));
typedef float v16f __attribute__((ext_vector_type(16)));
typedef unsigned long long u64;
#define LAS __attribute__((address_space(3)))
#define LP(T, a) ((LAS T*)(u32)(a))

#define TAU 222
#define CREDIT 4u
#define WB 5
#define WA 2
#define W3 W2
#define KEND 8
#define PRE 5
#define W2 6
#define PACE 60
#define NE 3
#define TREF 250
#define CMAX 60
#define PSLEEP 8
#define CSYS true
#define LOCF 2
#define CLIQUE 0x4D
#define NPUSH 3

constexpr int NR = 8;
constexpr int KTOT = 4096, NL = 512, MS = 512;
constexpr int BM = 64, BN = 128, KS = 128, NSL = KTOT / KS;
constexpr int NWG = 256, NTHR = 640;
constexpr int PL = 4096;
constexpr int U_C = 4096, U_H = 6144, U_V = 6160, USTRIDE = 9216;
constexpr int N0 = NPUSH + NE;
static_assert(NPUSH >= 1 && NPUSH <= 3 && NE >= 1 && N0 <= 8, "");
constexpr size_t WS_SLOT  = (size_t)NWG * USTRIDE;
constexpr size_t WS_PUSH  = 2 * WS_SLOT;
constexpr size_t PUSH_PAR = (size_t)NR * 8 * 8 * PL;
constexpr size_t WS_E     = WS_PUSH + 2 * PUSH_PAR;
constexpr size_t WS_FLAG  = WS_E + (size_t)64 * NSL * PL;
constexpr size_t FLAG_PAR = (size_t)NR * 8 * NSL * 128;
constexpr size_t WS_PFLAG = WS_FLAG + 2 * FLAG_PAR;
constexpr size_t WS_NFLAG = WS_PFLAG + NR * 8 * 128;
constexpr size_t WS_EFLAG = WS_NFLAG + NR * 8 * 8 * 128;
constexpr size_t WS_CNT   = WS_EFLAG + 64 * 512;
constexpr size_t WS_INV   = WS_CNT + NWG * 128;
constexpr size_t WS_ROT   = WS_INV + 16 * 128;
constexpr size_t WS_TMO   = WS_ROT + 32 * 128;
constexpr size_t WS_DBG   = WS_TMO + 128;
constexpr size_t WS_TR = WS_DBG + NWG * 64 * 8;
constexpr size_t WS_END   = WS_TR + NWG * 1024;
constexpr u64 DEADLINE = 30000000ull;
constexpr u64 SLOW = 20000ull;
constexpr int WSL = KS * BN;
constexpr int LDS_LUT = 0;
constexpr int LDS_CLUT = 128;
constexpr int LDS_MISC = 256;
constexpr int L_BAD = LDS_MISC, L_VAL = LDS_MISC + 4, L_NGO = LDS_MISC + 8, L_PUB = LDS_MISC + 12, L_LEN = LDS_MISC + 32, L_ROT = LDS_MISC + 192;
constexpr int L_ARR = LDS_MISC + 16, L_NL = LDS_MISC + 20, L_RDY = LDS_MISC + 24, L_CVL = LDS_MISC + 28, L_DUMMY = LDS_MISC + 64;
constexpr int SCR = 2048;
constexpr int A3O = 1040, A4O = 1312, VBO = 1536;
constexpr int LDS_SCR = 512;
constexpr int LDS_E = 9 * 1024;
constexpr int LDS_N = LDS_E + 4 * PL;
constexpr int LDS_W = LDS_N + 8 * PL;
constexpr int CVS = 4352;
constexpr int LDS_CV = LDS_W + 4 * WSL;
constexpr int LDS_TR = LDS_CV + 8 * CVS;
constexpr int LDS_TOT = LDS_TR + 1024;
static_assert(LDS_SCR + 4 * SCR <= LDS_E && LDS_TOT <= 160 * 1024, "lds");

struct Args {
  char* ws[NR];
  const unsigned char* x; const unsigned char* w;
  const float* sx; const float* sw; float* out;
  int rank; u32 order;
};

#define STAMP(i) do {} while (0)
#define TDECL
#define TSTART() do {} while (0)
#define TACC(v) do {} while (0)
#define TOUT(i, v) do {} while (0)
#define RMARK() do {} while (0)

#define OKIND 3
#define OST(p, v) __hip_atomic_store((p), (v), __ATOMIC_RELAXED, __HIP_MEMORY_SCOPE_SYSTEM)
#define LNWG NWG
#define LNTHR NTHR
#define LLDS LDS_TOT
#define TR(i) do {} while (0)
#define SCHED() __builtin_amdgcn_sched_barrier(0)
template <int N> __device__ __forceinline__ void waitv() { asm volatile("s_waitcnt vmcnt(%0)" :: "n"(N) : "memory"); }
__device__ __forceinline__ void cfence() { asm volatile("" ::: "memory"); }
__device__ __forceinline__ void bar() {
  cfence(); SCHED();
  __builtin_amdgcn_s_waitcnt(0xC07F);
  __builtin_amdgcn_s_barrier();
  SCHED(); cfence();
}
__device__ __forceinline__ void set_m0(u32 lds_addr) { asm volatile("s_mov_b32 m0, %0\n\ts_nop 0" :: "s"(lds_addr) : "memory"); }
template <int OFF, bool SYS> __device__ __forceinline__ void glds(const void* sbase, u32 voff) {
  if (SYS) asm volatile("global_load_lds_dwordx4 %0, %1 offset:%2 sc0 sc1" :: "v"(voff), "s"(sbase), "n"(OFF) : "memory");
  else asm volatile("global_load_lds_dwordx4 %0, %1 offset:%2" :: "v"(voff), "s"(sbase), "n"(OFF) : "memory");
}
__device__ __forceinline__ void st16(void* p, u32x4 v) { asm volatile("global_store_dwordx4 %0, %1, off sc0 sc1\n\ts_nop 1" :: "v"(p), "v"(v) : "memory"); }
__device__ __forceinline__ void st8(void* p, u32x2 v) { asm volatile("global_store_dwordx2 %0, %1, off sc0 sc1\n\ts_nop 1" :: "v"(p), "v"(v) : "memory"); }
__device__ __forceinline__ void st4(void* p, u32 v) { asm volatile("global_store_dword %0, %1, off sc0 sc1\n\ts_nop 1" :: "v"(p), "v"(v) : "memory"); }
__device__ __forceinline__ u32 ld4(const void* p) {
  u32 v;
  asm volatile("global_load_dword %0, %1, off sc0 sc1\n\ts_waitcnt vmcnt(0)" : "=&v"(v) : "v"(p) : "memory");
  return v;
}

__device__ __forceinline__ bool expired(u64 t0, u32* tmo, u32 call) {
  const u64 dt = __builtin_amdgcn_s_memrealtime() - t0;
  if (dt <= SLOW) return false;
  if (dt > DEADLINE) { st4(tmo, call); return true; }
  __builtin_amdgcn_s_sleep(64);
  return __builtin_amdgcn_readfirstlane(ld4(tmo)) == call;
}

__device__ __forceinline__ u32 wave_scan(u32 x) {
  x += __builtin_amdgcn_update_dpp(0u, x, 0x111, 0xf, 0xf, true);
  x += __builtin_amdgcn_update_dpp(0u, x, 0x112, 0xf, 0xf, true);
  x += __builtin_amdgcn_update_dpp(0u, x, 0x114, 0xf, 0xf, true);
  x += __builtin_amdgcn_update_dpp(0u, x, 0x118, 0xf, 0xf, true);
  x += __builtin_amdgcn_update_dpp(0u, x, 0x142, 0xa, 0xf, false);
  x += __builtin_amdgcn_update_dpp(0u, x, 0x143, 0xc, 0xf, false);
  return x;
}
constexpr u32 B1 = 0x01010101u, M0F = 0x0F0F0F0Fu, MSM = 0x87878787u, MEX = 0x78787878u, M03 = 0x03030303u;
__device__ __forceinline__ u32 rotl4(u32 x) { return __builtin_amdgcn_alignbit(x, x, 28); }
__device__ __forceinline__ u32 rotr4(u32 x) { return __builtin_amdgcn_alignbit(x, x, 4); }
__device__ __forceinline__ u32x2 centry(u32 p) {
  return *LP(const u32x2, __builtin_amdgcn_udot4(p & 0x04040404u, 0x10080402u, (u32)LDS_CLUT, false));
}
__device__ __forceinline__ u32x2 dentry(u32 c, bool with_count = true) {
  const u32 p = __builtin_amdgcn_udot4(__builtin_amdgcn_perm(0u, 0x01000000u, c), 0x40201008u, (u32)LDS_LUT, false);
  if (with_count) return *LP(const u32x2, p);
  return u32x2{*LP(const u32, p), 0u};
}
__device__ __forceinline__ void put4(u32 o, u32 y) {
  const u64 w = (u64)y << ((o & 3u) * 8u);
  LAS u32* p = LP(u32, o & ~3u);
  __hip_atomic_fetch_or(p, (u32)w, __ATOMIC_RELAXED, __HIP_MEMORY_SCOPE_WAVEFRONT);
  __hip_atomic_fetch_or(p + 1, (u32)(w >> 32), __ATOMIC_RELAXED, __HIP_MEMORY_SCOPE_WAVEFRONT);
}
__device__ __forceinline__ u32x2 pair(u32 o) {
  const LAS u32* p = LP(const u32, o & ~3u);
  return u32x2{p[0], p[1]};
}
__device__ __forceinline__ u32 expand(u32x2 w, u32 o, u32 table, u32 sel) {
  return __builtin_amdgcn_perm(__builtin_amdgcn_alignbyte(w[1], w[0], o), table, sel);
}

__global__ void __launch_bounds__(NTHR, 3) ag_gemm(Args a) {
  const int tid = threadIdx.x, lane = tid & 63;
  const int wid = __builtin_amdgcn_readfirstlane(tid >> 6);
  const int b = blockIdx.x;
  const int rank = a.rank;
  char* const me = a.ws[rank];
  if (wid == 0) STAMP(0);
  if (wid == 0) TR(255);
  const int jp = b & 7, idx = b >> 3, ct = idx & 3;
  int j, q, src;
  auto role = [&]() {
    u32 r;
    while (!(r = __builtin_amdgcn_readfirstlane(*LP(volatile u32, L_ROT)))) __builtin_amdgcn_s_sleep(1);
    const int x = (b + r - 1) & 7;
    j = (x & 3) * 2 + ((idx >> 2) & 1); q = (x & 4) | (idx >> 3);
    src = (a.order >> (4 * q)) & (NR - 1);
  };
  const u32 xcc = __builtin_amdgcn_s_getreg((3 << 11) | 20) & 15u;
  u32* const tmo = (u32*)(me + WS_TMO);
  const u32 dummy = L_DUMMY + (lane & 31) * 4;
  const u32 l16 = lane * 16u;
  char* const inv = me + WS_INV + (size_t)xcc * 128;
  u32* const cnt = (u32*)(me + WS_CNT + (size_t)b * 128);
  const int brow = (wid & 3) * 16 + (lane >> 2);
  const unsigned char* const xlane = a.x + (size_t)(jp * BM + brow) * KTOT + (((lane & 3) ^ ((brow >> 2) & 3)) << 5);

  if (wid < 4) {
    const int pe = idx >> 3;
    const bool early = pe == 3 && (idx & 7) < NE;
    const int ps = pe == 3 ? NPUSH + (idx & 7) : pe;
    auto nplane = [&](const u32x4 v0, const u32x4 v1) {
      u32x4 np;
#pragma unroll
      for (int k = 0; k < 4; ++k) np[k] = (v0[k] & MSM) | rotl4(v1[k] & MSM);
      return np;
    };
    const u32x4 pv0 = *(const u32x4*)(xlane + ps * KS), pv1 = *(const u32x4*)(xlane + ps * KS + 16);
    char* nflag = a.ws[lane & (NR - 1)] + WS_NFLAG + (size_t)((rank * 8 + jp) * 8 + (idx & 7)) * 128 + wid * 4;
    const u32 callv = __hip_atomic_load(cnt, __ATOMIC_RELAXED, __HIP_MEMORY_SCOPE_AGENT);
    if (tid < 16) {
      u32 add = 0, cs = 0x0C0C0C0Cu, n = 0;
#pragma unroll
      for (int i = 0; i < 4; ++i)
        if ((tid >> i) & 1) { add |= (1 + n) << (8 * i); cs = (cs & ~(0xFFu << (8 * n))) | (i << (8 * n)); ++n; }
      *LP(u32x2, LDS_LUT + tid * 8) = u32x2{add, n};
      *LP(u32x2, LDS_CLUT + tid * 8) = u32x2{cs, n};
    }
    const int kr = lane >> 3, cp = lane & 7;
    u32 wvo[4];
#pragma unroll
    for (int i = 0; i < 4; ++i) wvo[i] = (kr + i * 8) * NL + ((cp >> 1) ^ ((kr >> 1) & 3)) * 32 + (cp & 1) * 16 - i * 1024;
    const unsigned char* const wsrc = a.w + (size_t)(wid * 32) * NL + ct * BN;
    auto stage_w = [&](int s) {
      set_m0(LDS_W + (s & 3) * WSL + wid * 4096);
      const unsigned char* sb = wsrc + (size_t)s * KS * NL;
      glds<0, false>(sb, wvo[0]); glds<1024, false>(sb, wvo[1]); glds<2048, false>(sb, wvo[2]); glds<3072, false>(sb, wvo[3]);
    };
    char* const pdst = a.ws[(rank + idx) & (NR - 1)];
    bar();
    const u32 call = __builtin_amdgcn_readfirstlane(callv) + 1u;
    const size_t par = call & 1;
    const size_t poff = WS_PUSH + par * PUSH_PAR + (size_t)((rank * 8 + jp) * 8) * PL + wid * 1024 + l16;
    const size_t pfoff = WS_PFLAG + (size_t)(rank * 8 + jp) * 128 + wid * 4;
    const u32x4 np0 = nplane(pv0, pv1);
    asm volatile("" :: "v"(np0));
    const u64 tpush = __builtin_amdgcn_s_memrealtime();
    auto until = [&](int p) { while (__builtin_amdgcn_s_memrealtime() - tpush < (u64)(p * PACE)) __builtin_amdgcn_s_sleep(1); };
    asm volatile("" : "+v"(nflag));
    if (pe < NPUSH) { until(pe); st16(pdst + poff + pe * PL, np0); }
    if (early) {
      st16(me + par * WS_SLOT + (size_t)(jp * NSL + ps) * USTRIDE + wid * 1024 + l16, np0);
      waitv<0>();
      if (lane < NR) st4(nflag, call);
    }
    if (wid == 0) TR(244);
    float sc = a.sx[0] * a.sw[0];
    while (!__builtin_amdgcn_readfirstlane(*LP(volatile u32, L_NGO))) __builtin_amdgcn_s_sleep(4);
    waitv<0>();
    if (lane == 0 && pe < NPUSH) st4(pdst + pfoff + pe * 16, call);
    asm volatile("" : "+v"(sc));
    if (wid == 0) { STAMP(11); TR(245); }
    role();
    const char* const esrc = me + WS_E + (size_t)((q * 8 + j) * NSL) * PL + wid * 1024;
    const char* const psrc = me + WS_PUSH + par * PUSH_PAR + (size_t)((src * 8 + j) * 8) * PL + wid * 1024;
    auto stage_e = [&](int s, bool push) {
      if (push) { set_m0(LDS_N + (s & 7) * PL + wid * 1024); glds<0, true>(psrc + s * PL, l16); }
      set_m0(LDS_E + (s & 3) * PL + wid * 1024); glds<0, true>(esrc + s * PL, l16);
    };
    v16f acc[2] = {};
    const int r31 = lane & 31, h = lane >> 5, wm = wid >> 1, wn = wid & 1;
    const int q8 = (lane & 15) >> 1, p8 = lane & 1, g1 = (lane >> 4) & 1;
    u32 boff[2], aoff[2];
#pragma unroll
    for (int i = 0; i < 2; ++i) {
      boff[i] = LDS_W + (32 * h + q8) * 128 + (((wn * 2 + i) ^ ((q8 >> 1) & 3)) << 5) + g1 * 16 + p8 * 8;
      aoff[i] = (wm * 32 + r31) * 64 + (((i * 2 + h) ^ ((r31 >> 2) & 3)) << 4);
    }
    TDECL;
    struct Frag { u32x4 ep[2]; v8i bf[2][2]; };
    auto ldfrag = [&](Frag& f, int kk, int s) {
      f.ep[kk] = *LP(const u32x4, aoff[kk] + (s & 3) * PL + LDS_E);
#pragma unroll
      for (int n = 0; n < 2; ++n) {
        v2i t[4];
#pragma unroll
        for (int r = 0; r < 4; ++r) t[r] = __builtin_amdgcn_ds_read_tr8_b64_v2i32(LP(v2i, boff[n] + (s & 3) * WSL) + (kk * 64 + r * 8) * 16);
        f.bf[kk][n] = v8i{t[0][0], t[0][1], t[1][0], t[1][1], t[2][0], t[2][1], t[3][0], t[3][1]};
      }
    };
    auto afrag = [&](const u32x4 np, const u32x4 ep) {
      v8i af;
#pragma unroll
      for (int k = 0; k < 4; ++k) {
        af[k] = (np[k] & MSM) | (ep[k] & MEX);
        af[4 + k] = rotr4((np[k] & MEX) | (ep[k] & MSM));
      }
      return af;
    };
#define MFMA(n, af, bf) acc[n] = __builtin_amdgcn_mfma_scale_f32_32x32x64_f8f6f4(af, bf, acc[n], 0, 0, 0, 0, 0, 0)
    auto step = [&](int s, const Frag& cur, Frag& nxt, bool more, bool stg) {
      const u32x4 np0 = *LP(const u32x4, aoff[0] + (s & 7) * PL + LDS_N);
      const u32x4 np1 = *LP(const u32x4, aoff[1] + (s & 7) * PL + LDS_N);
      SCHED();
      if (stg) { stage_e(s + 4, s + 4 < NPUSH); stage_w(s + 4); }
      SCHED();
      const v8i a0 = afrag(np0, cur.ep[0]);
      MFMA(0, a0, cur.bf[0][0]);
      SCHED();
      const v8i a1 = afrag(np1, cur.ep[1]);
      SCHED();
      MFMA(1, a0, cur.bf[0][1]);
      SCHED();
      if (more) ldfrag(nxt, 0, s + 1);
      SCHED();
      MFMA(0, a1, cur.bf[1][0]);
      SCHED();
      if (more) ldfrag(nxt, 1, s + 1);
      SCHED();
      MFMA(1, a1, cur.bf[1][1]);
      SCHED();
      asm volatile("" :: "v"(a0), "v"(cur.bf[0][0]), "v"(cur.bf[0][1]), "v"(cur.bf[1][0]));
    };
    const u32 arr = lane ? dummy : (u32)L_ARR;
    auto spin = [&](int na, int nl, int nr) {
      while (true) {
        u32x4 w;
        asm volatile("ds_read_b128 %0, %1\n\ts_waitcnt lgkmcnt(0)" : "=&v"(w) : "v"((u32)L_ARR) : "memory");
        if (__all((int)w[0] >= na && (int)w[1] >= nl && (int)w[2] >= nr)) break;
      }
    };
    auto meet = [&](int s) {
      SCHED();
      asm volatile("s_waitcnt lgkmcnt(0)\n\tds_add_u32 %0, %1" :: "v"(arr), "v"(1u) : "memory");
      if (wid == 0 && s >= 0) TR(96 + s);
      spin(4 * s + 8, s + 1, s == NSL - 1 ? NSL + 1 : min(s + 5, NSL));
      if (wid == 0 && s >= 0) TR(32 + s);
      __builtin_amdgcn_s_waitcnt(0xC07F);
      SCHED();
    };
    {
      Frag fa, fb;
#pragma unroll 1
      for (int s = 0; s < 4; ++s) { stage_w(s); __builtin_amdgcn_s_sleep(8); }
      spin(0, 0, 4);
#pragma unroll 1
      for (int s = 0; s < 4; ++s) stage_e(s, s < NPUSH);
      waitv<0>(); meet(-1);
      ldfrag(fa, 0, 0); ldfrag(fa, 1, 0);
      meet(0); step(0, fa, fb, true, true);
      TSTART();
#pragma unroll 1
      for (int s = 1; s < NSL - 5; s += 2) {
        if (s > 9) TACC(_ta); else TACC(_tb);
        waitv<10>(); meet(s);
        if (s >= 9) TACC(_tc); else TACC(_tb);
        step(s, fb, fa, true, true);
        if (s >= 9) TACC(_ta); else TACC(_tb);
        waitv<10>(); meet(s + 1);
        if (s >= 9) TACC(_tc); else TACC(_tb);
        step(s + 1, fa, fb, true, true);
      }
      waitv<10>(); meet(NSL - 5); step(NSL - 5, fb, fa, true, true);
      waitv<10>(); meet(NSL - 4); step(NSL - 4, fa, fb, true, false);
      waitv<5>(); meet(NSL - 3); step(NSL - 3, fb, fa, true, false);
      waitv<0>(); meet(NSL - 2); step(NSL - 2, fa, fb, true, false);
      meet(NSL - 1); step(NSL - 1, fb, fa, false, false);
    }
    if (*LP(volatile u32, L_BAD)) sc = __builtin_nanf("");
    if (tid == 0) st4(cnt, call);
    if (wid == 0) { STAMP(5); TOUT(35, _ta / 17); TOUT(37, _tc / 18); }
    float* o = a.out + (size_t)(src * MS + j * BM + wm * 32 + 4 * h) * NL + ct * BN + wn * 64 + r31;
#pragma unroll
    for (int i = 0; i < 16; ++i)
#pragma unroll
      for (int n = 0; n < 2; ++n)
        OST(o + (size_t)((i & 3) + 8 * (i >> 2)) * NL + n * 32, acc[n][i] * sc);
  } else if (wid < 8) {
    const int lw = wid - 4;
    const u32 scr = LDS_SCR + lw * SCR;
    u32 call;
    {
      const u32x4 v0 = *(const u32x4*)(xlane + idx * KS);
      const u32x4 v1 = *(const u32x4*)(xlane + idx * KS + 16);
      const u32 callv = __hip_atomic_load(cnt, __ATOMIC_RELAXED, __HIP_MEMORY_SCOPE_AGENT);
      char* fdst = a.ws[lane & (NR - 1)] + WS_FLAG + (size_t)((rank * 8 + jp) * NSL + idx) * 128;
      *LP(u32x4, scr + l16) = u32x4{0, 0, 0, 0};
      *LP(u32x2, scr + 1024 + lane * 8) = u32x2{0, 0};
      bar();
      call = __builtin_amdgcn_readfirstlane(callv) + 1u;
      asm volatile("" : "+v"(fdst));
      if (lw == 0) { STAMP(9); TR(242); }
      char* const dst = me + (size_t)(call & 1) * WS_SLOT + (size_t)(jp * NSL + idx) * USTRIDE;
      const u32 vbuf = scr + VBO;
      const u32 v[8] = {v0[0], v0[1], v0[2], v0[3], v1[0], v1[1], v1[2], v1[3]};
      u32 z[8], sum = 0;
#pragma unroll
      for (int k = 0; k < 8; ++k) { z[k] = v[k] & MEX; if (k & 1) sum = __builtin_amdgcn_udot4(z[k], B1, sum, false); }
      sum = __builtin_amdgcn_readlane(wave_scan(sum), 63);
      const u32 bb = ((sum + 7946u) >> 13) & 15u;
      const u32 kz = ((7u - bb) & 15u) * 0x08080808u;
      u32x4 np;
#pragma unroll
      for (int k = 0; k < 4; ++k) np[k] = (v[k] & MSM) | rotl4(v[4 + k] & MSM);
      if (idx >= N0) st16(dst + lw * 1024 + l16, np);
      u32x2 e1[8], cp2 = {0, 0};
#pragma unroll
      for (int k = 0; k < 8; ++k) {
        z[k] = (((z[k] + kz) >> 3) & M0F) ^ 0x04040404u;
        const u32 p = __builtin_amdgcn_perm(0xFFFFFFFFu, 0x8201FFFFu, z[k]);
        cp2[k >> 2] |= (p & M03) << (2 * (k & 3));
        e1[k] = centry(p);
      }
      const u32 t1 = (e1[0][1] + e1[1][1] + e1[2][1] + e1[3][1]) | ((e1[4][1] + e1[5][1] + e1[6][1] + e1[7][1]) << 16);
      const u32 sc1 = wave_scan(t1);
      const u32 tot1 = __builtin_amdgcn_readlane(sc1, 63);
      const u32 n2 = (tot1 & 0xFFFFu) + (tot1 >> 16);
      bool raw = n2 > 1024u;
      u32 n2d = 0, n3d = 0, n4d = 0;
      if (!raw) {
        const u32 ex = sc1 - t1 + (scr | ((scr + (tot1 & 0xFFFFu)) << 16));
        u32 o = ex & 0xFFFFu;
#pragma unroll
        for (int k = 0; k < 8; ++k) {
          if (k == 4) o = ex >> 16;
          put4(o, __builtin_amdgcn_perm(0u, z[k], e1[k][0])); o += e1[k][1];
        }
        cfence();
        const u32x4 q2 = *LP(const u32x4, scr + l16);
        u32 l2 = 0; u32x2 e2[4];
#pragma unroll
        for (int k = 0; k < 4; ++k) {
          const u32 p = __builtin_amdgcn_perm(0xFFFFFFFFu, 0xFFFF8201u, q2[k] ^ B1);
          l2 |= (p & M03) << (2 * k); e2[k] = centry(p);
        }
        *LP(u32, vbuf + lane * 4) = l2;
        const u32 t2 = e2[0][1] + e2[1][1] + e2[2][1] + e2[3][1];
        const u32 sc2 = wave_scan(t2);
        const u32 n3 = __builtin_amdgcn_readlane(sc2, 63);
        raw = n3 > 256u;
        if (!raw) {
          n2d = (n2 + 15u) >> 4; n3d = (n3 + 15u) >> 4;
          if (t2) {
            o = scr + A3O + sc2 - t2;
#pragma unroll
            for (int k = 0; k < 4; ++k) { put4(o, __builtin_amdgcn_perm(0u, q2[k] ^ 0x05050505u, e2[k][0])); o += e2[k][1]; }
          }
          cfence();
          const u32 q3 = *LP(const u32, scr + A3O + lane * 4) ^ 0x05050505u;
          const u32 p3 = __builtin_amdgcn_perm(0x800182FFu, 0xFFFFFFFFu, q3);
          u32 l3 = (p3 & M03) << (2 * (lane & 3));
          l3 |= __builtin_amdgcn_update_dpp(0u, l3, 0xB1, 0xf, 0xf, false);
          l3 |= __builtin_amdgcn_update_dpp(0u, l3, 0x4E, 0xf, 0xf, false);
          *LP(u32, vbuf + (n2d + (lane >> 2)) * 4) = l3;
          if (__any(p3 & 0x04040404u)) {
            const u32x2 e3 = centry(p3);
            const u32 sc3 = wave_scan(e3[1]);
            const u32 n4 = __builtin_amdgcn_readlane(sc3, 63);
            raw = n4 > 128u;
            if (!raw) {
              if (e3[1]) put4(scr + A4O + sc3 - e3[1], __builtin_amdgcn_perm(0u, q3, e3[0]));
              cfence();
              const u32x2 q4 = *LP(const u32x2, scr + A4O + (lane & 15) * 8);
              n4d = (n4 + 7u) >> 3;
              if (lane < 16) *LP(u32, vbuf + (n2d + n3d + lane) * 4) = q4[0] | (q4[1] << 4);
            }
          }
        }
      }
      cfence();
      u32 vd = n2d + n3d + n4d;
      if (raw) {
        cp2 = u32x2{(v[0] & MEX) | rotl4(v[4] & MEX), (v[1] & MEX) | rotl4(v[5] & MEX)}; vd = 128;
        *LP(u32, vbuf + lane * 4) = (v[2] & MEX) | rotl4(v[6] & MEX);
        *LP(u32, vbuf + 256 + lane * 4) = (v[3] & MEX) | rotl4(v[7] & MEX);
      }
      cfence();
      u32 voff;
      asm volatile("ds_add_rtn_u32 %0, %1, %2\n\ts_waitcnt lgkmcnt(0)" : "=&v"(voff) : "v"(lane ? dummy : (u32)L_VAL), "v"(vd) : "memory");
      const u32 va = *LP(const u32, vbuf + lane * 4), vb = *LP(const u32, vbuf + 256 + lane * 4);
      st8(dst + U_C + lw * 512 + lane * 8, cp2);
      voff = __builtin_amdgcn_readfirstlane(voff);
      if ((u32)lane < vd) st4(dst + U_V + (voff + lane) * 4, va);
      if ((u32)lane + 64u < vd) st4(dst + U_V + (voff + 64 + lane) * 4, vb);
      if (lane == 0) st4(dst + U_H + lw * 4, voff | (n2d << 10) | (n3d << 17) | (n4d << 22) | (bb << 27) | (raw ? 0x80000000u : 0u));
      if (lw == 0) { STAMP(10); TR(243); }
      waitv<0>();
      u32 nth;
      asm volatile("ds_add_rtn_u32 %0, %1, %2\n\ts_waitcnt lgkmcnt(0)" : "=&v"(nth) : "v"(lane ? dummy : (u32)L_PUB), "v"(1u) : "memory");
      if (__builtin_amdgcn_readfirstlane(nth) == 3) {
        const u32 vtot = *LP(volatile u32, L_VAL);
        if (lane < NR) st4(fdst + (size_t)(call & 1) * FLAG_PAR, (call << 6) | ((16u + vtot * 4u + 63u) >> 6));
        STAMP(1); TR(241);
      }
    }
    role();
    char* const edst = me + WS_E + (size_t)((q * 8 + j) * NSL + ct) * PL + lw * 1024 + l16;
    char* const eflag = me + WS_EFLAG + (size_t)(q * 8 + j) * 512 + ct * 16 + lw * 4;
    int pend = -1;
    TDECL;
#pragma unroll 1
    for (int t = 0; t < 8; ++t) {
      if ((int)__builtin_amdgcn_readfirstlane(*LP(volatile u32, L_CVL)) <= t) {
        if (pend >= 0) { waitv<0>(); if (lane == 0) st4(eflag + pend * 64, call); pend = -1; }
        while ((int)__builtin_amdgcn_readfirstlane(*LP(volatile u32, L_CVL)) <= t) __builtin_amdgcn_s_sleep(2);
      }
      cfence();
      TSTART();
      const u32 S = LDS_CV + t * CVS;
      const u32 info = __builtin_amdgcn_readfirstlane(*LP(const u32, S + 2048 + lw * 4));
      const u32x2 cc = *LP(const u32x2, S + lw * 512 + lane * 8);
      const u32 V = S + 2064 + (info & 0x1FFu) * 4;
      const u32 n2d = (info >> 10) & 0x7Fu, n3d = (info >> 17) & 0x1Fu, bb = (info >> 27) & 15u;
      u32 l2 = *LP(const u32, V + lane * 4);
      u32x4 ep;
      if (info >> 31) {
        ep = u32x4{cc[0], cc[1], l2, *LP(const u32, V + 256 + lane * 4)};
      } else {
        const u32 l3 = *LP(const u32, V + n2d * 4 + (lane >> 2) * 4);
        u32 c1[8], c2[4]; u32x2 e1[8], e2[4];
#pragma unroll
        for (int k = 0; k < 8; ++k) { c1[k] = (cc[k >> 2] >> (2 * (k & 3))) & M03; e1[k] = dentry(c1[k], (k & 3) != 3); }
        SCHED();
        l2 = (u32)lane < n2d ? l2 : 0u;
        u32 w4 = 0, c3 = (l3 >> (2 * (lane & 3))) & M03;
#pragma unroll
        for (int k = 0; k < 4; ++k) { c2[k] = (l2 >> (2 * k)) & M03; e2[k] = dentry(c2[k], k != 3); }
        SCHED();
        const u32 bbv = bb * B1;
        if (info & (0x1Fu << 22)) {
          c3 = (u32)lane < 4 * n3d ? c3 : 0u;
          const u32x2 e3 = dentry(c3);
          const u32 l4 = *LP(const u32, V + (n2d + n3d + (lane & 15)) * 4) ^ 0x44444444u;
          const u32 K4 = ((bb + 9u) & 15u) * B1;
          *LP(u32x2, scr + A4O + (lane & 15) * 8) = u32x2{(((l4 & M0F) + K4) & M0F) << 3, ((((l4 >> 4) & M0F) + K4) & M0F) << 3};
          cfence();
          const u32 o4 = scr + A4O + min(wave_scan(e3[1]) - e3[1], 128u);
          const u32x2 ww = pair(o4);
          w4 = __builtin_amdgcn_alignbyte(ww[1], ww[0], o4);
          c3 += e3[0];
        }
        *LP(u32, scr + A3O + lane * 4) = __builtin_amdgcn_perm(w4, ((bbv + 0x000A0B0Cu) & M0F) << 3, c3);
        const u32 ta = __builtin_popcount(cc[0] & (cc[0] >> 1) & 0x55555555u), tb = __builtin_popcount(cc[1] & (cc[1] >> 1) & 0x55555555u);
        const u32 t2 = __builtin_popcount(l2 & (l2 >> 1) & 0x55555555u);
        const u32 tt = ta | (tb << 11) | (t2 << 22);
        const u32 sc = wave_scan(tt);
        const u32 na = __builtin_amdgcn_readlane(sc, 63) & 0x7FFu;
        const u32 ex = sc - tt;
        cfence();
        u32 o2[4], o1[8]; u32x2 w[8];
        o2[0] = (ex >> 22) + (scr + A3O);
#pragma unroll
        for (int k = 1; k < 4; ++k) o2[k] = o2[k - 1] + e2[k - 1][1];
#pragma unroll
        for (int k = 0; k < 4; ++k) w[k] = pair(o2[k]);
        SCHED();
        o1[0] = (ex & 0x7FFu) + scr; o1[4] = ((ex >> 11) & 0x7FFu) + (scr + na);
#pragma unroll
        for (int k = 1; k < 8; ++k) if (k != 4) o1[k] = o1[k - 1] + e1[k - 1][1];
#pragma unroll
        for (int k = 0; k < 4; ++k) c2[k] += e2[k][0];
        SCHED();
        u32x4 y;
#pragma unroll
        for (int k = 0; k < 4; ++k) y[k] = expand(w[k], o2[k], ((bbv + 0x000D0E02u) & M0F) << 3, c2[k]);
        *LP(u32x4, scr + l16) = y;
        cfence();
#pragma unroll
        for (int k = 0; k < 8; ++k) w[k] = pair(o1[k]);
        SCHED();
#pragma unroll
        for (int k = 0; k < 8; ++k) c1[k] += e1[k][0];
        SCHED();
        const u32 T1 = ((bbv + 0x00000F01u) & M0F) << 3;
#pragma unroll
        for (int k = 0; k < 4; ++k) ep[k] = expand(w[k], o1[k], T1, c1[k]) | rotl4(expand(w[4 + k], o1[4 + k], T1, c1[4 + k]));
      }
      st16(edst + (size_t)t * 4 * PL, ep);
      if (lw == 0) TR(72 + t);
      TACC(_ta);
      if (pend >= 0) { waitv<1>(); if (lane == 0) st4(eflag + pend * 64, call); }
      pend = t;
      TACC(_tb);
    }
    waitv<0>();
    if (lane == 0) st4(eflag + 7 * 64, call);
    if (lw == 0) { STAMP(8); TOUT(32, _ta / 8); TOUT(33, _tb / 8); }
  } else if (wid == 9) {
    if (b < 8) asm volatile("buffer_inv sc0 sc1" ::: "memory");
    bar();
    const u32 call = __builtin_amdgcn_readfirstlane(ld4(cnt)) + 1u;
    if (b < 8 && lane < 4) st4(inv + lane * 4, call);
    while (__builtin_amdgcn_readfirstlane(*LP(volatile u32, L_PUB)) < 4) __builtin_amdgcn_s_sleep(4);
    role();
    const u64 t0 = __builtin_amdgcn_s_memrealtime();
    {
      const char* f = me + WS_FLAG + (size_t)(call & 1) * FLAG_PAR + (size_t)((src * 8 + j) * NSL + (lane & 31)) * 128;
      u32 go = 1, fv;
      while (true) {
        fv = ld4(f);
        const bool ok = (fv >> 6) >= (call & 0x3FFFFFFu);
        if ((lane & 35) == ct && ok) *LP(volatile u32, L_LEN + (lane >> 2) * 4) = max(min(fv & 63u, 33u), 1u) * 64u + 2048u;
        if (__all(ok)) break;
        if (expired(t0, tmo, call)) { go = 2; *LP(volatile u32, L_BAD) = 1; break; }
        __builtin_amdgcn_s_sleep(2);
      }
      *LP(volatile u32, L_NGO) = go;
      TR(246);
    }
    const char* const f = lane < 32 ? me + WS_EFLAG + (size_t)(q * 8 + j) * 512 + l16
                                    : me + WS_PFLAG + (size_t)(src * 8 + j) * 128 + (lane & 7) * 16;
    while (true) {
      u32x4 v;
      asm volatile("global_load_dwordx4 %0, %1, off sc0 sc1\n\ts_waitcnt vmcnt(0)" : "=&v"(v) : "v"(f) : "memory");
      const u64 known = __ballot(min(min(v[0], v[1]), min(v[2], v[3])) >= call);
      const int nready = __builtin_ctzll(~(u64)((u32)known & ((u32)(known >> 32) | (~0u << NPUSH))));
      if (nready == NSL) break;
      *LP(volatile u32, L_RDY) = nready;
      if (expired(t0, tmo, call)) break;
      __builtin_amdgcn_s_sleep(PSLEEP);
    }
    *LP(volatile u32, L_RDY) = NSL;
    if (__builtin_amdgcn_readfirstlane(ld4(tmo)) == call) *LP(volatile u32, L_BAD) = 1;
    *LP(volatile u32, L_RDY) = NSL + 1;
    STAMP(13);
  } else {
    __builtin_amdgcn_s_setprio(3);
    const u32 tstart = (u32)__builtin_amdgcn_s_memrealtime();
    *LP(u32x4, LDS_MISC) = u32x4{0, 0, 0, 0};
    *LP(u32x4, LDS_MISC + 16) = u32x4{0, NPUSH, 0, 0};
    *LP(u32, L_ROT) = 0;
    *LP(u32x4, L_LEN) = u32x4{0, 0, 0, 0}; *LP(u32x4, L_LEN + 16) = u32x4{0, 0, 0, 0};
    bar();
    const u32 call = __builtin_amdgcn_readfirstlane(ld4(cnt)) + 1u;
    {
      if (b == 0 && lane < 32) st4(me + WS_ROT + lane * 128, (call << 4) | (xcc & 7u));
      const u64 t0 = __builtin_amdgcn_s_memrealtime();
      u32 r;
      while (((r = __builtin_amdgcn_readfirstlane(ld4(me + WS_ROT + (b & 31) * 128))) >> 4) < (call & 0xFFFFFFFu)) {
        if (expired(t0, tmo, call)) { *LP(volatile u32, L_BAD) = 1; r = 0; break; }
        __builtin_amdgcn_s_sleep(1);
      }
      *LP(volatile u32, L_ROT) = (r & 7u) + 1u;
      role();
      TR(248);
    }
    {
      const char* f = lane < NE ? me + WS_NFLAG + (size_t)((src * 8 + j) * 8 + lane) * 128 : inv;
      const u64 t0 = __builtin_amdgcn_s_memrealtime();
      while (true) {
        u32x4 v;
        asm volatile("global_load_dwordx4 %0, %1, off sc0 sc1\n\ts_waitcnt vmcnt(0)" : "=&v"(v) : "v"(f) : "memory");
        const u64 ok = __ballot(min(min(v[0], v[1]), min(v[2], v[3])) >= call);
        if (ok == ~0ull) break;
        if ((~ok & ((1u << NE) - 1)) == 0 && __builtin_amdgcn_s_memrealtime() - t0 > 500) {
          asm volatile("buffer_inv sc0 sc1\n\ts_waitcnt vmcnt(0)" ::: "memory");
          break;
        }
        if (expired(t0, tmo, call)) {
          *LP(volatile u32, L_BAD) = 1; *LP(volatile u32, L_CVL) = 8; *LP(volatile u32, L_NL) = NSL;
          return;
        }
        __builtin_amdgcn_s_sleep(2);
      }
      STAMP(2); TR(240);
      waitv<0>();
    }
    u32 lim = 0;
    bool dead = false;
    const char* const base = a.ws[src] + (size_t)(call & 1) * WS_SLOT + (size_t)(j * NSL) * USTRIDE;
    const u32 tau = q ? TAU : TAU * LOCF / 8;
    u32 due = (tstart + (u32)max((int)((u32)__builtin_amdgcn_s_memrealtime() - tstart) - CMAX, TREF)) << 4;
    auto pace = [&](u32 bytes) {
      while ((int)(((u32)__builtin_amdgcn_s_memrealtime() << 4) - due) < 0) {}
      due += bytes * tau >> 10;
    };
    int ct_ = 0, cp_ = 0;
    auto issue_c = [&]() {
      const char* sb = base + (size_t)(4 * ct_ + ct) * USTRIDE + U_C;
      const u32 la = LDS_CV + ct_ * CVS;
      if (cp_ == 0) {
        while (!(lim = __builtin_amdgcn_readfirstlane(*LP(volatile u32, L_LEN + ct_ * 4))))
          if (__builtin_amdgcn_readfirstlane(*LP(volatile u32, L_NGO)) == 2) { dead = true; break; }
        if (ct_ == 0) { const u32 now = ((u32)__builtin_amdgcn_s_memrealtime() - CMAX) << 4; if ((int)(now - due) > 0) due = now; TR(247); }
      }
      if (dead) return;
      pace(cp_ == 2 ? 4u * (lim - 2048u) : 4096u);
      if (cp_ == 2 && lim > 3072u) {
        set_m0(la + 3072); if (l16 + 3072u < lim) glds<0, CSYS>(sb, l16 + 3072u);
        set_m0(la + 4096); if (l16 + 4096u < lim) glds<0, CSYS>(sb, l16 + 4096u);
      }
      set_m0(la + cp_ * 1024);
      if (l16 + cp_ * 1024u < lim) glds<0, CSYS>(sb, l16 + cp_ * 1024u);
      if (++cp_ == 3) { cp_ = 0; ++ct_; }
    };
    TDECL;
    int arrived = 0;
    auto issue_n = [&](int s, int p) {
      const char* sb = base + (size_t)s * USTRIDE;
      pace(1024u);
      if (p == 0) {
        TACC(_ta);
        while (s >= 8 && arrived < 4 * (s - 5)) arrived = __builtin_amdgcn_readfirstlane(*LP(volatile u32, L_ARR));
        TACC(_tb);
        set_m0(LDS_N + (s & 7) * PL); glds<0, false>(sb, l16);
      }
      if (p == 1) glds<1024, false>(sb, l16);
      if (p == 2) glds<2048, false>(sb, l16);
      if (p == 3) glds<3072, false>(sb, l16);
    };
    auto post = [&](u32 addr, u32 v) { *LP(volatile u32, addr) = v; TR((addr == L_NL ? 0 : 64) + v - 1); };
    constexpr int NB = 24 - 3 * PRE;
    static_assert(PRE >= 2 && PRE <= 7 && W2 >= 4 && W2 <= 16 && N0 + NB + 4 <= NSL, "");
    int cdone = 0;
    auto landed_c = [&]() { if (cdone < 16) TR(224 + cdone); ++cdone; if (cdone % 3 == 0) post(L_CVL, cdone / 3); };
#pragma unroll 1
    for (int s = NPUSH; s < N0; ++s) {
#pragma unroll
      for (int p = 0; p < 4; ++p) { waitv<W2 - 1>(); issue_n(s, p); }
    }
    waitv<3>(); issue_c();
    waitv<1>(); post(L_NL, N0); issue_c();
    TSTART();
#pragma unroll 1
    for (int n = 0; n < 3 * PRE - 2; ++n) { waitv<1>(); issue_c(); landed_c(); }
    waitv<1>(); landed_c();
    u32 go;
    do go = __builtin_amdgcn_readfirstlane(*LP(volatile u32, L_NGO)); while (!go);
    if (go == 2) { waitv<0>(); post(L_CVL, 8); post(L_NL, NSL); return; }
#pragma unroll
    for (int p = 0; p < 4; ++p) issue_n(N0, p);
    waitv<4>(); issue_c(); landed_c();
#pragma unroll 1
    for (int s = N0; s < N0 + NB; ++s) {
#pragma unroll
      for (int p = 0; p < 4; ++p) { waitv<4>(); if (p == 3) post(L_NL, s + 1); issue_n(s + 1, p); }
      waitv<4>();
      if (s + 1 < N0 + NB) issue_c();
      landed_c();
    }
#pragma unroll
    for (int n = 4; n < W2; ++n) issue_n(N0 + NB + n / 4, n % 4);
    RMARK();
    constexpr int S0 = NSL - KEND, S1 = NSL - (W3 + 3) / 4;
    static_assert(S0 >= N0 + NB && S0 <= S1 && W3 >= W2 && W3 <= 16, "");
#pragma unroll 1
    for (int s = N0 + NB; s < S0; ++s) {
#pragma unroll
      for (int p = 0; p < 4; ++p) { waitv<W2 - 1>(); if (p == 3) post(L_NL, s + 1); issue_n(s + (p + W2) / 4, (p + W2) % 4); }
    }
#pragma unroll
    for (int n = W2; n < W3; ++n) issue_n(S0 + n / 4, n % 4);
#pragma unroll 1
    for (int s = S0; s < S1; ++s) {
#pragma unroll
      for (int p = 0; p < 4; ++p) { waitv<W3 - 1>(); if (p == 3) post(L_NL, s + 1); issue_n(s + (p + W3) / 4, (p + W3) % 4); }
    }
#pragma unroll
    for (int n = 0; n < 4 * (NSL - S1) - W3; ++n) { waitv<W3 - 1>(); issue_n(S1 + (n + W3) / 4, (n + W3) % 4); }
    if (NSL - S1 >= 4) { waitv<12>(); post(L_NL, NSL - 3); }
    if (NSL - S1 >= 3) { waitv<8>(); post(L_NL, NSL - 2); }
    if (NSL - S1 >= 2) { waitv<4>(); post(L_NL, NSL - 1); }
    waitv<0>(); post(L_NL, NSL);
    STAMP(4); TOUT(39, _tb / (NSL - NPUSH)); TOUT(12, _r1);
  }
}

extern "C" void kernel_launch(int rank, int nranks, ncclComm_t comm,
                              hipStream_t stream,
                              void* const* d_in, const int* in_sizes, int n_in,
                              void* d_out, int out_size,
                              const int* in_dtypes, int out_dtype,
                              void* const* d_out_all,
                              void* const* d_ws_all, size_t ws_bytes) {
  if (nranks != NR || (unsigned)rank >= (unsigned)NR || ws_bytes < WS_END || n_in < 4) return;
  static bool init = false;
  if (!init) {
    HIP_CHECK(hipFuncSetAttribute((const void*)ag_gemm, hipFuncAttributeMaxDynamicSharedMemorySize, LDS_TOT));
    init = true;
  }
  Args a;
  for (int p = 0; p < NR; ++p) a.ws[p] = (char*)d_ws_all[p];
  a.x = (const unsigned char*)d_in[0]; a.w = (const unsigned char*)d_in[1];
  a.sx = (const float*)d_in[2]; a.sw = (const float*)d_in[3];
  a.out = (float*)d_out; a.rank = rank;
  {
    const unsigned cl = CLIQUE;
    unsigned o = 0; int n = 0;
    for (int pass = 0; pass < 2; ++pass)
      for (int i = 0; i < NR; ++i) { const int p = (rank + i) & 7; if ((int)(((cl >> p) ^ (cl >> rank)) & 1) == pass) o |= (unsigned)p << (4 * n++); }
    a.order = o;
  }
  hipLaunchKernelGGL(ag_gemm, dim3(LNWG), dim3(LNTHR), LLDS, stream, a);
  HIP_CHECK(hipGetLastError());
}
```

```cpp
#include <hip/hip_runtime.h>
#include <rccl/rccl.h>
#include <cstdio>
#include <cstdint>

#pragma clang diagnostic ignored "-Wint-to-pointer-cast"

#define HIP_CHECK(expr) do { hipError_t _e = (expr); if (_e != hipSuccess) { \
    fprintf(stderr, "HIP error %s at %s:%d\n", hipGetErrorString(_e), __FILE__, __LINE__); \
    return; } } while (0)

typedef unsigned u32;
typedef unsigned u32x2 __attribute__((ext_vector_type(2)));
typedef unsigned u32x4 __attribute__((ext_vector_type(4)));
typedef int v2i __attribute__((ext_vector_type(2)));
typedef int v8i __attribute__((ext_vector_type(8)));
typedef float v16f __attribute__((ext_vector_type(16)));
typedef unsigned long long u64;
#define LAS __attribute__((address_space(3)))
#define LP(T, a) ((LAS T*)(u32)(a))

#define TAU 219
#define CREDIT 4u
#define WB 5
#define WA 2
#define W3 W2
#define KEND 8
#define PRE 5
#define W2 6
#define PACE 60
#define NE 3
#define TREF 190
#define CMAX 90
#define PSLEEP 64
#define CSYS true
#define LOCF 2
#define CLIQUE 0x4D
#define KT 3
#define ELD " sc0 sc1"
#define EST " sc0 sc1"
#define EPL " sc0 sc1"
#define S_ROT 1
#define D_ROT 0
#define S_EF 2
#define D_EF 0
#define S_UF 2
#define D_UF 0
#define NPUSH 3

constexpr int NR = 8;
constexpr int KTOT = 4096, NL = 512, MS = 512;
constexpr int BM = 64, BN = 128, KS = 128, NSL = KTOT / KS;
constexpr int NWG = 256, NTHR = 640;
constexpr int PL = 4096;
constexpr int U_C = 4096, U_H = 6144, U_V = 6160, USTRIDE = 9216;
constexpr int N0 = NPUSH + NE;
static_assert(NPUSH >= 1 && NPUSH <= 3 && NE >= 1 && N0 <= 8, "");
constexpr size_t WS_SLOT  = (size_t)NWG * USTRIDE;
constexpr size_t WS_PUSH  = 2 * WS_SLOT;
constexpr size_t PUSH_PAR = (size_t)NR * 8 * 8 * PL;
constexpr size_t WS_E     = WS_PUSH + 2 * PUSH_PAR;
constexpr size_t WS_FLAG  = WS_E + (size_t)64 * NSL * PL;
constexpr size_t FLAG_PAR = (size_t)NR * 8 * NSL * 128;
constexpr size_t WS_PFLAG = WS_FLAG + 2 * FLAG_PAR;
constexpr size_t WS_NFLAG = WS_PFLAG + NR * 8 * 128;
constexpr size_t WS_EFLAG = WS_NFLAG + NR * 8 * 8 * 128;
constexpr size_t WS_CNT   = WS_EFLAG + 64 * 512;
constexpr size_t WS_INV   = WS_CNT + NWG * 128;
constexpr size_t WS_ROT   = WS_INV + 16 * 128;
constexpr size_t WS_TMO   = WS_ROT + 32 * 128;
constexpr size_t WS_DBG   = WS_TMO + 128;
constexpr size_t WS_TR = WS_DBG + NWG * 64 * 8;
constexpr size_t WS_END   = WS_TR + NWG * 1024;
constexpr u64 DEADLINE = 30000000ull;
constexpr u64 SLOW = 20000ull;
constexpr int WSL = KS * BN;
constexpr int LDS_LUT = 0;
constexpr int LDS_CLUT = 128;
constexpr int LDS_MISC = 256;
constexpr int L_BAD = LDS_MISC, L_VAL = LDS_MISC + 4, L_NGO = LDS_MISC + 8, L_PUB = LDS_MISC + 12, L_LEN = LDS_MISC + 32, L_ROT = LDS_MISC + 192;
constexpr int L_ARR = LDS_MISC + 16, L_NL = LDS_MISC + 20, L_RDY = LDS_MISC + 24, L_CVL = LDS_MISC + 28, L_DUMMY = LDS_MISC + 64;
constexpr int SCR = 2048;
constexpr int A3O = 1040, A4O = 1312, VBO = 1536;
constexpr int LDS_SCR = 512;
constexpr int LDS_E = 9 * 1024;
constexpr int LDS_N = LDS_E + 4 * PL;
constexpr int LDS_W = LDS_N + 8 * PL;
constexpr int CVS = 4352;
constexpr int LDS_CV = LDS_W + 4 * WSL;
constexpr int LDS_TR = LDS_CV + 8 * CVS;
constexpr int LDS_TOT = LDS_TR + 1024;
static_assert(LDS_SCR + 4 * SCR <= LDS_E && LDS_TOT <= 160 * 1024, "lds");

struct Args {
  const unsigned char* x;
  char* ws[NR];
  const unsigned char* w;
  const float* sx; const float* sw; float* out;
  int rank; u32 order;
};

#define STAMP(i) do {} while (0)
#define TDECL
#define TSTART() do {} while (0)
#define TACC(v) do {} while (0)
#define TOUT(i, v) do {} while (0)
#define RMARK() do {} while (0)

#define OKIND 3
#define OST(p, v) __hip_atomic_store((p), (v), __ATOMIC_RELAXED, __HIP_MEMORY_SCOPE_SYSTEM)
#define LNWG NWG
#define LNTHR NTHR
#define LLDS LDS_TOT
#define TR(i) do {} while (0)
#define SCHED() __builtin_amdgcn_sched_barrier(0)
template <int N> __device__ __forceinline__ void waitv() { asm volatile("s_waitcnt vmcnt(%0)" :: "n"(N) : "memory"); }
__device__ __forceinline__ void waitn(int n) {
  switch (n) {
    case 0: waitv<0>(); break; case 1: waitv<1>(); break; case 2: waitv<2>(); break; case 3: waitv<3>(); break;
    case 4: waitv<4>(); break; case 5: waitv<5>(); break; case 6: waitv<6>(); break; default: waitv<7>();
  }
}
__device__ __forceinline__ void cfence() { asm volatile("" ::: "memory"); }
__device__ __forceinline__ void bar() {
  cfence(); SCHED();
  __builtin_amdgcn_s_waitcnt(0xC07F);
  __builtin_amdgcn_s_barrier();
  SCHED(); cfence();
}
__device__ __forceinline__ void set_m0(u32 lds_addr) { asm volatile("s_mov_b32 m0, %0\n\ts_nop 0" :: "s"(lds_addr) : "memory"); }
template <int OFF, bool SYS> __device__ __forceinline__ void glds(const void* sbase, u32 voff) {
  if (SYS) asm volatile("global_load_lds_dwordx4 %0, %1 offset:%2 sc0 sc1" :: "v"(voff), "s"(sbase), "n"(OFF) : "memory");
  else asm volatile("global_load_lds_dwordx4 %0, %1 offset:%2" :: "v"(voff), "s"(sbase), "n"(OFF) : "memory");
}
__device__ __forceinline__ void gldse(const void* sbase, u32 voff) {
  asm volatile("global_load_lds_dwordx4 %0, %1" ELD :: "v"(voff), "s"(sbase) : "memory");
}
__device__ __forceinline__ void st16e(void* p, u32x4 v) { asm volatile("global_store_dwordx4 %0, %1, off" EST "\n\ts_nop 1" :: "v"(p), "v"(v) : "memory"); }
__device__ __forceinline__ void st4e(void* p, u32 v) { asm volatile("global_store_dword %0, %1, off" EST "\n\ts_nop 1" :: "v"(p), "v"(v) : "memory"); }
__device__ __forceinline__ void st16(void* p, u32x4 v) { asm volatile("global_store_dwordx4 %0, %1, off sc0 sc1\n\ts_nop 1" :: "v"(p), "v"(v) : "memory"); }
__device__ __forceinline__ void st8(void* p, u32x2 v) { asm volatile("global_store_dwordx2 %0, %1, off sc0 sc1\n\ts_nop 1" :: "v"(p), "v"(v) : "memory"); }
__device__ __forceinline__ void st4(void* p, u32 v) { asm volatile("global_store_dword %0, %1, off sc0 sc1\n\ts_nop 1" :: "v"(p), "v"(v) : "memory"); }
__device__ __forceinline__ u32 ld4(const void* p) {
  u32 v;
  asm volatile("global_load_dword %0, %1, off sc0 sc1\n\ts_waitcnt vmcnt(0)" : "=&v"(v) : "v"(p) : "memory");
  return v;
}

__device__ __forceinline__ bool expired(u64 t0, u32* tmo, u32 call) {
  const u64 dt = __builtin_amdgcn_s_memrealtime() - t0;
  if (dt <= SLOW) return false;
  if (dt > DEADLINE) { st4(tmo, call); return true; }
  __builtin_amdgcn_s_sleep(64);
  return __builtin_amdgcn_readfirstlane(ld4(tmo)) == call;
}

__device__ __forceinline__ u32 wave_scan(u32 x) {
  x += __builtin_amdgcn_update_dpp(0u, x, 0x111, 0xf, 0xf, true);
  x += __builtin_amdgcn_update_dpp(0u, x, 0x112, 0xf, 0xf, true);
  x += __builtin_amdgcn_update_dpp(0u, x, 0x114, 0xf, 0xf, true);
  x += __builtin_amdgcn_update_dpp(0u, x, 0x118, 0xf, 0xf, true);
  x += __builtin_amdgcn_update_dpp(0u, x, 0x142, 0xa, 0xf, false);
  x += __builtin_amdgcn_update_dpp(0u, x, 0x143, 0xc, 0xf, false);
  return x;
}
constexpr u32 B1 = 0x01010101u, M0F = 0x0F0F0F0Fu, MSM = 0x87878787u, MEX = 0x78787878u, M03 = 0x03030303u;
__device__ __forceinline__ u32 rotl4(u32 x) { return __builtin_amdgcn_alignbit(x, x, 28); }
__device__ __forceinline__ u32 rotr4(u32 x) { return __builtin_amdgcn_alignbit(x, x, 4); }
__device__ __forceinline__ u32x2 centry(u32 p) {
  return *LP(const u32x2, __builtin_amdgcn_udot4(p & 0x04040404u, 0x10080402u, (u32)LDS_CLUT, false));
}
__device__ __forceinline__ u32x2 dentry(u32 c, bool with_count = true) {
  const u32 p = __builtin_amdgcn_udot4(__builtin_amdgcn_perm(0u, 0x01000000u, c), 0x40201008u, (u32)LDS_LUT, false);
  if (with_count) return *LP(const u32x2, p);
  return u32x2{*LP(const u32, p), 0u};
}
__device__ __forceinline__ void put4(u32 o, u32 y) {
  const u64 w = (u64)y << ((o & 3u) * 8u);
  LAS u32* p = LP(u32, o & ~3u);
  __hip_atomic_fetch_or(p, (u32)w, __ATOMIC_RELAXED, __HIP_MEMORY_SCOPE_WAVEFRONT);
  __hip_atomic_fetch_or(p + 1, (u32)(w >> 32), __ATOMIC_RELAXED, __HIP_MEMORY_SCOPE_WAVEFRONT);
}
__device__ __forceinline__ u32x2 pair(u32 o) {
  const LAS u32* p = LP(const u32, o & ~3u);
  return u32x2{p[0], p[1]};
}
__device__ __forceinline__ u32 expand(u32x2 w, u32 o, u32 table, u32 sel) {
  return __builtin_amdgcn_perm(__builtin_amdgcn_alignbyte(w[1], w[0], o), table, sel);
}

__global__ void __launch_bounds__(NTHR, 3) ag_gemm(Args a) {
  const int tid = threadIdx.x, lane = tid & 63;
  const int wid = __builtin_amdgcn_readfirstlane(tid >> 6);
  const int b = blockIdx.x;
  char* const me = a.ws[0];
  const int jp = b & 7, idx = b >> 3, ct = idx & 3, pe = idx >> 3;
  const int ps = pe == 3 ? NPUSH + (idx & 7) : pe;
  const int brow = (wid & 3) * 16 + (lane >> 2);
  const unsigned char* const xlane = a.x + (size_t)(jp * BM) * KTOT + (wid < 4 ? ps : idx) * KS + (u32)(brow * KTOT + (((lane & 3) ^ ((brow >> 2) & 3)) << 5));
  const u32x4 v0 = *(const u32x4*)xlane, v1 = *(const u32x4*)(xlane + 16);
  u32* const cnt = (u32*)(me + WS_CNT + (size_t)b * 128);
  const u32 callv = __hip_atomic_load(cnt, __ATOMIC_RELAXED, __HIP_MEMORY_SCOPE_AGENT);
  cfence(); SCHED();
  const int rank = a.rank;
  if (wid == 0) STAMP(0);
  if (wid == 0) TR(255);
  int j, q, src;
  auto role = [&]() {
    u32 r;
    while (!(r = __builtin_amdgcn_readfirstlane(*LP(volatile u32, L_ROT)))) __builtin_amdgcn_s_sleep(1);
    const int x = (b + r - 1) & 7;
    j = (x & 3) * 2 + ((idx >> 2) & 1); q = (x & 4) | (idx >> 3);
    src = (a.order >> (4 * q)) & (NR - 1);
  };
  const u32 xcc = __builtin_amdgcn_s_getreg((3 << 11) | 20) & 15u;
  u32* const tmo = (u32*)(me + WS_TMO);
  const u32 dummy = L_DUMMY + (lane & 31) * 4;
  const u32 l16 = lane * 16u;
  char* const inv = me + WS_INV + (size_t)xcc * 128;

  if (wid < 4) {
    const bool early = pe == 3 && (idx & 7) < NE;
    auto nplane = [&](const u32x4 v0, const u32x4 v1) {
      u32x4 np;
#pragma unroll
      for (int k = 0; k < 4; ++k) np[k] = (v0[k] & MSM) | rotl4(v1[k] & MSM);
      return np;
    };
    char* nflag = a.ws[lane & (NR - 1)] + WS_NFLAG + (size_t)((rank * 8 + jp) * 8 + (idx & 7)) * 128 + wid * 4;
    if (tid < 16) {
      u32 add = 0, cs = 0x0C0C0C0Cu, n = 0;
#pragma unroll
      for (int i = 0; i < 4; ++i)
        if ((tid >> i) & 1) { add |= (1 + n) << (8 * i); cs = (cs & ~(0xFFu << (8 * n))) | (i << (8 * n)); ++n; }
      *LP(u32x2, LDS_LUT + tid * 8) = u32x2{add, n};
      *LP(u32x2, LDS_CLUT + tid * 8) = u32x2{cs, n};
    }
    const int kr = lane >> 3, cp = lane & 7;
    u32 wvo[4];
#pragma unroll
    for (int i = 0; i < 4; ++i) wvo[i] = (kr + i * 8) * NL + ((cp >> 1) ^ ((kr >> 1) & 3)) * 32 + (cp & 1) * 16 - i * 1024;
    const unsigned char* const wsrc = a.w + (size_t)(wid * 32) * NL + ct * BN;
    auto stage_w = [&](int s) {
      set_m0(LDS_W + (s & 3) * WSL + wid * 4096);
      const unsigned char* sb = wsrc + (size_t)s * KS * NL;
      { glds<0, false>(sb, wvo[0]); glds<1024, false>(sb, wvo[1]); glds<2048, false>(sb, wvo[2]); glds<3072, false>(sb, wvo[3]); }
    };
    char* const pdst = a.ws[idx & (NR - 1)];
    bar();
    const u32 call = __builtin_amdgcn_readfirstlane(callv) + 1u;
    const size_t par = call & 1;
    const size_t poff = WS_PUSH + par * PUSH_PAR + (size_t)((rank * 8 + jp) * 8) * PL + wid * 1024 + l16;
    const size_t pfoff = WS_PFLAG + (size_t)(rank * 8 + jp) * 128 + wid * 4;
    const u32x4 np0 = nplane(v0, v1);
    asm volatile("" :: "v"(np0));
    const u64 tpush = __builtin_amdgcn_s_memrealtime();
    auto until = [&](int p) { while (__builtin_amdgcn_s_memrealtime() - tpush < (u64)(p * PACE)) __builtin_amdgcn_s_sleep(1); };
    asm volatile("" : "+v"(nflag));
    if (pe < NPUSH) { until(pe); st16(pdst + poff + pe * PL, np0); }
    if (early) {
      st16(me + par * WS_SLOT + (size_t)(jp * NSL + ps) * USTRIDE + wid * 1024 + l16, np0);
      waitv<0>();
      if (lane < NR) st4(nflag, call);
    }
    if (wid == 0) TR(244);
    float sc = a.sx[0] * a.sw[0];
    while (!__builtin_amdgcn_readfirstlane(*LP(volatile u32, L_NGO))) __builtin_amdgcn_s_sleep(4);
    waitv<0>();
    if (lane == 0 && pe < NPUSH) st4(pdst + pfoff + pe * 16, call);
    asm volatile("" : "+v"(sc));
    if (wid == 0) { STAMP(11); TR(245); }
    role();
    const char* const esrc = me + WS_E + (size_t)((q * 8 + j) * NSL) * PL + wid * 1024;
    const char* const psrc = me + WS_PUSH + par * PUSH_PAR + (size_t)((src * 8 + j) * 8) * PL + wid * 1024;
    auto stage_e = [&](int s, bool push) {
      if (push) { set_m0(LDS_N + (s & 7) * PL + wid * 1024); glds<0, true>(psrc + s * PL, l16); }
      set_m0(LDS_E + (s & 3) * PL + wid * 1024); gldse(esrc + s * PL, l16);
    };
    v16f acc[2] = {};
    const int r31 = lane & 31, h = lane >> 5, wm = wid >> 1, wn = wid & 1;
    const int q8 = (lane & 15) >> 1, p8 = lane & 1, g1 = (lane >> 4) & 1;
    u32 boff[2], aoff[2];
#pragma unroll
    for (int i = 0; i < 2; ++i) {
      boff[i] = LDS_W + (32 * h + q8) * 128 + (((wn * 2 + i) ^ ((q8 >> 1) & 3)) << 5) + g1 * 16 + p8 * 8;
      aoff[i] = (wm * 32 + r31) * 64 + (((i * 2 + h) ^ ((r31 >> 2) & 3)) << 4);
    }
    TDECL;
    struct Frag { u32x4 ep[2]; v8i bf[2][2]; };
    auto ldfrag = [&](Frag& f, int kk, int s) {
      f.ep[kk] = *LP(const u32x4, aoff[kk] + (s & 3) * PL + LDS_E);
#pragma unroll
      for (int n = 0; n < 2; ++n) {
        v2i t[4];
#pragma unroll
        for (int r = 0; r < 4; ++r) t[r] = __builtin_amdgcn_ds_read_tr8_b64_v2i32(LP(v2i, boff[n] + (s & 3) * WSL) + (kk * 64 + r * 8) * 16);
        f.bf[kk][n] = v8i{t[0][0], t[0][1], t[1][0], t[1][1], t[2][0], t[2][1], t[3][0], t[3][1]};
      }
    };
    auto afrag = [&](const u32x4 np, const u32x4 ep) {
      v8i af;
#pragma unroll
      for (int k = 0; k < 4; ++k) {
        af[k] = (np[k] & MSM) | (ep[k] & MEX);
        af[4 + k] = rotr4((np[k] & MEX) | (ep[k] & MSM));
      }
      return af;
    };
#define MFMA(n, af, bf) acc[n] = __builtin_amdgcn_mfma_scale_f32_32x32x64_f8f6f4(af, bf, acc[n], 0, 0, 0, 0, 0, 0)
    auto step = [&](int s, const Frag& cur, Frag& nxt, bool more, bool stg) {
      const u32x4 np0 = *LP(const u32x4, aoff[0] + (s & 7) * PL + LDS_N);
      const u32x4 np1 = *LP(const u32x4, aoff[1] + (s & 7) * PL + LDS_N);
      SCHED();
      if (stg) { stage_e(s + 4, s + 4 < NPUSH); stage_w(s + 4); }
      SCHED();
      const v8i a0 = afrag(np0, cur.ep[0]);
      MFMA(0, a0, cur.bf[0][0]);
      SCHED();
      const v8i a1 = afrag(np1, cur.ep[1]);
      SCHED();
      if (!more) {
        MFMA(0, a1, cur.bf[1][0]); MFMA(1, a0, cur.bf[0][1]); MFMA(1, a1, cur.bf[1][1]);
        return;
      }
      MFMA(1, a0, cur.bf[0][1]);
      SCHED();
      if (more) ldfrag(nxt, 0, s + 1);
      SCHED();
      MFMA(0, a1, cur.bf[1][0]);
      SCHED();
      if (more) ldfrag(nxt, 1, s + 1);
      SCHED();
      MFMA(1, a1, cur.bf[1][1]);
      SCHED();
      asm volatile("" :: "v"(a0), "v"(cur.bf[0][0]), "v"(cur.bf[0][1]), "v"(cur.bf[1][0]));
    };
    const u32 arr = lane ? dummy : (u32)L_ARR;
    auto count = [&](u32 addr) { asm volatile("ds_add_u32 %0, %1" :: "v"(addr), "v"(1u) : "memory"); };
    auto spin = [&](int na, int nl, int nr, int nc = 0) {
      while (true) {
        u32x4 w;
        asm volatile("ds_read_b128 %0, %1\n\ts_waitcnt lgkmcnt(0)" : "=&v"(w) : "v"((u32)L_ARR) : "memory");
        if (__all((int)w[0] >= na && (int)w[1] >= nl && (int)w[2] >= nr && (int)w[3] >= nc)) break;
      }
    };
    auto meet = [&](int s) {
      SCHED();
      asm volatile("s_waitcnt lgkmcnt(0)\n\tds_add_u32 %0, %1" :: "v"(arr), "v"(1u) : "memory");
      if (wid == 0 && s >= 0) TR(96 + s);
      spin(4 * s + 8, s + 1, min(s + 5, NSL));
      if (wid == 0 && s >= 0) TR(32 + s);
      __builtin_amdgcn_s_waitcnt(0xC07F);
      SCHED();
    };
    {
      Frag fa, fb;
#pragma unroll 1
      for (int s = 0; s < 4; ++s) { stage_w(s); __builtin_amdgcn_s_sleep(8); }
      spin(0, 0, 4);
#pragma unroll 1
      for (int s = 0; s < 4; ++s) stage_e(s, s < NPUSH);
      waitv<0>(); meet(-1);
      ldfrag(fa, 0, 0); ldfrag(fa, 1, 0);
      meet(0); step(0, fa, fb, true, true);
      TSTART();
#pragma unroll 1
      for (int s = 1; s < NSL - 5; s += 2) {
        if (s > 9) TACC(_ta); else TACC(_tb);
        waitv<10>(); meet(s);
        if (s >= 9) TACC(_tc); else TACC(_tb);
        step(s, fb, fa, true, true);
        if (s >= 9) TACC(_ta); else TACC(_tb);
        waitv<10>(); meet(s + 1);
        if (s >= 9) TACC(_tc); else TACC(_tb);
        step(s + 1, fa, fb, true, true);
      }
      waitv<10>(); meet(NSL - 5); step(NSL - 5, fb, fa, true, true);
      static_assert(KT == 3, "");
      waitv<10>(); meet(NSL - 4); step(NSL - 4, fa, fb, true, false);
      const int hb = wm * KT;
      const u32 arr2 = lane ? dummy : (u32)L_CVL;
      waitv<5>(); count(arr);
      if (wm) { waitv<0>(); count(arr2); }
      SCHED(); spin(4 * (NSL - 3) + 8, NSL - 2 + hb, 0, 0); SCHED(); step(NSL - 3, fb, fa, true, false);
      if (!wm) { waitv<0>(); count(arr2); }
      SCHED(); spin(0, NSL - 1 + hb, 0, 12); SCHED(); step(NSL - 2, fa, fb, true, false);
      SCHED(); spin(0, NSL + hb, NSL + 1, 0); SCHED(); step(NSL - 1, fb, fa, false, false);
    }
    if (*LP(volatile u32, L_BAD)) sc = __builtin_nanf("");
    if (tid == 0) st4(cnt, call);
    if (wid == 0) { STAMP(5); TOUT(35, _ta / 17); TOUT(37, _tc / 18); }
    float* o = a.out + (size_t)(src * MS + j * BM + wm * 32 + 4 * h) * NL + ct * BN + wn * 64 + r31;
#pragma unroll
    for (int n = 0; n < 2; ++n)
#pragma unroll
      for (int i = 0; i < 16; ++i)
        OST(o + (size_t)((i & 3) + 8 * (i >> 2)) * NL + n * 32, acc[n][i] * sc);
  } else if (wid < 8) {
    const int lw = wid - 4;
    const u32 scr = LDS_SCR + lw * SCR;
    u32 call;
    {
      char* fdst = a.ws[lane & (NR - 1)] + WS_FLAG + (size_t)((rank * 8 + jp) * NSL + idx) * 128;
      *LP(u32x4, scr + l16) = u32x4{0, 0, 0, 0};
      *LP(u32x2, scr + 1024 + lane * 8) = u32x2{0, 0};
      bar();
      call = __builtin_amdgcn_readfirstlane(callv) + 1u;
      asm volatile("" : "+v"(fdst));
      if (lw == 0) { STAMP(9); TR(242); }
      char* const dst = me + (size_t)(call & 1) * WS_SLOT + (size_t)(jp * NSL + idx) * USTRIDE;
      const u32 vbuf = scr + VBO;
      const u32 v[8] = {v0[0], v0[1], v0[2], v0[3], v1[0], v1[1], v1[2], v1[3]};
      u32 z[8], sum = 0;
#pragma unroll
      for (int k = 0; k < 8; ++k) { z[k] = v[k] & MEX; if (k & 1) sum = __builtin_amdgcn_udot4(z[k], B1, sum, false); }
      sum = __builtin_amdgcn_readlane(wave_scan(sum), 63);
      const u32 bb = ((sum + 7946u) >> 13) & 15u;
      const u32 kz = ((7u - bb) & 15u) * 0x08080808u;
      u32x4 np;
#pragma unroll
      for (int k = 0; k < 4; ++k) np[k] = (v[k] & MSM) | rotl4(v[4 + k] & MSM);
      if (idx >= N0) st16(dst + lw * 1024 + l16, np);
      u32x2 e1[8], cp2 = {0, 0};
#pragma unroll
      for (int k = 0; k < 8; ++k) {
        z[k] = (((z[k] + kz) >> 3) & M0F) ^ 0x04040404u;
        const u32 p = __builtin_amdgcn_perm(0xFFFFFFFFu, 0x8201FFFFu, z[k]);
        cp2[k >> 2] |= (p & M03) << (2 * (k & 3));
        e1[k] = centry(p);
      }
      const u32 t1 = (e1[0][1] + e1[1][1] + e1[2][1] + e1[3][1]) | ((e1[4][1] + e1[5][1] + e1[6][1] + e1[7][1]) << 16);
      const u32 sc1 = wave_scan(t1);
      const u32 tot1 = __builtin_amdgcn_readlane(sc1, 63);
      const u32 n2 = (tot1 & 0xFFFFu) + (tot1 >> 16);
      bool raw = n2 > 1024u;
      u32 n2d = 0, n3d = 0, n4d = 0;
      if (!raw) {
        const u32 ex = sc1 - t1 + (scr | ((scr + (tot1 & 0xFFFFu)) << 16));
        u32 o = ex & 0xFFFFu;
#pragma unroll
        for (int k = 0; k < 8; ++k) {
          if (k == 4) o = ex >> 16;
          put4(o, __builtin_amdgcn_perm(0u, z[k], e1[k][0])); o += e1[k][1];
        }
        cfence();
        const u32x4 q2 = *LP(const u32x4, scr + l16);
        u32 l2 = 0; u32x2 e2[4];
#pragma unroll
        for (int k = 0; k < 4; ++k) {
          const u32 p = __builtin_amdgcn_perm(0xFFFFFFFFu, 0xFFFF8201u, q2[k] ^ B1);
          l2 |= (p & M03) << (2 * k); e2[k] = centry(p);
        }
        *LP(u32, vbuf + lane * 4) = l2;
        const u32 t2 = e2[0][1] + e2[1][1] + e2[2][1] + e2[3][1];
        const u32 sc2 = wave_scan(t2);
        const u32 n3 = __builtin_amdgcn_readlane(sc2, 63);
        raw = n3 > 256u;
        if (!raw) {
          n2d = (n2 + 15u) >> 4; n3d = (n3 + 15u) >> 4;
          if (t2) {
            o = scr + A3O + sc2 - t2;
#pragma unroll
            for (int k = 0; k < 4; ++k) { put4(o, __builtin_amdgcn_perm(0u, q2[k] ^ 0x05050505u, e2[k][0])); o += e2[k][1]; }
          }
          cfence();
          const u32 q3 = *LP(const u32, scr + A3O + lane * 4) ^ 0x05050505u;
          const u32 p3 = __builtin_amdgcn_perm(0x800182FFu, 0xFFFFFFFFu, q3);
          u32 l3 = (p3 & M03) << (2 * (lane & 3));
          l3 |= __builtin_amdgcn_update_dpp(0u, l3, 0xB1, 0xf, 0xf, false);
          l3 |= __builtin_amdgcn_update_dpp(0u, l3, 0x4E, 0xf, 0xf, false);
          *LP(u32, vbuf + (n2d + (lane >> 2)) * 4) = l3;
          if (__any(p3 & 0x04040404u)) {
            const u32x2 e3 = centry(p3);
            const u32 sc3 = wave_scan(e3[1]);
            const u32 n4 = __builtin_amdgcn_readlane(sc3, 63);
            raw = n4 > 128u;
            if (!raw) {
              if (e3[1]) put4(scr + A4O + sc3 - e3[1], __builtin_amdgcn_perm(0u, q3, e3[0]));
              cfence();
              const u32x2 q4 = *LP(const u32x2, scr + A4O + (lane & 15) * 8);
              n4d = (n4 + 7u) >> 3;
              if (lane < 16) *LP(u32, vbuf + (n2d + n3d + lane) * 4) = q4[0] | (q4[1] << 4);
            }
          }
        }
      }
      cfence();
      u32 vd = n2d + n3d + n4d;
      if (raw) {
        cp2 = u32x2{(v[0] & MEX) | rotl4(v[4] & MEX), (v[1] & MEX) | rotl4(v[5] & MEX)}; vd = 128;
        *LP(u32, vbuf + lane * 4) = (v[2] & MEX) | rotl4(v[6] & MEX);
        *LP(u32, vbuf + 256 + lane * 4) = (v[3] & MEX) | rotl4(v[7] & MEX);
      }
      cfence();
      u32 voff;
      asm volatile("ds_add_rtn_u32 %0, %1, %2\n\ts_waitcnt lgkmcnt(0)" : "=&v"(voff) : "v"(lane ? dummy : (u32)L_VAL), "v"(vd) : "memory");
      const u32 va = *LP(const u32, vbuf + lane * 4), vb = *LP(const u32, vbuf + 256 + lane * 4);
      st8(dst + U_C + lw * 512 + lane * 8, cp2);
      voff = __builtin_amdgcn_readfirstlane(voff);
      if ((u32)lane < vd) st4(dst + U_V + (voff + lane) * 4, va);
      if ((u32)lane + 64u < vd) st4(dst + U_V + (voff + 64 + lane) * 4, vb);
      if (lane == 0) st4(dst + U_H + lw * 4, voff | (n2d << 10) | (n3d << 17) | (n4d << 22) | (bb << 27) | (raw ? 0x80000000u : 0u));
      if (lw == 0) { STAMP(10); TR(243); }
      waitv<0>();
      u32 nth;
      asm volatile("ds_add_rtn_u32 %0, %1, %2\n\ts_waitcnt lgkmcnt(0)" : "=&v"(nth) : "v"(lane ? dummy : (u32)L_PUB), "v"(1u) : "memory");
      if (__builtin_amdgcn_readfirstlane(nth) == 3) {
        const u32 vtot = *LP(volatile u32, L_VAL);
        if (lane < NR) st4(fdst + (size_t)(call & 1) * FLAG_PAR, (call << 6) | ((16u + vtot * 4u + 63u) >> 6));
        STAMP(1); TR(241);
      }
    }
    role();
    char* const edst = me + WS_E + (size_t)((q * 8 + j) * NSL + ct) * PL + lw * 1024 + l16;
    char* const eflag = me + WS_EFLAG + (size_t)(q * 8 + j) * 512 + ct * 16 + lw * 4;
    int pend = -1;
    TDECL;
#pragma unroll 1
    for (int t = 0; t < 8; ++t) {
      if ((int)__builtin_amdgcn_readfirstlane(*LP(volatile u32, L_CVL)) <= t) {
        if (pend >= 0) { waitv<0>(); if (lane == 0) st4e(eflag + pend * 64, call); pend = -1; }
        while ((int)__builtin_amdgcn_readfirstlane(*LP(volatile u32, L_CVL)) <= t) __builtin_amdgcn_s_sleep(2);
      }
      cfence();
      TSTART();
      const u32 S = LDS_CV + t * CVS;
      const u32 info = __builtin_amdgcn_readfirstlane(*LP(const u32, S + 2048 + lw * 4));
      const u32x2 cc = *LP(const u32x2, S + lw * 512 + lane * 8);
      const u32 V = S + 2064 + (info & 0x1FFu) * 4;
      const u32 n2d = (info >> 10) & 0x7Fu, n3d = (info >> 17) & 0x1Fu, bb = (info >> 27) & 15u;
      u32 l2 = *LP(const u32, V + lane * 4);
      u32x4 ep;
      if (info >> 31) {
        ep = u32x4{cc[0], cc[1], l2, *LP(const u32, V + 256 + lane * 4)};
      } else {
        const u32 l3 = *LP(const u32, V + n2d * 4 + (lane >> 2) * 4);
        u32 c1[8], c2[4]; u32x2 e1[8], e2[4];
#pragma unroll
        for (int k = 0; k < 8; ++k) { c1[k] = (cc[k >> 2] >> (2 * (k & 3))) & M03; e1[k] = dentry(c1[k], (k & 3) != 3); }
        SCHED();
        l2 = (u32)lane < n2d ? l2 : 0u;
        u32 w4 = 0, c3 = (l3 >> (2 * (lane & 3))) & M03;
#pragma unroll
        for (int k = 0; k < 4; ++k) { c2[k] = (l2 >> (2 * k)) & M03; e2[k] = dentry(c2[k], k != 3); }
        SCHED();
        const u32 bbv = bb * B1;
        if (info & (0x1Fu << 22)) {
          c3 = (u32)lane < 4 * n3d ? c3 : 0u;
          const u32x2 e3 = dentry(c3);
          const u32 l4 = *LP(const u32, V + (n2d + n3d + (lane & 15)) * 4) ^ 0x44444444u;
          const u32 K4 = ((bb + 9u) & 15u) * B1;
          *LP(u32x2, scr + A4O + (lane & 15) * 8) = u32x2{(((l4 & M0F) + K4) & M0F) << 3, ((((l4 >> 4) & M0F) + K4) & M0F) << 3};
          cfence();
          const u32 o4 = scr + A4O + min(wave_scan(e3[1]) - e3[1], 128u);
          const u32x2 ww = pair(o4);
          w4 = __builtin_amdgcn_alignbyte(ww[1], ww[0], o4);
          c3 += e3[0];
        }
        *LP(u32, scr + A3O + lane * 4) = __builtin_amdgcn_perm(w4, ((bbv + 0x000A0B0Cu) & M0F) << 3, c3);
        const u32 ta = __builtin_popcount(cc[0] & (cc[0] >> 1) & 0x55555555u), tb = __builtin_popcount(cc[1] & (cc[1] >> 1) & 0x55555555u);
        const u32 t2 = __builtin_popcount(l2 & (l2 >> 1) & 0x55555555u);
        const u32 tt = ta | (tb << 11) | (t2 << 22);
        const u32 sc = wave_scan(tt);
        const u32 na = __builtin_amdgcn_readlane(sc, 63) & 0x7FFu;
        const u32 ex = sc - tt;
        cfence();
        u32 o2[4], o1[8]; u32x2 w[8];
        o2[0] = (ex >> 22) + (scr + A3O);
#pragma unroll
        for (int k = 1; k < 4; ++k) o2[k] = o2[k - 1] + e2[k - 1][1];
#pragma unroll
        for (int k = 0; k < 4; ++k) w[k] = pair(o2[k]);
        SCHED();
        o1[0] = (ex & 0x7FFu) + scr; o1[4] = ((ex >> 11) & 0x7FFu) + (scr + na);
#pragma unroll
        for (int k = 1; k < 8; ++k) if (k != 4) o1[k] = o1[k - 1] + e1[k - 1][1];
#pragma unroll
        for (int k = 0; k < 4; ++k) c2[k] += e2[k][0];
        SCHED();
        u32x4 y;
#pragma unroll
        for (int k = 0; k < 4; ++k) y[k] = expand(w[k], o2[k], ((bbv + 0x000D0E02u) & M0F) << 3, c2[k]);
        *LP(u32x4, scr + l16) = y;
        cfence();
#pragma unroll
        for (int k = 0; k < 8; ++k) w[k] = pair(o1[k]);
        SCHED();
#pragma unroll
        for (int k = 0; k < 8; ++k) c1[k] += e1[k][0];
        SCHED();
        const u32 T1 = ((bbv + 0x00000F01u) & M0F) << 3;
#pragma unroll
        for (int k = 0; k < 4; ++k) ep[k] = expand(w[k], o1[k], T1, c1[k]) | rotl4(expand(w[4 + k], o1[4 + k], T1, c1[4 + k]));
      }
      st16e(edst + (size_t)t * 4 * PL, ep);
      if (lw == 0) TR(72 + t);
      TACC(_ta);
      if (pend >= 0) { waitv<1>(); if (lane == 0) st4e(eflag + pend * 64, call); }
      pend = t;
      TACC(_tb);
    }
    waitv<0>();
    if (lane == 0) st4e(eflag + 7 * 64, call);
    if (lw == 0) { STAMP(8); TOUT(32, _ta / 8); TOUT(33, _tb / 8); }
  } else if (wid == 9) {
    if (b < 8) asm volatile("buffer_inv sc0 sc1" ::: "memory");
    bar();
    const u32 call = __builtin_amdgcn_readfirstlane(callv) + 1u;
    waitv<0>();
    if (b < 8 && lane < 4) st4(inv + lane * 4, call);
    while (__builtin_amdgcn_readfirstlane(*LP(volatile u32, L_PUB)) < 4) __builtin_amdgcn_s_sleep(4);
    role();
    if (D_UF) __builtin_amdgcn_s_sleep(D_UF);
    const u64 t0 = __builtin_amdgcn_s_memrealtime();
    {
      const char* f = me + WS_FLAG + (size_t)(call & 1) * FLAG_PAR + (size_t)((src * 8 + j) * NSL + (lane & 31)) * 128;
      u32 go = 1, fv;
      while (true) {
        fv = ld4(f);
        const bool ok = (fv >> 6) >= (call & 0x3FFFFFFu);
        if ((lane & 35) == ct && ok) *LP(volatile u32, L_LEN + (lane >> 2) * 4) = max(min(fv & 63u, 33u), 1u) * 64u + 2048u;
        if (__all(ok)) break;
        if (expired(t0, tmo, call)) { go = 2; *LP(volatile u32, L_BAD) = 1; break; }
        __builtin_amdgcn_s_sleep(S_UF);
      }
      *LP(volatile u32, L_NGO) = go;
      TR(246);
    }
    const char* const f = lane < 32 ? me + WS_EFLAG + (size_t)(q * 8 + j) * 512 + l16
                                    : me + WS_PFLAG + (size_t)(src * 8 + j) * 128 + (lane & 7) * 16;
    u64 known = ~0ull << (32 + NPUSH);
    int nready = 0;
    while (true) {
      const int upto = 4 * min((int)__builtin_amdgcn_readfirstlane(*LP(volatile u32, L_CVL)) + 1, 8);
      bool ok = false;
      if (!((known >> lane) & 1) && (lane >= 32 || lane < upto)) {
        u32x4 v;
        asm volatile("global_load_dwordx4 %0, %1, off" EPL "\n\ts_waitcnt vmcnt(0)" : "=&v"(v) : "v"(f) : "memory");
        ok = min(min(v[0], v[1]), min(v[2], v[3])) >= call;
      }
      known |= __ballot(ok);
      nready = __builtin_ctzll(~(u64)((u32)known & (u32)(known >> 32)));
      if (nready == NSL) break;
      *LP(volatile u32, L_RDY) = nready;
      if (expired(t0, tmo, call)) break;
      __builtin_amdgcn_s_sleep(PSLEEP);
    }
    *LP(volatile u32, L_RDY) = NSL;
    if (__builtin_amdgcn_readfirstlane(ld4(tmo)) == call) *LP(volatile u32, L_BAD) = 1;
    *LP(volatile u32, L_RDY) = NSL + 1;
    STAMP(13);
  } else {
    __builtin_amdgcn_s_setprio(3);
    *LP(u32x4, LDS_MISC) = u32x4{0, 0, 0, 0};
    *LP(u32x4, LDS_MISC + 16) = u32x4{0, NPUSH, 0, 0};
    *LP(u32, L_ROT) = 0;
    *LP(u32x4, L_LEN) = u32x4{0, 0, 0, 0}; *LP(u32x4, L_LEN + 16) = u32x4{0, 0, 0, 0};
    bar();
    const u32 call = __builtin_amdgcn_readfirstlane(callv) + 1u;
    asm volatile("" :: "s"(call));
    const u32 tstart = (u32)__builtin_amdgcn_s_memrealtime();
    {
      if (b == 0 && lane < 32) st4(me + WS_ROT + lane * 128, (call << 4) | (xcc & 7u));
      const u64 t0 = __builtin_amdgcn_s_memrealtime();
      u32 r;
      while ((u32)__builtin_amdgcn_s_memrealtime() - tstart < D_ROT) __builtin_amdgcn_s_sleep(1);
      while (((r = __builtin_amdgcn_readfirstlane(ld4(me + WS_ROT + (b & 31) * 128))) >> 4) < (call & 0xFFFFFFFu)) {
        if (expired(t0, tmo, call)) { *LP(volatile u32, L_BAD) = 1; r = 0; break; }
        __builtin_amdgcn_s_sleep(S_ROT);
      }
      *LP(volatile u32, L_ROT) = (r & 7u) + 1u;
      role();
      TR(248);
    }
    {
      const char* f = lane < NE ? me + WS_NFLAG + (size_t)((src * 8 + j) * 8 + lane) * 128 : inv;
      while ((u32)__builtin_amdgcn_s_memrealtime() - tstart < D_EF) __builtin_amdgcn_s_sleep(1);
      const u64 t0 = __builtin_amdgcn_s_memrealtime();
      while (true) {
        u32x4 v;
        asm volatile("global_load_dwordx4 %0, %1, off sc0 sc1\n\ts_waitcnt vmcnt(0)" : "=&v"(v) : "v"(f) : "memory");
        const u64 ok = __ballot(min(min(v[0], v[1]), min(v[2], v[3])) >= call);
        if (ok == ~0ull) break;
        if ((~ok & ((1u << NE) - 1)) == 0 && __builtin_amdgcn_s_memrealtime() - t0 > 500) {
          asm volatile("buffer_inv sc0 sc1\n\ts_waitcnt vmcnt(0)" ::: "memory");
          break;
        }
        if (expired(t0, tmo, call)) {
          *LP(volatile u32, L_BAD) = 1; *LP(volatile u32, L_CVL) = 8; *LP(volatile u32, L_NL) = NSL + KT;
          return;
        }
        __builtin_amdgcn_s_sleep(S_EF);
      }
      STAMP(2); TR(240);
      waitv<0>();
    }
    u32 lim = 0;
    bool dead = false;
    const char* const base = a.ws[(src - rank) & (NR - 1)] + (size_t)(call & 1) * WS_SLOT + (size_t)(j * NSL) * USTRIDE;
    const u32 tau = q ? TAU : TAU * LOCF / 8;
    u32 due = (tstart + (u32)max((int)((u32)__builtin_amdgcn_s_memrealtime() - tstart) - CMAX, TREF)) << 4;
    auto pace = [&](u32 bytes) {
      while ((int)(((u32)__builtin_amdgcn_s_memrealtime() << 4) - due) < 0) {}
      due += bytes * tau >> 10;
    };
    int ct_ = 0, cp_ = 0;
    auto issue_c = [&]() {
      const char* sb = base + (size_t)(4 * ct_ + ct) * USTRIDE + U_C;
      const u32 la = LDS_CV + ct_ * CVS;
      if (cp_ == 0) {
        while (!(lim = __builtin_amdgcn_readfirstlane(*LP(volatile u32, L_LEN + ct_ * 4))))
          if (__builtin_amdgcn_readfirstlane(*LP(volatile u32, L_NGO)) == 2) { dead = true; break; }
        if (ct_ == 0) { const u32 now = ((u32)__builtin_amdgcn_s_memrealtime() - CMAX) << 4; if ((int)(now - due) > 0) due = now; TR(247); }
      }
      if (dead) return;
      pace(cp_ == 2 ? 4u * (lim - 2048u) : 4096u);
      if (cp_ == 2 && lim > 3072u) {
        set_m0(la + 3072); if (l16 + 3072u < lim) glds<0, CSYS>(sb, l16 + 3072u);
        set_m0(la + 4096); if (l16 + 4096u < lim) glds<0, CSYS>(sb, l16 + 4096u);
      }
      set_m0(la + cp_ * 1024);
      if (l16 + cp_ * 1024u < lim) glds<0, CSYS>(sb, l16 + cp_ * 1024u);
      if (++cp_ == 3) { cp_ = 0; ++ct_; }
    };
    TDECL;
    int arrived = 0;
    auto issue_n = [&](int s, int p) {
      const char* sb = base + (size_t)s * USTRIDE;
      pace(1024u);
      if (p == 0) {
        TACC(_ta);
        while (s >= 8 && arrived < 4 * (s - 5)) arrived = __builtin_amdgcn_readfirstlane(*LP(volatile u32, L_ARR));
        TACC(_tb);
        set_m0(LDS_N + (s & 7) * PL); glds<0, false>(sb, l16);
      }
      if (p == 1) glds<1024, false>(sb, l16);
      if (p == 2) glds<2048, false>(sb, l16);
      if (p == 3) glds<3072, false>(sb, l16);
    };
    auto post = [&](u32 addr, u32 v) { *LP(volatile u32, addr) = v; TR(addr == L_NL ? (v > NSL - KT ? (v == NSL + KT ? 31 : 250) : v - 1) : 64 + v - 1); };
    constexpr int NB = 24 - 3 * PRE;
    static_assert(PRE >= 2 && PRE <= 7 && W2 >= 4 && W2 <= 8 && N0 + NB + 4 <= NSL, "");
    int cdone = 0;
    auto landed_c = [&]() { if (cdone < 16) TR(224 + cdone); ++cdone; if (cdone % 3 == 0) post(L_CVL, cdone / 3); };
#pragma unroll 1
    for (int s = NPUSH; s < N0; ++s) {
#pragma unroll
      for (int p = 0; p < 4; ++p) { waitv<W2 - 1>(); issue_n(s, p); if (p == 0 || p == 3) TR(200 + (s - NPUSH) * 2 + (p == 3)); }
    }
    waitv<3>(); issue_c();
    waitv<1>(); post(L_NL, N0); issue_c();
    TSTART();
#pragma unroll 1
    for (int n = 0; n < 3 * PRE - 2; ++n) { waitv<1>(); issue_c(); landed_c(); }
    waitv<1>(); landed_c();
    u32 go;
    do go = __builtin_amdgcn_readfirstlane(*LP(volatile u32, L_NGO)); while (!go);
    if (go == 2) { waitv<0>(); post(L_CVL, 8); post(L_NL, NSL + KT); return; }
#pragma unroll
    for (int p = 0; p < 4; ++p) issue_n(N0, p);
    waitv<4>(); issue_c(); landed_c();
#pragma unroll 1
    for (int s = N0; s < N0 + NB; ++s) {
#pragma unroll
      for (int p = 0; p < 4; ++p) { waitv<4>(); if (p == 3) post(L_NL, s + 1); issue_n(s + 1, p); }
      waitv<4>();
      if (s + 1 < N0 + NB) issue_c();
      landed_c();
    }
#pragma unroll
    for (int n = 4; n < W2; ++n) issue_n(N0 + NB + n / 4, n % 4);
    RMARK();
    constexpr int S1 = NSL - KT - (W2 + 3) / 4;
    static_assert(S1 >= N0 + NB, "");
#pragma unroll 1
    for (int s = N0 + NB; s < S1; ++s) {
#pragma unroll
      for (int p = 0; p < 4; ++p) { waitv<W2 - 1>(); if (p == 3) post(L_NL, s + 1); issue_n(s + (p + W2) / 4, (p + W2) % 4); }
    }
    constexpr int NA = 4 * (NSL - KT - S1), NZ = NA + 4 * KT;
#pragma unroll
    for (int n = 0; n < NZ; ++n) {
      waitn(NZ - 1 - n < W2 - 1 ? NZ - 1 - n : W2 - 1);
      if (n < NA ? n % 4 == 3 : n % 2 == 1) post(L_NL, n < NA ? S1 + n / 4 + 1 : NSL - KT + (n - NA) / 2 + 1);
      const int m = n + W2;
      if (m < NA) issue_n(S1 + m / 4, m % 4);
      else if (m < NZ) {
        const int hh = (m - NA) / 2, s = NSL - KT + hh % KT, p = (hh / KT) * 2 + (m - NA) % 2;
        if (p == 2) set_m0(LDS_N + (s & 7) * PL);
        issue_n(s, p);
      }
    }
    STAMP(4); TOUT(39, _tb / (NSL - NPUSH)); TOUT(12, _r1);
  }
}

extern "C" void kernel_launch(int rank, int nranks, ncclComm_t comm,
                              hipStream_t stream,
                              void* const* d_in, const int* in_sizes, int n_in,
                              void* d_out, int out_size,
                              const int* in_dtypes, int out_dtype,
                              void* const* d_out_all,
                              void* const* d_ws_all, size_t ws_bytes) {
  if (nranks != NR || (unsigned)rank >= (unsigned)NR || ws_bytes < WS_END || n_in < 4) return;
  static bool init = false;
  if (!init) {
    HIP_CHECK(hipFuncSetAttribute((const void*)ag_gemm, hipFuncAttributeMaxDynamicSharedMemorySize, LDS_TOT));
    init = true;
  }
  Args a;
  for (int p = 0; p < NR; ++p) a.ws[p] = (char*)d_ws_all[(rank + p) & (NR - 1)];
  a.x = (const unsigned char*)d_in[0]; a.w = (const unsigned char*)d_in[1];
  a.sx = (const float*)d_in[2]; a.sw = (const float*)d_in[3];
  a.out = (float*)d_out; a.rank = rank;
  {
    const unsigned cl = CLIQUE;
    unsigned o = 0; int n = 0;
    for (int pass = 0; pass < 2; ++pass)
      for (int i = 0; i < NR; ++i) { const int p = (rank + i) & 7; if ((int)(((cl >> p) ^ (cl >> rank)) & 1) == pass) o |= (unsigned)p << (4 * n++); }
    a.order = o;
  }
  hipLaunchKernelGGL(ag_gemm, dim3(LNWG), dim3(LNTHR), LLDS, stream, a);
  HIP_CHECK(hipGetLastError());
}
```

```cpp
#include <hip/hip_runtime.h>
#include <rccl/rccl.h>
#include <cstdio>
#include <cstdint>

#pragma clang diagnostic ignored "-Wint-to-pointer-cast"

#define HIP_CHECK(expr) do { hipError_t _e = (expr); if (_e != hipSuccess) { \
    fprintf(stderr, "HIP error %s at %s:%d\n", hipGetErrorString(_e), __FILE__, __LINE__); \
    return; } } while (0)

typedef unsigned u32;
typedef unsigned u32x2 __attribute__((ext_vector_type(2)));
typedef unsigned u32x4 __attribute__((ext_vector_type(4)));
typedef int v2i __attribute__((ext_vector_type(2)));
typedef int v8i __attribute__((ext_vector_type(8)));
typedef float v16f __attribute__((ext_vector_type(16)));
typedef unsigned long long u64;
#define LAS __attribute__((address_space(3)))
#define LP(T, a) ((LAS T*)(u32)(a))

constexpr int NPUSH = 3;
constexpr int PACE = 60;
constexpr int NE = 3;
constexpr int PRE = 5;
constexpr int W2 = 6;
constexpr int KT = 3;
constexpr int TAU = 219;
constexpr int TREF = 190;
constexpr int CMAX = 90;
constexpr unsigned CLIQUE = 0x4D;

constexpr int NR = 8;
constexpr int KTOT = 4096, NL = 512, MS = 512;
constexpr int BM = 64, BN = 128, KS = 128, NSL = KTOT / KS;
constexpr int NWG = 256, NTHR = 640;
constexpr int PL = 4096;
constexpr int U_C = 4096, U_H = 6144, U_V = 6160, USTRIDE = 9216;
constexpr int N0 = NPUSH + NE;
static_assert(NPUSH >= 1 && NPUSH <= 3 && NE >= 1 && N0 <= 8, "");
constexpr size_t WS_SLOT  = (size_t)NWG * USTRIDE;
constexpr size_t WS_PUSH  = 2 * WS_SLOT;
constexpr size_t PUSH_PAR = (size_t)NR * 8 * 8 * PL;
constexpr size_t WS_E     = WS_PUSH + 2 * PUSH_PAR;
constexpr size_t WS_FLAG  = WS_E + (size_t)64 * NSL * PL;
constexpr size_t FLAG_PAR = (size_t)NR * 8 * NSL * 128;
constexpr size_t WS_PFLAG = WS_FLAG + 2 * FLAG_PAR;
constexpr size_t WS_NFLAG = WS_PFLAG + NR * 8 * 128;
constexpr size_t WS_EFLAG = WS_NFLAG + NR * 8 * 8 * 128;
constexpr size_t WS_CNT   = WS_EFLAG + 64 * 512;
constexpr size_t WS_INV   = WS_CNT + NWG * 128;
constexpr size_t WS_ROT   = WS_INV + 16 * 128;
constexpr size_t WS_TMO   = WS_ROT + 32 * 128;
constexpr size_t WS_END   = WS_TMO + 128;
constexpr u64 DEADLINE = 30000000ull;
constexpr u64 SLOW = 20000ull;
constexpr int WSL = KS * BN;
constexpr int LDS_LUT = 0;
constexpr int LDS_CLUT = 128;
constexpr int LDS_MISC = 256;
constexpr int L_BAD = LDS_MISC, L_VAL = LDS_MISC + 4, L_NGO = LDS_MISC + 8, L_PUB = LDS_MISC + 12, L_LEN = LDS_MISC + 32, L_ROT = LDS_MISC + 192;
constexpr int L_ARR = LDS_MISC + 16, L_NL = LDS_MISC + 20, L_RDY = LDS_MISC + 24, L_CVL = LDS_MISC + 28, L_DUMMY = LDS_MISC + 64;
constexpr int SCR = 2048;
constexpr int A3O = 1040, A4O = 1312, VBO = 1536;
constexpr int LDS_SCR = 512;
constexpr int LDS_E = 9 * 1024;
constexpr int LDS_N = LDS_E + 4 * PL;
constexpr int LDS_W = LDS_N + 8 * PL;
constexpr int CVS = 4352;
constexpr int LDS_CV = LDS_W + 4 * WSL;
constexpr int LDS_TOT = LDS_CV + 8 * CVS;
static_assert(LDS_SCR + 4 * SCR <= LDS_E && LDS_TOT <= 160 * 1024, "lds");

struct Args {
  const unsigned char* x;
  char* ws[NR];
  const unsigned char* w;
  const float* sx; const float* sw; float* out;
  int rank; u32 order;
};

#define SCHED() __builtin_amdgcn_sched_barrier(0)
template <int N> __device__ __forceinline__ void waitv() { asm volatile("s_waitcnt vmcnt(%0)" :: "n"(N) : "memory"); }
__device__ __forceinline__ void waitn(int n) {
  switch (n) {
    case 0: waitv<0>(); break; case 1: waitv<1>(); break; case 2: waitv<2>(); break; case 3: waitv<3>(); break;
    case 4: waitv<4>(); break; case 5: waitv<5>(); break; case 6: waitv<6>(); break; default: waitv<7>();
  }
}
__device__ __forceinline__ void cfence() { asm volatile("" ::: "memory"); }
__device__ __forceinline__ void bar() {
  cfence(); SCHED();
  __builtin_amdgcn_s_waitcnt(0xC07F);
  __builtin_amdgcn_s_barrier();
  SCHED(); cfence();
}
__device__ __forceinline__ void set_m0(u32 lds_addr) { asm volatile("s_mov_b32 m0, %0\n\ts_nop 0" :: "s"(lds_addr) : "memory"); }
template <int OFF, bool SYS> __device__ __forceinline__ void glds(const void* sbase, u32 voff) {
  if (SYS) asm volatile("global_load_lds_dwordx4 %0, %1 offset:%2 sc0 sc1" :: "v"(voff), "s"(sbase), "n"(OFF) : "memory");
  else asm volatile("global_load_lds_dwordx4 %0, %1 offset:%2" :: "v"(voff), "s"(sbase), "n"(OFF) : "memory");
}
__device__ __forceinline__ void st16(void* p, u32x4 v) { asm volatile("global_store_dwordx4 %0, %1, off sc0 sc1\n\ts_nop 1" :: "v"(p), "v"(v) : "memory"); }
__device__ __forceinline__ void st8(void* p, u32x2 v) { asm volatile("global_store_dwordx2 %0, %1, off sc0 sc1\n\ts_nop 1" :: "v"(p), "v"(v) : "memory"); }
__device__ __forceinline__ void st4(void* p, u32 v) { asm volatile("global_store_dword %0, %1, off sc0 sc1\n\ts_nop 1" :: "v"(p), "v"(v) : "memory"); }
__device__ __forceinline__ u32 ld4(const void* p) {
  u32 v;
  asm volatile("global_load_dword %0, %1, off sc0 sc1\n\ts_waitcnt vmcnt(0)" : "=&v"(v) : "v"(p) : "memory");
  return v;
}

__device__ __forceinline__ bool expired(u64 t0, u32* tmo, u32 call) {
  const u64 dt = __builtin_amdgcn_s_memrealtime() - t0;
  if (dt <= SLOW) return false;
  if (dt > DEADLINE) { st4(tmo, call); waitv<0>(); return true; }
  __builtin_amdgcn_s_sleep(64);
  return __builtin_amdgcn_readfirstlane(ld4(tmo)) == call;
}

__device__ __forceinline__ u32 wave_scan(u32 x) {
  x += __builtin_amdgcn_update_dpp(0u, x, 0x111, 0xf, 0xf, true);
  x += __builtin_amdgcn_update_dpp(0u, x, 0x112, 0xf, 0xf, true);
  x += __builtin_amdgcn_update_dpp(0u, x, 0x114, 0xf, 0xf, true);
  x += __builtin_amdgcn_update_dpp(0u, x, 0x118, 0xf, 0xf, true);
  x += __builtin_amdgcn_update_dpp(0u, x, 0x142, 0xa, 0xf, false);
  x += __builtin_amdgcn_update_dpp(0u, x, 0x143, 0xc, 0xf, false);
  return x;
}
constexpr u32 B1 = 0x01010101u, M0F = 0x0F0F0F0Fu, MSM = 0x87878787u, MEX = 0x78787878u, M03 = 0x03030303u;
__device__ __forceinline__ u32 rotl4(u32 x) { return __builtin_amdgcn_alignbit(x, x, 28); }
__device__ __forceinline__ u32 rotr4(u32 x) { return __builtin_amdgcn_alignbit(x, x, 4); }
__device__ __forceinline__ u32x2 centry(u32 p) {
  return *LP(const u32x2, __builtin_amdgcn_udot4(p & 0x04040404u, 0x10080402u, (u32)LDS_CLUT, false));
}
__device__ __forceinline__ u32x2 dentry(u32 c, bool with_count = true) {
  const u32 p = __builtin_amdgcn_udot4(__builtin_amdgcn_perm(0u, 0x01000000u, c), 0x40201008u, (u32)LDS_LUT, false);
  if (with_count) return *LP(const u32x2, p);
  return u32x2{*LP(const u32, p), 0u};
}
__device__ __forceinline__ void put4(u32 o, u32 y) {
  const u64 w = (u64)y << ((o & 3u) * 8u);
  LAS u32* p = LP(u32, o & ~3u);
  __hip_atomic_fetch_or(p, (u32)w, __ATOMIC_RELAXED, __HIP_MEMORY_SCOPE_WAVEFRONT);
  __hip_atomic_fetch_or(p + 1, (u32)(w >> 32), __ATOMIC_RELAXED, __HIP_MEMORY_SCOPE_WAVEFRONT);
}
__device__ __forceinline__ u32x2 pair(u32 o) {
  const LAS u32* p = LP(const u32, o & ~3u);
  return u32x2{p[0], p[1]};
}
__device__ __forceinline__ u32 expand(u32x2 w, u32 o, u32 table, u32 sel) {
  return __builtin_amdgcn_perm(__builtin_amdgcn_alignbyte(w[1], w[0], o), table, sel);
}

__global__ void __launch_bounds__(NTHR, 3) ag_gemm(Args a) {
  const int tid = threadIdx.x, lane = tid & 63;
  const int wid = __builtin_amdgcn_readfirstlane(tid >> 6);
  const int b = blockIdx.x;
  char* const me = a.ws[0];
  const int jp = b & 7, idx = b >> 3, ct = idx & 3, pe = idx >> 3;
  const int ps = pe == 3 ? NPUSH + (idx & 7) : pe;
  const int brow = (wid & 3) * 16 + (lane >> 2);
  const unsigned char* const xlane = a.x + (size_t)(jp * BM) * KTOT + (wid < 4 ? ps : idx) * KS + (u32)(brow * KTOT + (((lane & 3) ^ ((brow >> 2) & 3)) << 5));
  const u32x4 v0 = *(const u32x4*)xlane, v1 = *(const u32x4*)(xlane + 16);
  u32* const cnt = (u32*)(me + WS_CNT + (size_t)b * 128);
  const u32 callv = __hip_atomic_load(cnt, __ATOMIC_RELAXED, __HIP_MEMORY_SCOPE_AGENT);
  cfence(); SCHED();
  const int rank = a.rank;
  int j, q, src;
  auto role = [&]() {
    u32 r;
    while (!(r = __builtin_amdgcn_readfirstlane(*LP(volatile u32, L_ROT)))) __builtin_amdgcn_s_sleep(1);
    const int x = (b + r - 1) & 7;
    j = (x & 3) * 2 + ((idx >> 2) & 1); q = (x & 4) | (idx >> 3);
    src = (a.order >> (4 * q)) & (NR - 1);
  };
  const u32 xcc = __builtin_amdgcn_s_getreg((3 << 11) | 20) & 15u;
  u32* const tmo = (u32*)(me + WS_TMO);
  const u32 dummy = L_DUMMY + (lane & 31) * 4;
  const u32 l16 = lane * 16u;
  char* const inv = me + WS_INV + (size_t)xcc * 128;

  if (wid < 4) {
    const bool early = pe == 3 && (idx & 7) < NE;
    auto nplane = [&](const u32x4 v0, const u32x4 v1) {
      u32x4 np;
#pragma unroll
      for (int k = 0; k < 4; ++k) np[k] = (v0[k] & MSM) | rotl4(v1[k] & MSM);
      return np;
    };
    char* nflag = a.ws[lane & (NR - 1)] + WS_NFLAG + (size_t)((rank * 8 + jp) * 8 + (idx & 7)) * 128 + wid * 4;
    if (tid < 16) {
      u32 add = 0, cs = 0x0C0C0C0Cu, n = 0;
#pragma unroll
      for (int i = 0; i < 4; ++i)
        if ((tid >> i) & 1) { add |= (1 + n) << (8 * i); cs = (cs & ~(0xFFu << (8 * n))) | (i << (8 * n)); ++n; }
      *LP(u32x2, LDS_LUT + tid * 8) = u32x2{add, n};
      *LP(u32x2, LDS_CLUT + tid * 8) = u32x2{cs, n};
    }
    const int kr = lane >> 3, cp = lane & 7;
    u32 wvo[4];
#pragma unroll
    for (int i = 0; i < 4; ++i) wvo[i] = (kr + i * 8) * NL + ((cp >> 1) ^ ((kr >> 1) & 3)) * 32 + (cp & 1) * 16 - i * 1024;
    const unsigned char* const wsrc = a.w + (size_t)(wid * 32) * NL + ct * BN;
    auto stage_w = [&](int s) {
      set_m0(LDS_W + (s & 3) * WSL + wid * 4096);
      const unsigned char* sb = wsrc + (size_t)s * KS * NL;
      glds<0, false>(sb, wvo[0]); glds<1024, false>(sb, wvo[1]); glds<2048, false>(sb, wvo[2]); glds<3072, false>(sb, wvo[3]);
    };
    char* const pdst = a.ws[idx & (NR - 1)];
    char* pp = (pe == 3 ? me + (size_t)(jp * NSL + ps) * USTRIDE : pdst + WS_PUSH + (size_t)((rank * 8 + jp) * 8 + pe) * PL) + wid * 1024 + l16;
    const size_t podd = pe == 3 ? WS_SLOT : PUSH_PAR;
    asm volatile("" : "+v"(pp));
    bar();
    const u32x4 np0 = nplane(v0, v1);
    const u32 call = __builtin_amdgcn_readfirstlane(callv) + 1u;
    const size_t par = call & 1;
    if (pe == 1 || pe == 2) {
      asm volatile("" :: "v"(np0));
      const u64 tpush = __builtin_amdgcn_s_memrealtime();
      while (__builtin_amdgcn_s_memrealtime() - tpush < (u64)(pe * PACE)) __builtin_amdgcn_s_sleep(1);
    }
    if (pe < NPUSH || early) st16(pp + par * podd, np0);
    if (early) {
      asm volatile("" : "+v"(nflag));
      waitv<0>();
      if (lane < NR) st4(nflag, call);
    }
    const size_t pfoff = WS_PFLAG + (size_t)(rank * 8 + jp) * 128 + wid * 4;
    float sc = a.sx[0] * a.sw[0];
    while (__builtin_amdgcn_readfirstlane(*LP(volatile u32, L_PUB)) < 4) __builtin_amdgcn_s_sleep(4);
    __builtin_amdgcn_s_sleep(32);
    waitv<0>();
    if (lane == 0 && pe < NPUSH) st4(pdst + pfoff + pe * 16, call);
    asm volatile("" : "+v"(sc));
    role();
    const char* const esrc = me + WS_E + (size_t)((q * 8 + j) * NSL) * PL + wid * 1024;
    const char* const psrc = me + WS_PUSH + par * PUSH_PAR + (size_t)((src * 8 + j) * 8) * PL + wid * 1024;
    auto stage_e = [&](int s, bool push) {
      if (push) { set_m0(LDS_N + (s & 7) * PL + wid * 1024); glds<0, true>(psrc + s * PL, l16); }
      set_m0(LDS_E + (s & 3) * PL + wid * 1024); glds<0, true>(esrc + s * PL, l16);
    };
    v16f acc[2] = {};
    const int r31 = lane & 31, h = lane >> 5, wm = wid >> 1, wn = wid & 1;
    const int q8 = (lane & 15) >> 1, p8 = lane & 1, g1 = (lane >> 4) & 1;
    u32 boff[2], aoff[2];
#pragma unroll
    for (int i = 0; i < 2; ++i) {
      boff[i] = LDS_W + (32 * h + q8) * 128 + (((wn * 2 + i) ^ ((q8 >> 1) & 3)) << 5) + g1 * 16 + p8 * 8;
      aoff[i] = (wm * 32 + r31) * 64 + (((i * 2 + h) ^ ((r31 >> 2) & 3)) << 4);
    }
    struct Frag { u32x4 ep[2]; v8i bf[2][2]; };
    auto ldfrag = [&](Frag& f, int kk, int s) {
      f.ep[kk] = *LP(const u32x4, aoff[kk] + (s & 3) * PL + LDS_E);
#pragma unroll
      for (int n = 0; n < 2; ++n) {
        v2i t[4];
#pragma unroll
        for (int r = 0; r < 4; ++r) t[r] = __builtin_amdgcn_ds_read_tr8_b64_v2i32(LP(v2i, boff[n] + (s & 3) * WSL) + (kk * 64 + r * 8) * 16);
        f.bf[kk][n] = v8i{t[0][0], t[0][1], t[1][0], t[1][1], t[2][0], t[2][1], t[3][0], t[3][1]};
      }
    };
    auto afrag = [&](const u32x4 np, const u32x4 ep) {
      v8i af;
#pragma unroll
      for (int k = 0; k < 4; ++k) {
        af[k] = (np[k] & MSM) | (ep[k] & MEX);
        af[4 + k] = rotr4((np[k] & MEX) | (ep[k] & MSM));
      }
      return af;
    };
#define MFMA(n, af, bf) acc[n] = __builtin_amdgcn_mfma_scale_f32_32x32x64_f8f6f4(af, bf, acc[n], 0, 0, 0, 0, 0, 0)
    auto step = [&](int s, const Frag& cur, Frag& nxt, bool more, bool stg) {
      const u32x4 np0 = *LP(const u32x4, aoff[0] + (s & 7) * PL + LDS_N);
      const u32x4 np1 = *LP(const u32x4, aoff[1] + (s & 7) * PL + LDS_N);
      SCHED();
      if (stg) { stage_e(s + 4, false); stage_w(s + 4); }
      SCHED();
      const v8i a0 = afrag(np0, cur.ep[0]);
      MFMA(0, a0, cur.bf[0][0]);
      SCHED();
      const v8i a1 = afrag(np1, cur.ep[1]);
      SCHED();
      if (!more) {
        MFMA(0, a1, cur.bf[1][0]); MFMA(1, a0, cur.bf[0][1]); MFMA(1, a1, cur.bf[1][1]);
        return;
      }
      MFMA(1, a0, cur.bf[0][1]);
      SCHED();
      ldfrag(nxt, 0, s + 1);
      SCHED();
      MFMA(0, a1, cur.bf[1][0]);
      SCHED();
      ldfrag(nxt, 1, s + 1);
      SCHED();
      MFMA(1, a1, cur.bf[1][1]);
      SCHED();
      asm volatile("" :: "v"(a0), "v"(cur.bf[0][0]), "v"(cur.bf[0][1]), "v"(cur.bf[1][0]));
    };
    const u32 arr = lane ? dummy : (u32)L_ARR;
    auto count = [&](u32 addr) { asm volatile("ds_add_u32 %0, %1" :: "v"(addr), "v"(1u) : "memory"); };
    auto spin = [&](int na, int nl, int nr, int nc = 0) {
      while (true) {
        u32x4 w;
        asm volatile("ds_read_b128 %0, %1\n\ts_waitcnt lgkmcnt(0)" : "=&v"(w) : "v"((u32)L_ARR) : "memory");
        if (__all((int)w[0] >= na && (int)w[1] >= nl && (int)w[2] >= nr && (int)w[3] >= nc)) break;
      }
    };
    auto meet = [&](int s) {
      SCHED();
      asm volatile("s_waitcnt lgkmcnt(0)\n\tds_add_u32 %0, %1" :: "v"(arr), "v"(1u) : "memory");
      spin(4 * s + 8, s + 1, min(s + 5, NSL));
      __builtin_amdgcn_s_waitcnt(0xC07F);
      SCHED();
    };
    {
      Frag fa, fb;
#pragma unroll 1
      for (int s = 0; s < 4; ++s) { stage_w(s); __builtin_amdgcn_s_sleep(8); }
      spin(0, 0, 4);
#pragma unroll 1
      for (int s = 0; s < 4; ++s) stage_e(s, s < NPUSH);
      waitv<0>(); meet(-1);
      ldfrag(fa, 0, 0); ldfrag(fa, 1, 0);
      meet(0); step(0, fa, fb, true, true);
#pragma unroll 1
      for (int s = 1; s < NSL - 5; s += 2) {
        waitv<10>(); meet(s);
        step(s, fb, fa, true, true);
        waitv<10>(); meet(s + 1);
        step(s + 1, fa, fb, true, true);
      }
      waitv<10>(); meet(NSL - 5); step(NSL - 5, fb, fa, true, true);
      static_assert(KT == 3, "");
      waitv<10>(); meet(NSL - 4); step(NSL - 4, fa, fb, true, false);
      const int hb = wm * KT;
      const u32 arr2 = lane ? dummy : (u32)L_CVL;
      waitv<5>(); count(arr);
      if (wm) { waitv<0>(); count(arr2); }
      SCHED(); spin(4 * (NSL - 3) + 8, NSL - 2 + hb, 0, 0); SCHED(); step(NSL - 3, fb, fa, true, false);
      if (!wm) { waitv<0>(); count(arr2); }
      SCHED(); spin(0, NSL - 1 + hb, 0, 12); SCHED(); step(NSL - 2, fa, fb, true, false);
      spin(0, 0, NSL + 1, 0);
      if (*LP(volatile u32, L_BAD)) sc = __builtin_nanf("");
      asm volatile("" : "+v"(sc));
      SCHED(); spin(0, NSL + hb, 0, 0); SCHED(); step(NSL - 1, fb, fa, false, false);
    }
    if (tid == 0) st4(cnt, call);
    float* o = a.out + (size_t)(src * MS + j * BM + wm * 32 + 4 * h) * NL + ct * BN + wn * 64 + r31;
#pragma unroll
    for (int n = 0; n < 2; ++n)
#pragma unroll
      for (int i = 0; i < 16; ++i)
        __hip_atomic_store(o + (size_t)((i & 3) + 8 * (i >> 2)) * NL + n * 32, acc[n][i] * sc, __ATOMIC_RELAXED, __HIP_MEMORY_SCOPE_SYSTEM);
  } else if (wid < 8) {
    const int lw = wid - 4;
    const u32 scr = LDS_SCR + lw * SCR;
    u32 call;
    {
      char* fdst = a.ws[lane & (NR - 1)] + WS_FLAG + (size_t)((rank * 8 + jp) * NSL + idx) * 128;
      *LP(u32x4, scr + l16) = u32x4{0, 0, 0, 0};
      *LP(u32x2, scr + 1024 + lane * 8) = u32x2{0, 0};
      bar();
      call = __builtin_amdgcn_readfirstlane(callv) + 1u;
      asm volatile("" : "+v"(fdst));
      char* const dst = me + (size_t)(call & 1) * WS_SLOT + (size_t)(jp * NSL + idx) * USTRIDE;
      const u32 vbuf = scr + VBO;
      const u32 v[8] = {v0[0], v0[1], v0[2], v0[3], v1[0], v1[1], v1[2], v1[3]};
      u32 z[8], sum = 0;
#pragma unroll
      for (int k = 0; k < 8; ++k) { z[k] = v[k] & MEX; if (k & 1) sum = __builtin_amdgcn_udot4(z[k], B1, sum, false); }
      sum = __builtin_amdgcn_readlane(wave_scan(sum), 63);
      const u32 bb = ((sum + 7946u) >> 13) & 15u;
      const u32 kz = ((7u - bb) & 15u) * 0x08080808u;
      u32x4 np;
#pragma unroll
      for (int k = 0; k < 4; ++k) np[k] = (v[k] & MSM) | rotl4(v[4 + k] & MSM);
      if (idx >= N0) st16(dst + lw * 1024 + l16, np);
      u32x2 e1[8], cp2 = {0, 0};
#pragma unroll
      for (int k = 0; k < 8; ++k) {
        z[k] = (((z[k] + kz) >> 3) & M0F) ^ 0x04040404u;
        const u32 p = __builtin_amdgcn_perm(0xFFFFFFFFu, 0x8201FFFFu, z[k]);
        cp2[k >> 2] |= (p & M03) << (2 * (k & 3));
        e1[k] = centry(p);
      }
      const u32 t1 = (e1[0][1] + e1[1][1] + e1[2][1] + e1[3][1]) | ((e1[4][1] + e1[5][1] + e1[6][1] + e1[7][1]) << 16);
      const u32 sc1 = wave_scan(t1);
      const u32 tot1 = __builtin_amdgcn_readlane(sc1, 63);
      const u32 n2 = (tot1 & 0xFFFFu) + (tot1 >> 16);
      bool raw = n2 > 1024u;
      u32 n2d = 0, n3d = 0, n4d = 0;
      if (!raw) {
        const u32 ex = sc1 - t1 + (scr | ((scr + (tot1 & 0xFFFFu)) << 16));
        u32 o = ex & 0xFFFFu;
#pragma unroll
        for (int k = 0; k < 8; ++k) {
          if (k == 4) o = ex >> 16;
          put4(o, __builtin_amdgcn_perm(0u, z[k], e1[k][0])); o += e1[k][1];
        }
        cfence();
        const u32x4 q2 = *LP(const u32x4, scr + l16);
        u32 l2 = 0; u32x2 e2[4];
#pragma unroll
        for (int k = 0; k < 4; ++k) {
          const u32 p = __builtin_amdgcn_perm(0xFFFFFFFFu, 0xFFFF8201u, q2[k] ^ B1);
          l2 |= (p & M03) << (2 * k); e2[k] = centry(p);
        }
        *LP(u32, vbuf + lane * 4) = l2;
        const u32 t2 = e2[0][1] + e2[1][1] + e2[2][1] + e2[3][1];
        const u32 sc2 = wave_scan(t2);
        const u32 n3 = __builtin_amdgcn_readlane(sc2, 63);
        raw = n3 > 256u;
        if (!raw) {
          n2d = (n2 + 15u) >> 4; n3d = (n3 + 15u) >> 4;
          if (t2) {
            o = scr + A3O + sc2 - t2;
#pragma unroll
            for (int k = 0; k < 4; ++k) { put4(o, __builtin_amdgcn_perm(0u, q2[k] ^ 0x05050505u, e2[k][0])); o += e2[k][1]; }
          }
          cfence();
          const u32 q3 = *LP(const u32, scr + A3O + lane * 4) ^ 0x05050505u;
          const u32 p3 = __builtin_amdgcn_perm(0x800182FFu, 0xFFFFFFFFu, q3);
          u32 l3 = (p3 & M03) << (2 * (lane & 3));
          l3 |= __builtin_amdgcn_update_dpp(0u, l3, 0xB1, 0xf, 0xf, false);
          l3 |= __builtin_amdgcn_update_dpp(0u, l3, 0x4E, 0xf, 0xf, false);
          *LP(u32, vbuf + (n2d + (lane >> 2)) * 4) = l3;
          if (__any(p3 & 0x04040404u)) {
            const u32x2 e3 = centry(p3);
            const u32 sc3 = wave_scan(e3[1]);
            const u32 n4 = __builtin_amdgcn_readlane(sc3, 63);
            raw = n4 > 128u;
            if (!raw) {
              if (e3[1]) put4(scr + A4O + sc3 - e3[1], __builtin_amdgcn_perm(0u, q3, e3[0]));
              cfence();
              const u32x2 q4 = *LP(const u32x2, scr + A4O + (lane & 15) * 8);
              n4d = (n4 + 7u) >> 3;
              if (lane < 16) *LP(u32, vbuf + (n2d + n3d + lane) * 4) = q4[0] | (q4[1] << 4);
            }
          }
        }
      }
      cfence();
      u32 vd = n2d + n3d + n4d;
      if (raw) {
        cp2 = u32x2{(v[0] & MEX) | rotl4(v[4] & MEX), (v[1] & MEX) | rotl4(v[5] & MEX)}; vd = 128;
        *LP(u32, vbuf + lane * 4) = (v[2] & MEX) | rotl4(v[6] & MEX);
        *LP(u32, vbuf + 256 + lane * 4) = (v[3] & MEX) | rotl4(v[7] & MEX);
      }
      cfence();
      u32 voff;
      asm volatile("ds_add_rtn_u32 %0, %1, %2\n\ts_waitcnt lgkmcnt(0)" : "=&v"(voff) : "v"(lane ? dummy : (u32)L_VAL), "v"(vd) : "memory");
      const u32 va = *LP(const u32, vbuf + lane * 4), vb = *LP(const u32, vbuf + 256 + lane * 4);
      st8(dst + U_C + lw * 512 + lane * 8, cp2);
      voff = __builtin_amdgcn_readfirstlane(voff);
      if ((u32)lane < vd) st4(dst + U_V + (voff + lane) * 4, va);
      if ((u32)lane + 64u < vd) st4(dst + U_V + (voff + 64 + lane) * 4, vb);
      if (lane == 0) st4(dst + U_H + lw * 4, voff | (n2d << 10) | (n3d << 17) | (n4d << 22) | (bb << 27) | (raw ? 0x80000000u : 0u));
      waitv<0>();
      u32 nth;
      asm volatile("ds_add_rtn_u32 %0, %1, %2\n\ts_waitcnt lgkmcnt(0)" : "=&v"(nth) : "v"(lane ? dummy : (u32)L_PUB), "v"(1u) : "memory");
      if (__builtin_amdgcn_readfirstlane(nth) == 3) {
        const u32 vtot = *LP(volatile u32, L_VAL);
        if (lane < NR) st4(fdst + (size_t)(call & 1) * FLAG_PAR, (call << 6) | ((16u + vtot * 4u + 63u) >> 6));
      }
    }
    role();
    char* const edst = me + WS_E + (size_t)((q * 8 + j) * NSL + ct) * PL + lw * 1024 + l16;
    char* const eflag = me + WS_EFLAG + (size_t)(q * 8 + j) * 512 + ct * 16 + lw * 4;
    int pend = -1;
#pragma unroll 1
    for (int t = 0; t < 8; ++t) {
      if ((int)__builtin_amdgcn_readfirstlane(*LP(volatile u32, L_CVL)) <= t) {
        if (pend >= 0) { waitv<0>(); if (lane == 0) st4(eflag + pend * 64, call); pend = -1; }
        while ((int)__builtin_amdgcn_readfirstlane(*LP(volatile u32, L_CVL)) <= t) __builtin_amdgcn_s_sleep(2);
      }
      cfence();
      const u32 S = LDS_CV + t * CVS;
      const u32 info = __builtin_amdgcn_readfirstlane(*LP(const u32, S + 2048 + lw * 4));
      const u32x2 cc = *LP(const u32x2, S + lw * 512 + lane * 8);
      const u32 V = S + 2064 + (info & 0x1FFu) * 4;
      const u32 n2d = (info >> 10) & 0x7Fu, n3d = (info >> 17) & 0x1Fu, bb = (info >> 27) & 15u;
      u32 l2 = *LP(const u32, V + lane * 4);
      u32x4 ep;
      if (info >> 31) {
        ep = u32x4{cc[0], cc[1], l2, *LP(const u32, V + 256 + lane * 4)};
      } else {
        const u32 l3 = *LP(const u32, V + n2d * 4 + (lane >> 2) * 4);
        u32 c1[8], c2[4]; u32x2 e1[8], e2[4];
#pragma unroll
        for (int k = 0; k < 8; ++k) { c1[k] = (cc[k >> 2] >> (2 * (k & 3))) & M03; e1[k] = dentry(c1[k], (k & 3) != 3); }
        SCHED();
        l2 = (u32)lane < n2d ? l2 : 0u;
        u32 w4 = 0, c3 = (l3 >> (2 * (lane & 3))) & M03;
#pragma unroll
        for (int k = 0; k < 4; ++k) { c2[k] = (l2 >> (2 * k)) & M03; e2[k] = dentry(c2[k], k != 3); }
        SCHED();
        const u32 bbv = bb * B1;
        if (info & (0x1Fu << 22)) {
          c3 = (u32)lane < 4 * n3d ? c3 : 0u;
          const u32x2 e3 = dentry(c3);
          const u32 l4 = *LP(const u32, V + (n2d + n3d + (lane & 15)) * 4) ^ 0x44444444u;
          const u32 K4 = ((bb + 9u) & 15u) * B1;
          *LP(u32x2, scr + A4O + (lane & 15) * 8) = u32x2{(((l4 & M0F) + K4) & M0F) << 3, ((((l4 >> 4) & M0F) + K4) & M0F) << 3};
          cfence();
          const u32 o4 = scr + A4O + min(wave_scan(e3[1]) - e3[1], 128u);
          const u32x2 ww = pair(o4);
          w4 = __builtin_amdgcn_alignbyte(ww[1], ww[0], o4);
          c3 += e3[0];
        }
        *LP(u32, scr + A3O + lane * 4) = __builtin_amdgcn_perm(w4, ((bbv + 0x000A0B0Cu) & M0F) << 3, c3);
        const u32 ta = __builtin_popcount(cc[0] & (cc[0] >> 1) & 0x55555555u), tb = __builtin_popcount(cc[1] & (cc[1] >> 1) & 0x55555555u);
        const u32 t2 = __builtin_popcount(l2 & (l2 >> 1) & 0x55555555u);
        const u32 tt = ta | (tb << 11) | (t2 << 22);
        const u32 sc = wave_scan(tt);
        const u32 na = __builtin_amdgcn_readlane(sc, 63) & 0x7FFu;
        const u32 ex = sc - tt;
        cfence();
        u32 o2[4], o1[8]; u32x2 w[8];
        o2[0] = (ex >> 22) + (scr + A3O);
#pragma unroll
        for (int k = 1; k < 4; ++k) o2[k] = o2[k - 1] + e2[k - 1][1];
#pragma unroll
        for (int k = 0; k < 4; ++k) w[k] = pair(o2[k]);
        SCHED();
        o1[0] = (ex & 0x7FFu) + scr; o1[4] = ((ex >> 11) & 0x7FFu) + (scr + na);
#pragma unroll
        for (int k = 1; k < 8; ++k) if (k != 4) o1[k] = o1[k - 1] + e1[k - 1][1];
#pragma unroll
        for (int k = 0; k < 4; ++k) c2[k] += e2[k][0];
        SCHED();
        u32x4 y;
#pragma unroll
        for (int k = 0; k < 4; ++k) y[k] = expand(w[k], o2[k], ((bbv + 0x000D0E02u) & M0F) << 3, c2[k]);
        *LP(u32x4, scr + l16) = y;
        cfence();
#pragma unroll
        for (int k = 0; k < 8; ++k) w[k] = pair(o1[k]);
        SCHED();
#pragma unroll
        for (int k = 0; k < 8; ++k) c1[k] += e1[k][0];
        SCHED();
        const u32 T1 = ((bbv + 0x00000F01u) & M0F) << 3;
#pragma unroll
        for (int k = 0; k < 4; ++k) ep[k] = expand(w[k], o1[k], T1, c1[k]) | rotl4(expand(w[4 + k], o1[4 + k], T1, c1[4 + k]));
      }
      st16(edst + (size_t)t * 4 * PL, ep);
      if (pend >= 0) { waitv<1>(); if (lane == 0) st4(eflag + pend * 64, call); }
      pend = t;
    }
    waitv<0>();
    if (lane == 0) st4(eflag + 7 * 64, call);
  } else if (wid == 9) {
    if (b < 8) asm volatile("buffer_inv sc0 sc1" ::: "memory");
    bar();
    const u32 call = __builtin_amdgcn_readfirstlane(callv) + 1u;
    waitv<0>();
    if (b < 8 && lane < 4) st4(inv + lane * 4, call);
    while (__builtin_amdgcn_readfirstlane(*LP(volatile u32, L_PUB)) < 4) __builtin_amdgcn_s_sleep(4);
    role();
    const u64 t0 = __builtin_amdgcn_s_memrealtime();
    {
      const char* f = me + WS_FLAG + (size_t)(call & 1) * FLAG_PAR + (size_t)((src * 8 + j) * NSL + (lane & 31)) * 128;
      u32 go = 1, fv;
      while (true) {
        fv = ld4(f);
        const bool ok = (fv >> 6) >= (call & 0x3FFFFFFu);
        if ((lane & 35) == ct && ok) *LP(volatile u32, L_LEN + (lane >> 2) * 4) = max(min(fv & 63u, 33u), 1u) * 64u + 2048u;
        if (__all(ok)) break;
        if (expired(t0, tmo, call)) { go = 2; *LP(volatile u32, L_BAD) = 1; break; }
        __builtin_amdgcn_s_sleep(2);
      }
      *LP(volatile u32, L_NGO) = go;
    }
    const char* const f = lane < 32 ? me + WS_EFLAG + (size_t)(q * 8 + j) * 512 + l16
                                    : me + WS_PFLAG + (size_t)(src * 8 + j) * 128 + (lane & 7) * 16;
    u64 known = ~0ull << (32 + NPUSH);
    int nready = 0;
    while (true) {
      const int upto = 4 * min((int)__builtin_amdgcn_readfirstlane(*LP(volatile u32, L_CVL)) + 1, 8);
      bool ok = false;
      if (!((known >> lane) & 1) && (lane >= 32 || lane < upto)) {
        u32x4 v;
        asm volatile("global_load_dwordx4 %0, %1, off sc0 sc1\n\ts_waitcnt vmcnt(0)" : "=&v"(v) : "v"(f) : "memory");
        ok = min(min(v[0], v[1]), min(v[2], v[3])) >= call;
      }
      known |= __ballot(ok);
      nready = __builtin_ctzll(~(u64)((u32)known & (u32)(known >> 32)));
      if (nready == NSL) break;
      *LP(volatile u32, L_RDY) = nready;
      if (expired(t0, tmo, call)) break;
      __builtin_amdgcn_s_sleep(64);
    }
    *LP(volatile u32, L_RDY) = NSL;
    if (__builtin_amdgcn_readfirstlane(ld4(tmo)) == call) *LP(volatile u32, L_BAD) = 1;
    *LP(volatile u32, L_RDY) = NSL + 1;
  } else {
    __builtin_amdgcn_s_setprio(3);
    *LP(u32x4, LDS_MISC) = u32x4{0, 0, 0, 0};
    *LP(u32x4, LDS_MISC + 16) = u32x4{0, NPUSH, 0, 0};
    *LP(u32, L_ROT) = 0;
    *LP(u32x4, L_LEN) = u32x4{0, 0, 0, 0}; *LP(u32x4, L_LEN + 16) = u32x4{0, 0, 0, 0};
    bar();
    const u32 call = __builtin_amdgcn_readfirstlane(callv) + 1u;
    asm volatile("" :: "s"(call));
    const u32 tstart = (u32)__builtin_amdgcn_s_memrealtime();
    {
      if (b == 0 && lane < 32) st4(me + WS_ROT + lane * 128, (call << 4) | (xcc & 7u));
      const u64 t0 = __builtin_amdgcn_s_memrealtime();
      u32 r;
      while (((r = __builtin_amdgcn_readfirstlane(ld4(me + WS_ROT + (b & 31) * 128))) >> 4) < (call & 0xFFFFFFFu)) {
        if (expired(t0, tmo, call)) { *LP(volatile u32, L_BAD) = 1; r = 0; break; }
        __builtin_amdgcn_s_sleep(1);
      }
      *LP(volatile u32, L_ROT) = (r & 7u) + 1u;
      role();
    }
    {
      const char* f = lane < NE ? me + WS_NFLAG + (size_t)((src * 8 + j) * 8 + lane) * 128 : inv;
      const u64 t0 = __builtin_amdgcn_s_memrealtime();
      while (true) {
        u32x4 v;
        asm volatile("global_load_dwordx4 %0, %1, off sc0 sc1\n\ts_waitcnt vmcnt(0)" : "=&v"(v) : "v"(f) : "memory");
        const u64 ok = __ballot(min(min(v[0], v[1]), min(v[2], v[3])) >= call);
        if (ok == ~0ull) break;
        if ((~ok & ((1u << NE) - 1)) == 0 && __builtin_amdgcn_s_memrealtime() - t0 > 500) {
          asm volatile("buffer_inv sc0 sc1\n\ts_waitcnt vmcnt(0)" ::: "memory");
          break;
        }
        if (expired(t0, tmo, call)) {
          *LP(volatile u32, L_BAD) = 1; *LP(volatile u32, L_CVL) = 8; *LP(volatile u32, L_NL) = NSL + KT;
          return;
        }
        __builtin_amdgcn_s_sleep(2);
      }
      waitv<0>();
    }
    u32 lim = 0;
    bool dead = false;
    const char* const base = a.ws[(src - rank) & (NR - 1)] + (size_t)(call & 1) * WS_SLOT + (size_t)(j * NSL) * USTRIDE;
    const u32 tau = q ? TAU : TAU / 4;
    u32 due = (tstart + (u32)max((int)((u32)__builtin_amdgcn_s_memrealtime() - tstart) - CMAX, TREF)) << 4;
    auto pace = [&](u32 bytes) {
      while ((int)(((u32)__builtin_amdgcn_s_memrealtime() << 4) - due) < 0) {}
      due += bytes * tau >> 10;
    };
    int ct_ = 0, cp_ = 0;
    auto issue_c = [&]() {
      const char* sb = base + (size_t)(4 * ct_ + ct) * USTRIDE + U_C;
      const u32 la = LDS_CV + ct_ * CVS;
      if (cp_ == 0) {
        while (!(lim = __builtin_amdgcn_readfirstlane(*LP(volatile u32, L_LEN + ct_ * 4))))
          if (__builtin_amdgcn_readfirstlane(*LP(volatile u32, L_NGO)) == 2) { dead = true; break; }
        if (ct_ == 0) { const u32 now = ((u32)__builtin_amdgcn_s_memrealtime() - CMAX) << 4; if ((int)(now - due) > 0) due = now; }
      }
      if (dead) return;
      pace(cp_ == 2 ? 4u * (lim - 2048u) : 4096u);
      if (cp_ == 2 && lim > 3072u) {
        set_m0(la + 3072); if (l16 + 3072u < lim) glds<0, true>(sb, l16 + 3072u);
        set_m0(la + 4096); if (l16 + 4096u < lim) glds<0, true>(sb, l16 + 4096u);
      }
      set_m0(la + cp_ * 1024);
      if (l16 + cp_ * 1024u < lim) glds<0, true>(sb, l16 + cp_ * 1024u);
      if (++cp_ == 3) { cp_ = 0; ++ct_; }
    };
    int arrived = 0;
    auto issue_n = [&](int s, int p) {
      const char* sb = base + (size_t)s * USTRIDE;
      pace(1024u);
      if (p == 0) {
        while (s >= 8 && arrived < 4 * (s - 5)) arrived = __builtin_amdgcn_readfirstlane(*LP(volatile u32, L_ARR));
        set_m0(LDS_N + (s & 7) * PL); glds<0, false>(sb, l16);
      }
      if (p == 1) glds<1024, false>(sb, l16);
      if (p == 2) glds<2048, false>(sb, l16);
      if (p == 3) glds<3072, false>(sb, l16);
    };
    auto post = [&](u32 addr, u32 v) { *LP(volatile u32, addr) = v; };
    constexpr int NB = 24 - 3 * PRE;
    static_assert(PRE >= 2 && PRE <= 7 && W2 >= 4 && W2 <= 8 && N0 + NB + 4 <= NSL, "");
    int cdone = 0;
    auto landed_c = [&]() { ++cdone; if (cdone % 3 == 0) post(L_CVL, cdone / 3); };
#pragma unroll 1
    for (int s = NPUSH; s < N0; ++s) {
#pragma unroll
      for (int p = 0; p < 4; ++p) { waitv<W2 - 1>(); issue_n(s, p); }
    }
    waitv<3>(); issue_c();
    waitv<1>(); post(L_NL, N0); issue_c();
#pragma unroll 1
    for (int n = 0; n < 3 * PRE - 2; ++n) { waitv<1>(); issue_c(); landed_c(); }
    waitv<1>(); landed_c();
    u32 go;
    do go = __builtin_amdgcn_readfirstlane(*LP(volatile u32, L_NGO)); while (!go);
    if (go == 2) { waitv<0>(); post(L_CVL, 8); post(L_NL, NSL + KT); return; }
#pragma unroll
    for (int p = 0; p < 4; ++p) issue_n(N0, p);
    waitv<4>(); issue_c(); landed_c();
#pragma unroll 1
    for (int s = N0; s < N0 + NB; ++s) {
#pragma unroll
      for (int p = 0; p < 4; ++p) { waitv<4>(); if (p == 3) post(L_NL, s + 1); issue_n(s + 1, p); }
      waitv<4>();
      if (s + 1 < N0 + NB) issue_c();
      landed_c();
    }
#pragma unroll
    for (int n = 4; n < W2; ++n) issue_n(N0 + NB + n / 4, n % 4);
    constexpr int S1 = NSL - KT - (W2 + 3) / 4;
    static_assert(S1 >= N0 + NB, "");
#pragma unroll 1
    for (int s = N0 + NB; s < S1; ++s) {
#pragma unroll
      for (int p = 0; p < 4; ++p) { waitv<W2 - 1>(); if (p == 3) post(L_NL, s + 1); issue_n(s + (p + W2) / 4, (p + W2) % 4); }
    }
    constexpr int NA = 4 * (NSL - KT - S1), NZ = NA + 4 * KT;
#pragma unroll
    for (int n = 0; n < NZ; ++n) {
      waitn(NZ - 1 - n < W2 - 1 ? NZ - 1 - n : W2 - 1);
      if (n < NA ? n % 4 == 3 : n % 2 == 1) post(L_NL, n < NA ? S1 + n / 4 + 1 : NSL - KT + (n - NA) / 2 + 1);
      const int m = n + W2;
      if (m < NA) issue_n(S1 + m / 4, m % 4);
      else if (m < NZ) {
        const int hh = (m - NA) / 2, s = NSL - KT + hh % KT, p = (hh / KT) * 2 + (m - NA) % 2;
        if (p == 2) set_m0(LDS_N + (s & 7) * PL);
        issue_n(s, p);
      }
    }
  }
}

extern "C" void kernel_launch(int rank, int nranks, ncclComm_t comm,
                              hipStream_t stream,
                              void* const* d_in, const int* in_sizes, int n_in,
                              void* d_out, int out_size,
                              const int* in_dtypes, int out_dtype,
                              void* const* d_out_all,
                              void* const* d_ws_all, size_t ws_bytes) {
  if (nranks != NR || (unsigned)rank >= (unsigned)NR || ws_bytes < WS_END || n_in < 4) return;
  static bool init = false;
  if (!init) {
    HIP_CHECK(hipFuncSetAttribute((const void*)ag_gemm, hipFuncAttributeMaxDynamicSharedMemorySize, LDS_TOT));
    init = true;
  }
  Args a;
  for (int p = 0; p < NR; ++p) a.ws[p] = (char*)d_ws_all[(rank + p) & (NR - 1)];
  a.x = (const unsigned char*)d_in[0]; a.w = (const unsigned char*)d_in[1];
  a.sx = (const float*)d_in[2]; a.sw = (const float*)d_in[3];
  a.out = (float*)d_out; a.rank = rank;
  {
    const unsigned cl = CLIQUE;
    unsigned o = 0; int n = 0;
    for (int pass = 0; pass < 2; ++pass)
      for (int i = 0; i < NR; ++i) { const int p = (rank + i) & 7; if ((int)(((cl >> p) ^ (cl >> rank)) & 1) == pass) o |= (unsigned)p << (4 * n++); }
    a.order = o;
  }
  hipLaunchKernelGGL(ag_gemm, dim3(NWG), dim3(NTHR), LDS_TOT, stream, a);
  HIP_CHECK(hipGetLastError());
}
```

```cpp
#include <hip/hip_runtime.h>
#include <rccl/rccl.h>
#include <cstdio>
#include <cstdint>

#pragma clang diagnostic ignored "-Wint-to-pointer-cast"

#define HIP_CHECK(expr) do { hipError_t _e = (expr); if (_e != hipSuccess) { \
    fprintf(stderr, "HIP error %s at %s:%d\n", hipGetErrorString(_e), __FILE__, __LINE__); \
    return; } } while (0)

typedef unsigned u32;
typedef unsigned u32x2 __attribute__((ext_vector_type(2)));
typedef unsigned u32x4 __attribute__((ext_vector_type(4)));
typedef int v2i __attribute__((ext_vector_type(2)));
typedef int v8i __attribute__((ext_vector_type(8)));
typedef float v16f __attribute__((ext_vector_type(16)));
typedef unsigned long long u64;
#define LAS __attribute__((address_space(3)))
#define LP(T, a) ((LAS T*)(u32)(a))

constexpr int NPUSH = 3;
constexpr int PACE = 60;
constexpr int NE = 3;
constexpr int PRE = 5;
constexpr int W2 = 6;
constexpr int KT = 3;
constexpr int TAU = 219;
constexpr int TREF = 185;
constexpr int CMAX = 90;
constexpr unsigned CLIQUE = 0x4D;

constexpr int NR = 8;
constexpr int KTOT = 4096, NL = 512, MS = 512;
constexpr int BM = 64, BN = 128, KS = 128, NSL = KTOT / KS;
constexpr int NWG = 256, NTHR = 640;
constexpr int PL = 4096;
constexpr int U_C = 4096, U_H = 6144, U_V = 6160, USTRIDE = 9216;
constexpr int N0 = NPUSH + NE;
static_assert(NPUSH >= 1 && NPUSH <= 3 && NE >= 1 && N0 <= 8, "");
constexpr size_t WS_SLOT  = (size_t)NWG * USTRIDE;
constexpr size_t WS_PUSH  = 2 * WS_SLOT;
constexpr size_t PUSH_PAR = (size_t)NR * 8 * 8 * PL;
constexpr size_t WS_E     = WS_PUSH + 2 * PUSH_PAR;
constexpr size_t WS_FLAG  = WS_E + (size_t)64 * NSL * PL;
constexpr size_t FLAG_PAR = (size_t)NR * 8 * NSL * 128;
constexpr size_t WS_PFLAG = WS_FLAG + 2 * FLAG_PAR;
constexpr size_t WS_NFLAG = WS_PFLAG + NR * 8 * 128;
constexpr size_t WS_EFLAG = WS_NFLAG + NR * 8 * 8 * 128;
constexpr size_t WS_CNT   = WS_EFLAG + 64 * 512;
constexpr size_t WS_INV   = WS_CNT + NWG * 128;
constexpr size_t WS_ROT   = WS_INV + 16 * 128;
constexpr size_t WS_TMO   = WS_ROT + 32 * 128;
constexpr size_t WS_END   = WS_TMO + 128;
constexpr u64 DEADLINE = 100000000ull;
constexpr u64 SLOW = 20000ull;
constexpr int WSL = KS * BN;
constexpr int LDS_LUT = 0;
constexpr int LDS_CLUT = 128;
constexpr int LDS_MISC = 256;
constexpr int L_BAD = LDS_MISC, L_VAL = LDS_MISC + 4, L_NGO = LDS_MISC + 8, L_PUB = LDS_MISC + 12, L_LEN = LDS_MISC + 32, L_ROT = LDS_MISC + 192, L_MC = LDS_MISC + 196, L_EC = LDS_MISC + 208;
constexpr int L_ARR = LDS_MISC + 16, L_NL = LDS_MISC + 20, L_RDY = LDS_MISC + 24, L_CVL = LDS_MISC + 28, L_DUMMY = LDS_MISC + 64;
constexpr int SCR = 2048;
constexpr int A3O = 1040, A4O = 1312, VBO = 1536;
constexpr int LDS_SCR = 512;
constexpr int LDS_E = 9 * 1024;
constexpr int LDS_N = LDS_E + 4 * PL;
constexpr int LDS_W = LDS_N + 8 * PL;
constexpr int CVS = 4352;
constexpr int LDS_CV = LDS_W + 4 * WSL;
constexpr int LDS_TOT = LDS_CV + 8 * CVS;
static_assert(LDS_SCR + 4 * SCR <= LDS_E && LDS_TOT <= 160 * 1024, "lds");

struct Args {
  const unsigned char* x;
  char* ws[NR];
  const unsigned char* w;
  const float* sx; const float* sw; float* out;
  int rank; u32 order;
};

#define SCHED() __builtin_amdgcn_sched_barrier(0)
template <int N> __device__ __forceinline__ void waitv() { asm volatile("s_waitcnt vmcnt(%0)" :: "n"(N) : "memory"); }
__device__ __forceinline__ void waitn(int n) {
  switch (n) {
    case 0: waitv<0>(); break; case 1: waitv<1>(); break; case 2: waitv<2>(); break; case 3: waitv<3>(); break;
    case 4: waitv<4>(); break; case 5: waitv<5>(); break; case 6: waitv<6>(); break; default: waitv<7>();
  }
}
__device__ __forceinline__ void cfence() { asm volatile("" ::: "memory"); }
__device__ __forceinline__ void bar() {
  cfence(); SCHED();
  __builtin_amdgcn_s_waitcnt(0xC07F);
  __builtin_amdgcn_s_barrier();
  SCHED(); cfence();
}
__device__ __forceinline__ void set_m0(u32 lds_addr) { asm volatile("s_mov_b32 m0, %0\n\ts_nop 0" :: "s"(lds_addr) : "memory"); }
template <int OFF, bool SYS> __device__ __forceinline__ void glds(const void* sbase, u32 voff) {
  if (SYS) asm volatile("global_load_lds_dwordx4 %0, %1 offset:%2 sc0 sc1" :: "v"(voff), "s"(sbase), "n"(OFF) : "memory");
  else asm volatile("global_load_lds_dwordx4 %0, %1 offset:%2" :: "v"(voff), "s"(sbase), "n"(OFF) : "memory");
}
__device__ __forceinline__ void st16(void* p, u32x4 v) { asm volatile("global_store_dwordx4 %0, %1, off sc0 sc1\n\ts_nop 1" :: "v"(p), "v"(v) : "memory"); }
__device__ __forceinline__ void st8(void* p, u32x2 v) { asm volatile("global_store_dwordx2 %0, %1, off sc0 sc1\n\ts_nop 1" :: "v"(p), "v"(v) : "memory"); }
__device__ __forceinline__ void st4(void* p, u32 v) { asm volatile("global_store_dword %0, %1, off sc0 sc1\n\ts_nop 1" :: "v"(p), "v"(v) : "memory"); }
__device__ __forceinline__ u32 ld4(const void* p) {
  u32 v;
  asm volatile("global_load_dword %0, %1, off sc0 sc1\n\ts_waitcnt vmcnt(0)" : "=&v"(v) : "v"(p) : "memory");
  return v;
}

__device__ __forceinline__ bool last_of_four(u32 addr) {
  u32 nth;
  asm volatile("ds_add_rtn_u32 %0, %1, %2\n\ts_waitcnt lgkmcnt(0)" : "=&v"(nth) : "v"(addr), "v"(1u) : "memory");
  return __builtin_amdgcn_readfirstlane(nth) == 3;
}
__device__ __forceinline__ bool expired(u64 t0, u32* tmo, u32 call) {
  const u64 dt = __builtin_amdgcn_s_memrealtime() - t0;
  if (dt <= SLOW) return false;
  if (dt > DEADLINE) { st4(tmo, call); waitv<0>(); return true; }
  __builtin_amdgcn_s_sleep(64);
  return __builtin_amdgcn_readfirstlane(ld4(tmo)) == call;
}

__device__ __forceinline__ u32 wave_scan(u32 x) {
  x += __builtin_amdgcn_update_dpp(0u, x, 0x111, 0xf, 0xf, true);
  x += __builtin_amdgcn_update_dpp(0u, x, 0x112, 0xf, 0xf, true);
  x += __builtin_amdgcn_update_dpp(0u, x, 0x114, 0xf, 0xf, true);
  x += __builtin_amdgcn_update_dpp(0u, x, 0x118, 0xf, 0xf, true);
  x += __builtin_amdgcn_update_dpp(0u, x, 0x142, 0xa, 0xf, false);
  x += __builtin_amdgcn_update_dpp(0u, x, 0x143, 0xc, 0xf, false);
  return x;
}
constexpr u32 B1 = 0x01010101u, M0F = 0x0F0F0F0Fu, MSM = 0x87878787u, MEX = 0x78787878u, M03 = 0x03030303u;
__device__ __forceinline__ u32 rotl4(u32 x) { return __builtin_amdgcn_alignbit(x, x, 28); }
__device__ __forceinline__ u32 rotr4(u32 x) { return __builtin_amdgcn_alignbit(x, x, 4); }
__device__ __forceinline__ u32x2 centry(u32 p) {
  return *LP(const u32x2, __builtin_amdgcn_udot4(p & 0x04040404u, 0x10080402u, (u32)LDS_CLUT, false));
}
__device__ __forceinline__ u32x2 dentry(u32 c, bool with_count = true) {
  const u32 p = __builtin_amdgcn_udot4(__builtin_amdgcn_perm(0u, 0x01000000u, c), 0x40201008u, (u32)LDS_LUT, false);
  if (with_count) return *LP(const u32x2, p);
  return u32x2{*LP(const u32, p), 0u};
}
__device__ __forceinline__ void put4(u32 o, u32 y) {
  const u64 w = (u64)y << ((o & 3u) * 8u);
  LAS u32* p = LP(u32, o & ~3u);
  __hip_atomic_fetch_or(p, (u32)w, __ATOMIC_RELAXED, __HIP_MEMORY_SCOPE_WAVEFRONT);
  __hip_atomic_fetch_or(p + 1, (u32)(w >> 32), __ATOMIC_RELAXED, __HIP_MEMORY_SCOPE_WAVEFRONT);
}
__device__ __forceinline__ u32x2 pair(u32 o) {
  const LAS u32* p = LP(const u32, o & ~3u);
  return u32x2{p[0], p[1]};
}
__device__ __forceinline__ u32 expand(u32x2 w, u32 o, u32 table, u32 sel) {
  return __builtin_amdgcn_perm(__builtin_amdgcn_alignbyte(w[1], w[0], o), table, sel);
}

__global__ void __launch_bounds__(NTHR, 3) ag_gemm(Args a) {
  const int tid = threadIdx.x, lane = tid & 63;
  const int wid = __builtin_amdgcn_readfirstlane(tid >> 6);
  const int b = blockIdx.x;
  char* const me = a.ws[0];
  const int jp = b & 7, idx = b >> 3, ct = idx & 3, pe = idx >> 3;
  const int ps = pe == 3 ? NPUSH + (idx & 7) : pe;
  const int brow = (wid & 3) * 16 + (lane >> 2);
  const unsigned char* const xlane = a.x + (size_t)(jp * BM) * KTOT + (wid < 4 ? ps : idx) * KS + (u32)(brow * KTOT + (((lane & 3) ^ ((brow >> 2) & 3)) << 5));
  const u32x4 v0 = *(const u32x4*)xlane, v1 = *(const u32x4*)(xlane + 16);
  u32* const cnt = (u32*)(me + WS_CNT + (size_t)b * 128);
  const u32 callv = __hip_atomic_load(cnt, __ATOMIC_RELAXED, __HIP_MEMORY_SCOPE_AGENT);
  cfence(); SCHED();
  const int rank = a.rank;
  int j, q, src;
  auto role = [&]() {
    u32 r;
    while (!(r = __builtin_amdgcn_readfirstlane(*LP(volatile u32, L_ROT)))) __builtin_amdgcn_s_sleep(1);
    const int x = (b + r - 1) & 7;
    j = (x & 3) * 2 + ((idx >> 2) & 1); q = (x & 4) | (idx >> 3);
    src = (a.order >> (4 * q)) & (NR - 1);
  };
  const u32 xcc = __builtin_amdgcn_s_getreg((3 << 11) | 20) & 15u;
  u32* const tmo = (u32*)(me + WS_TMO);
  const u32 dummy = L_DUMMY + (lane & 31) * 4;
  const u32 l16 = lane * 16u;
  char* const inv = me + WS_INV + (size_t)xcc * 128;

  if (wid < 4) {
    const bool early = pe == 3 && (idx & 7) < NE;
    auto nplane = [&](const u32x4 v0, const u32x4 v1) {
      u32x4 np;
#pragma unroll
      for (int k = 0; k < 4; ++k) np[k] = (v0[k] & MSM) | rotl4(v1[k] & MSM);
      return np;
    };
    char* nflag = a.ws[lane & (NR - 1)] + WS_NFLAG + (size_t)((rank * 8 + jp) * 8 + (idx & 7)) * 128;
    if (tid < 16) {
      u32 add = 0, cs = 0x0C0C0C0Cu, n = 0;
#pragma unroll
      for (int i = 0; i < 4; ++i)
        if ((tid >> i) & 1) { add |= (1 + n) << (8 * i); cs = (cs & ~(0xFFu << (8 * n))) | (i << (8 * n)); ++n; }
      *LP(u32x2, LDS_LUT + tid * 8) = u32x2{add, n};
      *LP(u32x2, LDS_CLUT + tid * 8) = u32x2{cs, n};
    }
    const int kr = lane >> 3, cp = lane & 7;
    u32 wvo[4];
#pragma unroll
    for (int i = 0; i < 4; ++i) wvo[i] = (kr + i * 8) * NL + ((cp >> 1) ^ ((kr >> 1) & 3)) * 32 + (cp & 1) * 16 - i * 1024;
    const unsigned char* const wsrc = a.w + (size_t)(wid * 32) * NL + ct * BN;
    auto stage_w = [&](int s) {
      set_m0(LDS_W + (s & 3) * WSL + wid * 4096);
      const unsigned char* sb = wsrc + (size_t)s * KS * NL;
      glds<0, false>(sb, wvo[0]); glds<1024, false>(sb, wvo[1]); glds<2048, false>(sb, wvo[2]); glds<3072, false>(sb, wvo[3]);
    };
    char* const pdst = a.ws[idx & (NR - 1)];
    char* pp = (pe == 3 ? me + (size_t)(jp * NSL + ps) * USTRIDE : pdst + WS_PUSH + (size_t)((rank * 8 + jp) * 8 + pe) * PL) + wid * 1024 + l16;
    const size_t podd = pe == 3 ? WS_SLOT : PUSH_PAR;
    asm volatile("" : "+v"(pp));
    bar();
    const u32x4 np0 = nplane(v0, v1);
    const u32 call = __builtin_amdgcn_readfirstlane(callv) + 1u;
    const size_t par = call & 1;
    if (pe == 1 || pe == 2) {
      asm volatile("" :: "v"(np0));
      const u64 tpush = __builtin_amdgcn_s_memrealtime();
      while (__builtin_amdgcn_s_memrealtime() - tpush < (u64)(pe * PACE)) __builtin_amdgcn_s_sleep(1);
    }
    if (pe < NPUSH || early) st16(pp + par * podd, np0);
    auto last = [&]() { waitv<0>(); return last_of_four(lane ? dummy : (u32)L_MC); };
    if (early) {
      asm volatile("" : "+v"(nflag));
      if (last() && lane < NR) st4(nflag, call);
    }
    float sc = a.sx[0] * a.sw[0];
    while (__builtin_amdgcn_readfirstlane(*LP(volatile u32, L_PUB)) < 4) __builtin_amdgcn_s_sleep(4);
    __builtin_amdgcn_s_sleep(32);
    if (pe < NPUSH && last() && lane == 0) st4(pdst + WS_PFLAG + (size_t)(rank * 8 + jp) * 128 + pe * 16, call);
    asm volatile("" : "+v"(sc));
    role();
    const char* const esrc = me + WS_E + (size_t)((q * 8 + j) * NSL) * PL + wid * 1024;
    const char* const psrc = me + WS_PUSH + par * PUSH_PAR + (size_t)((src * 8 + j) * 8) * PL + wid * 1024;
    auto stage_e = [&](int s, bool push) {
      if (push) { set_m0(LDS_N + (s & 7) * PL + wid * 1024); glds<0, true>(psrc + s * PL, l16); }
      set_m0(LDS_E + (s & 3) * PL + wid * 1024); glds<0, true>(esrc + s * PL, l16);
    };
    v16f acc[2] = {};
    const int r31 = lane & 31, h = lane >> 5, wm = wid >> 1, wn = wid & 1;
    const int q8 = (lane & 15) >> 1, p8 = lane & 1, g1 = (lane >> 4) & 1;
    u32 boff[2], aoff[2];
#pragma unroll
    for (int i = 0; i < 2; ++i) {
      boff[i] = LDS_W + (32 * h + q8) * 128 + (((wn * 2 + i) ^ ((q8 >> 1) & 3)) << 5) + g1 * 16 + p8 * 8;
      aoff[i] = (wm * 32 + r31) * 64 + (((i * 2 + h) ^ ((r31 >> 2) & 3)) << 4);
    }
    struct Frag { u32x4 ep[2]; v8i bf[2][2]; };
    auto ldfrag = [&](Frag& f, int kk, int s) {
      f.ep[kk] = *LP(const u32x4, aoff[kk] + (s & 3) * PL + LDS_E);
#pragma unroll
      for (int n = 0; n < 2; ++n) {
        v2i t[4];
#pragma unroll
        for (int r = 0; r < 4; ++r) t[r] = __builtin_amdgcn_ds_read_tr8_b64_v2i32(LP(v2i, boff[n] + (s & 3) * WSL) + (kk * 64 + r * 8) * 16);
        f.bf[kk][n] = v8i{t[0][0], t[0][1], t[1][0], t[1][1], t[2][0], t[2][1], t[3][0], t[3][1]};
      }
    };
    auto afrag = [&](const u32x4 np, const u32x4 ep) {
      v8i af;
#pragma unroll
      for (int k = 0; k < 4; ++k) {
        af[k] = (np[k] & MSM) | (ep[k] & MEX);
        af[4 + k] = rotr4((np[k] & MEX) | (ep[k] & MSM));
      }
      return af;
    };
#define MFMA(n, af, bf) acc[n] = __builtin_amdgcn_mfma_scale_f32_32x32x64_f8f6f4(af, bf, acc[n], 0, 0, 0, 0, 0, 0)
    auto step = [&](int s, const Frag& cur, Frag& nxt, bool more, bool stg) {
      const u32x4 np0 = *LP(const u32x4, aoff[0] + (s & 7) * PL + LDS_N);
      const u32x4 np1 = *LP(const u32x4, aoff[1] + (s & 7) * PL + LDS_N);
      SCHED();
      if (stg) { stage_e(s + 4, false); stage_w(s + 4); }
      SCHED();
      const v8i a0 = afrag(np0, cur.ep[0]);
      MFMA(0, a0, cur.bf[0][0]);
      SCHED();
      const v8i a1 = afrag(np1, cur.ep[1]);
      SCHED();
      if (!more) {
        MFMA(0, a1, cur.bf[1][0]); MFMA(1, a0, cur.bf[0][1]); MFMA(1, a1, cur.bf[1][1]);
        return;
      }
      MFMA(1, a0, cur.bf[0][1]);
      SCHED();
      ldfrag(nxt, 0, s + 1);
      SCHED();
      MFMA(0, a1, cur.bf[1][0]);
      SCHED();
      ldfrag(nxt, 1, s + 1);
      SCHED();
      MFMA(1, a1, cur.bf[1][1]);
      SCHED();
      asm volatile("" :: "v"(a0), "v"(cur.bf[0][0]), "v"(cur.bf[0][1]), "v"(cur.bf[1][0]));
    };
    const u32 arr = lane ? dummy : (u32)L_ARR;
    auto count = [&](u32 addr) { asm volatile("ds_add_u32 %0, %1" :: "v"(addr), "v"(1u) : "memory"); };
    auto spin = [&](int na, int nl, int nr, int nc = 0) {
      while (true) {
        u32x4 w;
        asm volatile("ds_read_b128 %0, %1\n\ts_waitcnt lgkmcnt(0)" : "=&v"(w) : "v"((u32)L_ARR) : "memory");
        if (__all((int)w[0] >= na && (int)w[1] >= nl && (int)w[2] >= nr && (int)w[3] >= nc)) break;
      }
    };
    auto meet = [&](int s) {
      SCHED();
      asm volatile("s_waitcnt lgkmcnt(0)\n\tds_add_u32 %0, %1" :: "v"(arr), "v"(1u) : "memory");
      spin(4 * s + 8, s + 1, min(s + 5, NSL));
      __builtin_amdgcn_s_waitcnt(0xC07F);
      SCHED();
    };
    {
      Frag fa, fb;
#pragma unroll 1
      for (int s = 0; s < 4; ++s) { stage_w(s); __builtin_amdgcn_s_sleep(8); }
      spin(0, 0, 4);
#pragma unroll 1
      for (int s = 0; s < 4; ++s) stage_e(s, s < NPUSH);
      waitv<0>(); meet(-1);
      ldfrag(fa, 0, 0); ldfrag(fa, 1, 0);
      meet(0); step(0, fa, fb, true, true);
#pragma unroll 1
      for (int s = 1; s < NSL - 5; s += 2) {
        waitv<10>(); meet(s);
        step(s, fb, fa, true, true);
        waitv<10>(); meet(s + 1);
        step(s + 1, fa, fb, true, true);
      }
      waitv<10>(); meet(NSL - 5); step(NSL - 5, fb, fa, true, true);
      static_assert(KT == 3, "");
      waitv<10>(); meet(NSL - 4); step(NSL - 4, fa, fb, true, false);
      const int hb = wm * KT;
      const u32 arr2 = lane ? dummy : (u32)L_CVL;
      waitv<5>(); count(arr);
      if (wm) { waitv<0>(); count(arr2); }
      SCHED(); spin(4 * (NSL - 3) + 8, NSL - 2 + hb, 0, 0); SCHED(); step(NSL - 3, fb, fa, true, false);
      if (!wm) { waitv<0>(); count(arr2); }
      SCHED(); spin(0, NSL - 1 + hb, 0, 12); SCHED(); step(NSL - 2, fa, fb, true, false);
      spin(0, 0, NSL + 1, 0);
      if (*LP(volatile u32, L_BAD)) sc = __builtin_nanf("");
      asm volatile("" : "+v"(sc));
      SCHED(); spin(0, NSL + hb, 0, 0); SCHED(); step(NSL - 1, fb, fa, false, false);
    }
    if (tid == 0) st4(cnt, call);
    float* o = a.out + (size_t)(src * MS + j * BM + wm * 32 + 4 * h) * NL + ct * BN + wn * 64 + r31;
#pragma unroll
    for (int n = 0; n < 2; ++n)
#pragma unroll
      for (int i = 0; i < 16; ++i)
        __hip_atomic_store(o + (size_t)((i & 3) + 8 * (i >> 2)) * NL + n * 32, acc[n][i] * sc, __ATOMIC_RELAXED, __HIP_MEMORY_SCOPE_SYSTEM);
  } else if (wid < 8) {
    const int lw = wid - 4;
    const u32 scr = LDS_SCR + lw * SCR;
    u32 call;
    {
      char* fdst = a.ws[lane & (NR - 1)] + WS_FLAG + (size_t)((rank * 8 + jp) * NSL + idx) * 128;
      *LP(u32x4, scr + l16) = u32x4{0, 0, 0, 0};
      *LP(u32x2, scr + 1024 + lane * 8) = u32x2{0, 0};
      bar();
      call = __builtin_amdgcn_readfirstlane(callv) + 1u;
      asm volatile("" : "+v"(fdst));
      char* const dst = me + (size_t)(call & 1) * WS_SLOT + (size_t)(jp * NSL + idx) * USTRIDE;
      const u32 vbuf = scr + VBO;
      const u32 v[8] = {v0[0], v0[1], v0[2], v0[3], v1[0], v1[1], v1[2], v1[3]};
      u32 z[8], sum = 0;
#pragma unroll
      for (int k = 0; k < 8; ++k) { z[k] = v[k] & MEX; if (k & 1) sum = __builtin_amdgcn_udot4(z[k], B1, sum, false); }
      sum = __builtin_amdgcn_readlane(wave_scan(sum), 63);
      const u32 bb = ((sum + 7946u) >> 13) & 15u;
      const u32 kz = ((7u - bb) & 15u) * 0x08080808u;
      u32x4 np;
#pragma unroll
      for (int k = 0; k < 4; ++k) np[k] = (v[k] & MSM) | rotl4(v[4 + k] & MSM);
      if (idx >= N0) st16(dst + lw * 1024 + l16, np);
      u32x2 e1[8], cp2 = {0, 0};
#pragma unroll
      for (int k = 0; k < 8; ++k) {
        z[k] = (((z[k] + kz) >> 3) & M0F) ^ 0x04040404u;
        const u32 p = __builtin_amdgcn_perm(0xFFFFFFFFu, 0x8201FFFFu, z[k]);
        cp2[k >> 2] |= (p & M03) << (2 * (k & 3));
        e1[k] = centry(p);
      }
      const u32 t1 = (e1[0][1] + e1[1][1] + e1[2][1] + e1[3][1]) | ((e1[4][1] + e1[5][1] + e1[6][1] + e1[7][1]) << 16);
      const u32 sc1 = wave_scan(t1);
      const u32 tot1 = __builtin_amdgcn_readlane(sc1, 63);
      const u32 n2 = (tot1 & 0xFFFFu) + (tot1 >> 16);
      bool raw = n2 > 1024u;
      u32 n2d = 0, n3d = 0, n4d = 0;
      if (!raw) {
        const u32 ex = sc1 - t1 + (scr | ((scr + (tot1 & 0xFFFFu)) << 16));
        u32 o = ex & 0xFFFFu;
#pragma unroll
        for (int k = 0; k < 8; ++k) {
          if (k == 4) o = ex >> 16;
          put4(o, __builtin_amdgcn_perm(0u, z[k], e1[k][0])); o += e1[k][1];
        }
        cfence();
        const u32x4 q2 = *LP(const u32x4, scr + l16);
        u32 l2 = 0; u32x2 e2[4];
#pragma unroll
        for (int k = 0; k < 4; ++k) {
          const u32 p = __builtin_amdgcn_perm(0xFFFFFFFFu, 0xFFFF8201u, q2[k] ^ B1);
          l2 |= (p & M03) << (2 * k); e2[k] = centry(p);
        }
        *LP(u32, vbuf + lane * 4) = l2;
        const u32 t2 = e2[0][1] + e2[1][1] + e2[2][1] + e2[3][1];
        const u32 sc2 = wave_scan(t2);
        const u32 n3 = __builtin_amdgcn_readlane(sc2, 63);
        raw = n3 > 256u;
        if (!raw) {
          n2d = (n2 + 15u) >> 4; n3d = (n3 + 15u) >> 4;
          if (t2) {
            o = scr + A3O + sc2 - t2;
#pragma unroll
            for (int k = 0; k < 4; ++k) { put4(o, __builtin_amdgcn_perm(0u, q2[k] ^ 0x05050505u, e2[k][0])); o += e2[k][1]; }
          }
          cfence();
          const u32 q3 = *LP(const u32, scr + A3O + lane * 4) ^ 0x05050505u;
          const u32 p3 = __builtin_amdgcn_perm(0x800182FFu, 0xFFFFFFFFu, q3);
          u32 l3 = (p3 & M03) << (2 * (lane & 3));
          l3 |= __builtin_amdgcn_update_dpp(0u, l3, 0xB1, 0xf, 0xf, false);
          l3 |= __builtin_amdgcn_update_dpp(0u, l3, 0x4E, 0xf, 0xf, false);
          *LP(u32, vbuf + (n2d + (lane >> 2)) * 4) = l3;
          if (__any(p3 & 0x04040404u)) {
            const u32x2 e3 = centry(p3);
            const u32 sc3 = wave_scan(e3[1]);
            const u32 n4 = __builtin_amdgcn_readlane(sc3, 63);
            raw = n4 > 128u;
            if (!raw) {
              if (e3[1]) put4(scr + A4O + sc3 - e3[1], __builtin_amdgcn_perm(0u, q3, e3[0]));
              cfence();
              const u32x2 q4 = *LP(const u32x2, scr + A4O + (lane & 15) * 8);
              n4d = (n4 + 7u) >> 3;
              if (lane < 16) *LP(u32, vbuf + (n2d + n3d + lane) * 4) = q4[0] | (q4[1] << 4);
            }
          }
        }
      }
      cfence();
      u32 vd = n2d + n3d + n4d;
      if (raw) {
        cp2 = u32x2{(v[0] & MEX) | rotl4(v[4] & MEX), (v[1] & MEX) | rotl4(v[5] & MEX)}; vd = 128;
        *LP(u32, vbuf + lane * 4) = (v[2] & MEX) | rotl4(v[6] & MEX);
        *LP(u32, vbuf + 256 + lane * 4) = (v[3] & MEX) | rotl4(v[7] & MEX);
      }
      cfence();
      u32 voff;
      asm volatile("ds_add_rtn_u32 %0, %1, %2\n\ts_waitcnt lgkmcnt(0)" : "=&v"(voff) : "v"(lane ? dummy : (u32)L_VAL), "v"(vd) : "memory");
      const u32 va = *LP(const u32, vbuf + lane * 4), vb = *LP(const u32, vbuf + 256 + lane * 4);
      st8(dst + U_C + lw * 512 + lane * 8, cp2);
      voff = __builtin_amdgcn_readfirstlane(voff);
      if ((u32)lane < vd) st4(dst + U_V + (voff + lane) * 4, va);
      if ((u32)lane + 64u < vd) st4(dst + U_V + (voff + 64 + lane) * 4, vb);
      if (lane == 0) st4(dst + U_H + lw * 4, voff | (n2d << 10) | (n3d << 17) | (n4d << 22) | (bb << 27) | (raw ? 0x80000000u : 0u));
      waitv<0>();
      if (last_of_four(lane ? dummy : (u32)L_PUB)) {
        const u32 vtot = *LP(volatile u32, L_VAL);
        if (lane < NR) st4(fdst + (size_t)(call & 1) * FLAG_PAR, (call << 6) | ((16u + vtot * 4u + 63u) >> 6));
      }
    }
    role();
    char* const edst = me + WS_E + (size_t)((q * 8 + j) * NSL + ct) * PL + lw * 1024 + l16;
    char* const eflag = me + WS_EFLAG + (size_t)((q * 8 + j) * 32 + ct) * 16;
    auto flag = [&](int t) {
      if (last_of_four(lane ? dummy : (u32)(L_EC + t * 4)) && lane == 0) st4(eflag + t * 64, call);
    };
    int pend = -1;
#pragma unroll 1
    for (int t = 0; t < 8; ++t) {
      if ((int)__builtin_amdgcn_readfirstlane(*LP(volatile u32, L_CVL)) <= t) {
        if (pend >= 0) { waitv<0>(); flag(pend); pend = -1; }
        while ((int)__builtin_amdgcn_readfirstlane(*LP(volatile u32, L_CVL)) <= t) __builtin_amdgcn_s_sleep(2);
      }
      cfence();
      const u32 S = LDS_CV + t * CVS;
      const u32 info = __builtin_amdgcn_readfirstlane(*LP(const u32, S + 2048 + lw * 4));
      const u32x2 cc = *LP(const u32x2, S + lw * 512 + lane * 8);
      const u32 V = S + 2064 + (info & 0x1FFu) * 4;
      const u32 n2d = (info >> 10) & 0x7Fu, n3d = (info >> 17) & 0x1Fu, bb = (info >> 27) & 15u;
      u32 l2 = *LP(const u32, V + lane * 4);
      u32x4 ep;
      if (info >> 31) {
        ep = u32x4{cc[0], cc[1], l2, *LP(const u32, V + 256 + lane * 4)};
      } else {
        const u32 l3 = *LP(const u32, V + n2d * 4 + (lane >> 2) * 4);
        u32 c1[8], c2[4]; u32x2 e1[8], e2[4];
#pragma unroll
        for (int k = 0; k < 8; ++k) { c1[k] = (cc[k >> 2] >> (2 * (k & 3))) & M03; e1[k] = dentry(c1[k], (k & 3) != 3); }
        SCHED();
        l2 = (u32)lane < n2d ? l2 : 0u;
        u32 w4 = 0, c3 = (l3 >> (2 * (lane & 3))) & M03;
#pragma unroll
        for (int k = 0; k < 4; ++k) { c2[k] = (l2 >> (2 * k)) & M03; e2[k] = dentry(c2[k], k != 3); }
        SCHED();
        const u32 bbv = bb * B1;
        if (info & (0x1Fu << 22)) {
          c3 = (u32)lane < 4 * n3d ? c3 : 0u;
          const u32x2 e3 = dentry(c3);
          const u32 l4 = *LP(const u32, V + (n2d + n3d + (lane & 15)) * 4) ^ 0x44444444u;
          const u32 K4 = ((bb + 9u) & 15u) * B1;
          *LP(u32x2, scr + A4O + (lane & 15) * 8) = u32x2{(((l4 & M0F) + K4) & M0F) << 3, ((((l4 >> 4) & M0F) + K4) & M0F) << 3};
          cfence();
          const u32 o4 = scr + A4O + min(wave_scan(e3[1]) - e3[1], 128u);
          const u32x2 ww = pair(o4);
          w4 = __builtin_amdgcn_alignbyte(ww[1], ww[0], o4);
          c3 += e3[0];
        }
        *LP(u32, scr + A3O + lane * 4) = __builtin_amdgcn_perm(w4, ((bbv + 0x000A0B0Cu) & M0F) << 3, c3);
        const u32 ta = __builtin_popcount(cc[0] & (cc[0] >> 1) & 0x55555555u), tb = __builtin_popcount(cc[1] & (cc[1] >> 1) & 0x55555555u);
        const u32 t2 = __builtin_popcount(l2 & (l2 >> 1) & 0x55555555u);
        const u32 tt = ta | (tb << 11) | (t2 << 22);
        const u32 sc = wave_scan(tt);
        const u32 na = __builtin_amdgcn_readlane(sc, 63) & 0x7FFu;
        const u32 ex = sc - tt;
        cfence();
        u32 o2[4], o1[8]; u32x2 w[8];
        o2[0] = (ex >> 22) + (scr + A3O);
#pragma unroll
        for (int k = 1; k < 4; ++k) o2[k] = o2[k - 1] + e2[k - 1][1];
#pragma unroll
        for (int k = 0; k < 4; ++k) w[k] = pair(o2[k]);
        SCHED();
        o1[0] = (ex & 0x7FFu) + scr; o1[4] = ((ex >> 11) & 0x7FFu) + (scr + na);
#pragma unroll
        for (int k = 1; k < 8; ++k) if (k != 4) o1[k] = o1[k - 1] + e1[k - 1][1];
#pragma unroll
        for (int k = 0; k < 4; ++k) c2[k] += e2[k][0];
        SCHED();
        u32x4 y;
#pragma unroll
        for (int k = 0; k < 4; ++k) y[k] = expand(w[k], o2[k], ((bbv + 0x000D0E02u) & M0F) << 3, c2[k]);
        *LP(u32x4, scr + l16) = y;
        cfence();
#pragma unroll
        for (int k = 0; k < 8; ++k) w[k] = pair(o1[k]);
        SCHED();
#pragma unroll
        for (int k = 0; k < 8; ++k) c1[k] += e1[k][0];
        SCHED();
        const u32 T1 = ((bbv + 0x00000F01u) & M0F) << 3;
#pragma unroll
        for (int k = 0; k < 4; ++k) ep[k] = expand(w[k], o1[k], T1, c1[k]) | rotl4(expand(w[4 + k], o1[4 + k], T1, c1[4 + k]));
      }
      st16(edst + (size_t)t * 4 * PL, ep);
      if (pend >= 0) { waitv<1>(); flag(pend); }
      pend = t;
    }
    waitv<0>();
    flag(7);
  } else if (wid == 9) {
    if (b < 8) asm volatile("buffer_inv sc0 sc1" ::: "memory");
    bar();
    const u32 call = __builtin_amdgcn_readfirstlane(callv) + 1u;
    waitv<0>();
    if (b < 8 && lane == 0) st4(inv, call);
    while (__builtin_amdgcn_readfirstlane(*LP(volatile u32, L_PUB)) < 4) __builtin_amdgcn_s_sleep(4);
    role();
    const u64 t0 = __builtin_amdgcn_s_memrealtime();
    {
      const char* f = me + WS_FLAG + (size_t)(call & 1) * FLAG_PAR + (size_t)((src * 8 + j) * NSL + (lane & 31)) * 128;
      u32 go = 1, fv;
      while (true) {
        fv = ld4(f);
        const bool ok = (fv >> 6) >= (call & 0x3FFFFFFu);
        if ((lane & 35) == ct && ok) *LP(volatile u32, L_LEN + (lane >> 2) * 4) = max(min(fv & 63u, 33u), 1u) * 64u + 2048u;
        if (__all(ok)) break;
        if (expired(t0, tmo, call)) { go = 2; *LP(volatile u32, L_BAD) = 1; break; }
        __builtin_amdgcn_s_sleep(2);
      }
      *LP(volatile u32, L_NGO) = go;
    }
    const char* const f = lane < 32 ? me + WS_EFLAG + (size_t)((q * 8 + j) * 32 + lane) * 16
                                    : me + WS_PFLAG + (size_t)(src * 8 + j) * 128 + (lane & 7) * 16;
    u64 known = ~0ull << (32 + NPUSH);
    int nready = 0;
    while (true) {
      const int upto = 4 * min((int)__builtin_amdgcn_readfirstlane(*LP(volatile u32, L_CVL)) + 1, 8);
      bool ok = false;
      if (!((known >> lane) & 1) && (lane >= 32 || lane < upto)) ok = ld4(f) >= call;
      known |= __ballot(ok);
      nready = __builtin_ctzll(~(u64)((u32)known & (u32)(known >> 32)));
      if (nready == NSL) break;
      *LP(volatile u32, L_RDY) = nready;
      if (expired(t0, tmo, call)) break;
      __builtin_amdgcn_s_sleep(32);
    }
    *LP(volatile u32, L_RDY) = NSL;
    if (__builtin_amdgcn_readfirstlane(ld4(tmo)) == call) *LP(volatile u32, L_BAD) = 1;
    *LP(volatile u32, L_RDY) = NSL + 1;
  } else {
    __builtin_amdgcn_s_setprio(3);
    *LP(u32x4, LDS_MISC) = u32x4{0, 0, 0, 0};
    *LP(u32x4, LDS_MISC + 16) = u32x4{0, NPUSH, 0, 0};
    *LP(u32x2, L_ROT) = u32x2{0, 0};
    *LP(u32x4, L_EC) = u32x4{0, 0, 0, 0}; *LP(u32x4, L_EC + 16) = u32x4{0, 0, 0, 0};
    *LP(u32x4, L_LEN) = u32x4{0, 0, 0, 0}; *LP(u32x4, L_LEN + 16) = u32x4{0, 0, 0, 0};
    bar();
    const u32 call = __builtin_amdgcn_readfirstlane(callv) + 1u;
    asm volatile("" :: "s"(call));
    const u32 tstart = (u32)__builtin_amdgcn_s_memrealtime();
    {
      if (b == 0 && lane < 32) st4(me + WS_ROT + lane * 128, (call << 4) | (xcc & 7u));
      const u64 t0 = __builtin_amdgcn_s_memrealtime();
      u32 r;
      while (((r = __builtin_amdgcn_readfirstlane(ld4(me + WS_ROT + (b & 31) * 128))) >> 4) < (call & 0xFFFFFFFu)) {
        if (expired(t0, tmo, call)) { *LP(volatile u32, L_BAD) = 1; r = 0; break; }
        __builtin_amdgcn_s_sleep(1);
      }
      *LP(volatile u32, L_ROT) = (r & 7u) + 1u;
      role();
    }
    {
      const char* f = lane < NE ? me + WS_NFLAG + (size_t)((src * 8 + j) * 8 + lane) * 128 : inv;
      const u64 t0 = __builtin_amdgcn_s_memrealtime();
      while (true) {
        const u64 ok = __ballot(ld4(f) >= call);
        if (ok == ~0ull) break;
        if ((~ok & ((1u << NE) - 1)) == 0 && __builtin_amdgcn_s_memrealtime() - t0 > 500) {
          asm volatile("buffer_inv sc0 sc1\n\ts_waitcnt vmcnt(0)" ::: "memory");
          break;
        }
        if (expired(t0, tmo, call)) {
          *LP(volatile u32, L_BAD) = 1; *LP(volatile u32, L_CVL) = 8; *LP(volatile u32, L_NL) = NSL + KT;
          return;
        }
        __builtin_amdgcn_s_sleep(2);
      }
      waitv<0>();
    }
    u32 lim = 0;
    bool dead = false;
    const char* const base = a.ws[(src - rank) & (NR - 1)] + (size_t)(call & 1) * WS_SLOT + (size_t)(j * NSL) * USTRIDE;
    const u32 tau = q ? TAU : TAU / 4;
    u32 due = (tstart + (u32)max((int)((u32)__builtin_amdgcn_s_memrealtime() - tstart) - CMAX, TREF)) << 4;
    auto pace = [&](u32 bytes) {
      while ((int)(((u32)__builtin_amdgcn_s_memrealtime() << 4) - due) < 0) {}
      due += bytes * tau >> 10;
    };
    int ct_ = 0, cp_ = 0;
    auto issue_c = [&]() {
      const char* sb = base + (size_t)(4 * ct_ + ct) * USTRIDE + U_C;
      const u32 la = LDS_CV + ct_ * CVS;
      if (cp_ == 0) {
        while (!(lim = __builtin_amdgcn_readfirstlane(*LP(volatile u32, L_LEN + ct_ * 4))))
          if (__builtin_amdgcn_readfirstlane(*LP(volatile u32, L_NGO)) == 2) { dead = true; break; }
        if (ct_ == 0) { const u32 now = ((u32)__builtin_amdgcn_s_memrealtime() - CMAX) << 4; if ((int)(now - due) > 0) due = now; }
      }
      if (dead) return;
      pace(cp_ == 2 ? 4u * (lim - 2048u) : 4096u);
      if (cp_ == 2 && lim > 3072u) {
        set_m0(la + 3072); if (l16 + 3072u < lim) glds<0, true>(sb, l16 + 3072u);
        set_m0(la + 4096); if (l16 + 4096u < lim) glds<0, true>(sb, l16 + 4096u);
      }
      set_m0(la + cp_ * 1024);
      if (l16 + cp_ * 1024u < lim) glds<0, true>(sb, l16 + cp_ * 1024u);
      if (++cp_ == 3) { cp_ = 0; ++ct_; }
    };
    int arrived = 0;
    auto issue_n = [&](int s, int p) {
      const char* sb = base + (size_t)s * USTRIDE;
      pace(1024u);
      if (p == 0) {
        while (s >= 8 && arrived < 4 * (s - 5)) arrived = __builtin_amdgcn_readfirstlane(*LP(volatile u32, L_ARR));
        set_m0(LDS_N + (s & 7) * PL); glds<0, false>(sb, l16);
      }
      if (p == 1) glds<1024, false>(sb, l16);
      if (p == 2) glds<2048, false>(sb, l16);
      if (p == 3) glds<3072, false>(sb, l16);
    };
    auto post = [&](u32 addr, u32 v) { *LP(volatile u32, addr) = v; };
    constexpr int NB = 24 - 3 * PRE;
    static_assert(PRE >= 2 && PRE <= 7 && W2 >= 4 && W2 <= 8 && N0 + NB + 4 <= NSL, "");
    int cdone = 0;
    auto landed_c = [&]() { ++cdone; if (cdone % 3 == 0) post(L_CVL, cdone / 3); };
#pragma unroll 1
    for (int s = NPUSH; s < N0; ++s) {
#pragma unroll
      for (int p = 0; p < 4; ++p) { waitv<W2 - 1>(); issue_n(s, p); }
    }
    waitv<3>(); issue_c();
    waitv<1>(); post(L_NL, N0); issue_c();
#pragma unroll 1
    for (int n = 0; n < 3 * PRE - 2; ++n) { waitv<1>(); issue_c(); landed_c(); }
    waitv<1>(); landed_c();
    u32 go;
    do go = __builtin_amdgcn_readfirstlane(*LP(volatile u32, L_NGO)); while (!go);
    if (go == 2) { waitv<0>(); post(L_CVL, 8); post(L_NL, NSL + KT); return; }
#pragma unroll
    for (int p = 0; p < 4; ++p) issue_n(N0, p);
    waitv<4>(); issue_c(); landed_c();
#pragma unroll 1
    for (int s = N0; s < N0 + NB; ++s) {
#pragma unroll
      for (int p = 0; p < 4; ++p) { waitv<4>(); if (p == 3) post(L_NL, s + 1); issue_n(s + 1, p); }
      waitv<4>();
      if (s + 1 < N0 + NB) issue_c();
      landed_c();
    }
#pragma unroll
    for (int n = 4; n < W2; ++n) issue_n(N0 + NB + n / 4, n % 4);
    constexpr int S1 = NSL - KT - (W2 + 3) / 4;
    static_assert(S1 >= N0 + NB, "");
#pragma unroll 1
    for (int s = N0 + NB; s < S1; ++s) {
#pragma unroll
      for (int p = 0; p < 4; ++p) { waitv<W2 - 1>(); if (p == 3) post(L_NL, s + 1); issue_n(s + (p + W2) / 4, (p + W2) % 4); }
    }
    constexpr int NA = 4 * (NSL - KT - S1), NZ = NA + 4 * KT;
#pragma unroll
    for (int n = 0; n < NZ; ++n) {
      waitn(NZ - 1 - n < W2 - 1 ? NZ - 1 - n : W2 - 1);
      if (n < NA ? n % 4 == 3 : n % 2 == 1) post(L_NL, n < NA ? S1 + n / 4 + 1 : NSL - KT + (n - NA) / 2 + 1);
      const int m = n + W2;
      if (m < NA) issue_n(S1 + m / 4, m % 4);
      else if (m < NZ) {
        const int hh = (m - NA) / 2, s = NSL - KT + hh % KT, p = (hh / KT) * 2 + (m - NA) % 2;
        if (p == 2) set_m0(LDS_N + (s & 7) * PL);
        issue_n(s, p);
      }
    }
  }
}

extern "C" void kernel_launch(int rank, int nranks, ncclComm_t comm,
                              hipStream_t stream,
                              void* const* d_in, const int* in_sizes, int n_in,
                              void* d_out, int out_size,
                              const int* in_dtypes, int out_dtype,
                              void* const* d_out_all,
                              void* const* d_ws_all, size_t ws_bytes) {
  if (nranks != NR || (unsigned)rank >= (unsigned)NR || ws_bytes < WS_END || n_in < 4) return;
  static bool init = false;
  if (!init) {
    HIP_CHECK(hipFuncSetAttribute((const void*)ag_gemm, hipFuncAttributeMaxDynamicSharedMemorySize, LDS_TOT));
    init = true;
  }
  Args a;
  for (int p = 0; p < NR; ++p) a.ws[p] = (char*)d_ws_all[(rank + p) & (NR - 1)];
  a.x = (const unsigned char*)d_in[0]; a.w = (const unsigned char*)d_in[1];
  a.sx = (const float*)d_in[2]; a.sw = (const float*)d_in[3];
  a.out = (float*)d_out; a.rank = rank;
  {
    const unsigned cl = CLIQUE;
    unsigned o = 0; int n = 0;
    for (int pass = 0; pass < 2; ++pass)
      for (int i = 0; i < NR; ++i) { const int p = (rank + i) & 7; if ((int)(((cl >> p) ^ (cl >> rank)) & 1) == pass) o |= (unsigned)p << (4 * n++); }
    a.order = o;
  }
  hipLaunchKernelGGL(ag_gemm, dim3(NWG), dim3(NTHR), LDS_TOT, stream, a);
  HIP_CHECK(hipGetLastError());
}
```

```cpp
#include <hip/hip_runtime.h>
#include <rccl/rccl.h>
#include <cstdio>
#include <cstdint>

#pragma clang diagnostic ignored "-Wint-to-pointer-cast"

#define HIP_CHECK(expr) do { hipError_t _e = (expr); if (_e != hipSuccess) { \
    fprintf(stderr, "HIP error %s at %s:%d\n", hipGetErrorString(_e), __FILE__, __LINE__); \
    return; } } while (0)

typedef unsigned u32;
typedef unsigned u32x2 __attribute__((ext_vector_type(2)));
typedef unsigned u32x4 __attribute__((ext_vector_type(4)));
typedef int v2i __attribute__((ext_vector_type(2)));
typedef int v8i __attribute__((ext_vector_type(8)));
typedef float v16f __attribute__((ext_vector_type(16)));
typedef unsigned long long u64;
#define LAS __attribute__((address_space(3)))
#define LP(T, a) ((LAS T*)(u32)(a))

constexpr int NPUSH = 3;
constexpr int PACE = 60;
constexpr int NE = 3;
constexpr int PRE = 5;
constexpr int W2 = 6;
constexpr int KT = 3;
constexpr int TAU = 219;
constexpr int TREF = 185;
constexpr int CMAX = 90;
constexpr unsigned CLIQUE = 0x4D;

constexpr int NR = 8;
constexpr int KTOT = 4096, NL = 512, MS = 512;
constexpr int BM = 64, BN = 128, KS = 128, NSL = KTOT / KS;
constexpr int NWG = 256, NTHR = 640;
constexpr int PL = 4096;
constexpr int U_C = 4096, U_H = 6144, U_V = 6160, USTRIDE = 9216;
constexpr int N0 = NPUSH + NE;
static_assert(NPUSH >= 1 && NPUSH <= 3 && NE >= 1 && N0 <= 8, "");
constexpr size_t WS_SLOT  = (size_t)NWG * USTRIDE;
constexpr size_t WS_PUSH  = 2 * WS_SLOT;
constexpr size_t PUSH_PAR = (size_t)NR * 8 * 8 * PL;
constexpr size_t WS_E     = WS_PUSH + 2 * PUSH_PAR;
constexpr size_t WS_FLAG  = WS_E + (size_t)64 * NSL * PL;
constexpr size_t FLAG_PAR = (size_t)NR * 8 * NSL * 128;
constexpr size_t WS_PFLAG = WS_FLAG + 2 * FLAG_PAR;
constexpr size_t WS_NFLAG = WS_PFLAG + NR * 8 * 128;
constexpr size_t WS_EFLAG = WS_NFLAG + NR * 8 * 8 * 128;
constexpr size_t WS_CNT   = WS_EFLAG + 64 * 512;
constexpr size_t WS_INV   = WS_CNT + NWG * 128;
constexpr size_t WS_ROT   = WS_INV + 16 * 128;
constexpr size_t WS_TMO   = WS_ROT + 32 * 128;
constexpr size_t WS_END   = WS_TMO + 128;
constexpr u64 DEADLINE = 100000000ull;
constexpr u64 SLOW = 20000ull;
constexpr int WSL = KS * BN;
constexpr int LDS_LUT = 0;
constexpr int LDS_CLUT = 128;
constexpr int LDS_MISC = 256;
constexpr int L_BAD = LDS_MISC;
constexpr int L_VAL = LDS_MISC + 4;
constexpr int L_NGO = LDS_MISC + 8;
constexpr int L_PUB = LDS_MISC + 12;
constexpr int L_ARR = LDS_MISC + 16;
constexpr int L_NL = LDS_MISC + 20;
constexpr int L_RDY = LDS_MISC + 24;
constexpr int L_CVL = LDS_MISC + 28;
constexpr int L_LEN = LDS_MISC + 32;
constexpr int L_DUMMY = LDS_MISC + 64;
constexpr int L_ROT = LDS_MISC + 192;
constexpr int L_MC = LDS_MISC + 196;
constexpr int L_EC = LDS_MISC + 208;
constexpr int SCR = 2048;
constexpr int A3O = 1040, A4O = 1312, VBO = 1536;
constexpr int LDS_SCR = 512;
constexpr int LDS_E = 9 * 1024;
constexpr int LDS_N = LDS_E + 4 * PL;
constexpr int LDS_W = LDS_N + 8 * PL;
constexpr int CVS = 4352;
constexpr int LDS_CV = LDS_W + 4 * WSL;
constexpr int LDS_TOT = LDS_CV + 8 * CVS;
static_assert(LDS_SCR + 4 * SCR <= LDS_E && LDS_TOT <= 160 * 1024, "lds");

struct Args {
  const unsigned char* x;
  char* ws[NR];
  const unsigned char* w;
  const float* sx; const float* sw; float* out;
  int rank; u32 order;
};

#define SCHED() __builtin_amdgcn_sched_barrier(0)
template <int N> __device__ __forceinline__ void waitv() { asm volatile("s_waitcnt vmcnt(%0)" :: "n"(N) : "memory"); }
__device__ __forceinline__ void waitn(int n) {
  switch (n) {
    case 0: waitv<0>(); break; case 1: waitv<1>(); break; case 2: waitv<2>(); break; case 3: waitv<3>(); break;
    case 4: waitv<4>(); break; case 5: waitv<5>(); break; case 6: waitv<6>(); break; default: waitv<7>();
  }
}
__device__ __forceinline__ void cfence() { asm volatile("" ::: "memory"); }
__device__ __forceinline__ void bar() {
  cfence(); SCHED();
  __builtin_amdgcn_s_waitcnt(0xC07F);
  __builtin_amdgcn_s_barrier();
  SCHED(); cfence();
}
template <int OFF, bool SYS> __device__ __forceinline__ void glds(u32 lds, const void* sbase, u32 voff) {
  if (SYS) asm volatile("s_mov_b32 m0, %2\n\ts_nop 0\n\tglobal_load_lds_dwordx4 %0, %1 offset:%3 sc0 sc1" :: "v"(voff), "s"(sbase), "s"(lds), "n"(OFF) : "memory");
  else asm volatile("s_mov_b32 m0, %2\n\ts_nop 0\n\tglobal_load_lds_dwordx4 %0, %1 offset:%3" :: "v"(voff), "s"(sbase), "s"(lds), "n"(OFF) : "memory");
}
__device__ __forceinline__ void glds4(u32 lds, const void* sbase, const u32 (&voff)[4]) {
  asm volatile("s_mov_b32 m0, %5\n\ts_nop 0\n\tglobal_load_lds_dwordx4 %0, %4\n\tglobal_load_lds_dwordx4 %1, %4 offset:1024\n\t"
               "global_load_lds_dwordx4 %2, %4 offset:2048\n\tglobal_load_lds_dwordx4 %3, %4 offset:3072"
               :: "v"(voff[0]), "v"(voff[1]), "v"(voff[2]), "v"(voff[3]), "s"(sbase), "s"(lds) : "memory");
}
__device__ __forceinline__ void st16(void* p, u32x4 v) { asm volatile("global_store_dwordx4 %0, %1, off sc0 sc1\n\ts_nop 1" :: "v"(p), "v"(v) : "memory"); }
__device__ __forceinline__ void st8(void* p, u32x2 v) { asm volatile("global_store_dwordx2 %0, %1, off sc0 sc1\n\ts_nop 1" :: "v"(p), "v"(v) : "memory"); }
__device__ __forceinline__ void st4(void* p, u32 v) { asm volatile("global_store_dword %0, %1, off sc0 sc1\n\ts_nop 1" :: "v"(p), "v"(v) : "memory"); }
__device__ __forceinline__ u32 ld4(const void* p) {
  u32 v;
  asm volatile("global_load_dword %0, %1, off sc0 sc1\n\ts_waitcnt vmcnt(0)" : "=&v"(v) : "v"(p) : "memory");
  return v;
}

__device__ __forceinline__ bool last_of_four(u32 addr) {
  u32 nth;
  asm volatile("ds_add_rtn_u32 %0, %1, %2\n\ts_waitcnt lgkmcnt(0)" : "=&v"(nth) : "v"(addr), "v"(1u) : "memory");
  return __builtin_amdgcn_readfirstlane(nth) == 3;
}
__device__ __forceinline__ bool expired(u64 t0, u32* tmo, u32 call) {
  const u64 dt = __builtin_amdgcn_s_memrealtime() - t0;
  if (dt <= SLOW) return false;
  if (dt > DEADLINE) { st4(tmo, call); waitv<0>(); return true; }
  __builtin_amdgcn_s_sleep(64);
  return __builtin_amdgcn_readfirstlane(ld4(tmo)) == call;
}

__device__ __forceinline__ u32 wave_scan(u32 x) {
  x += __builtin_amdgcn_update_dpp(0u, x, 0x111, 0xf, 0xf, true);
  x += __builtin_amdgcn_update_dpp(0u, x, 0x112, 0xf, 0xf, true);
  x += __builtin_amdgcn_update_dpp(0u, x, 0x114, 0xf, 0xf, true);
  x += __builtin_amdgcn_update_dpp(0u, x, 0x118, 0xf, 0xf, true);
  x += __builtin_amdgcn_update_dpp(0u, x, 0x142, 0xa, 0xf, false);
  x += __builtin_amdgcn_update_dpp(0u, x, 0x143, 0xc, 0xf, false);
  return x;
}
constexpr u32 B1 = 0x01010101u, M0F = 0x0F0F0F0Fu, MSM = 0x87878787u, MEX = 0x78787878u, M03 = 0x03030303u;
__device__ __forceinline__ u32 rotl4(u32 x) { return __builtin_amdgcn_alignbit(x, x, 28); }
__device__ __forceinline__ u32 rotr4(u32 x) { return __builtin_amdgcn_alignbit(x, x, 4); }
__device__ __forceinline__ u32x2 centry(u32 p) {
  return *LP(const u32x2, __builtin_amdgcn_udot4(p & 0x04040404u, 0x10080402u, (u32)LDS_CLUT, false));
}
__device__ __forceinline__ u32x2 dentry(u32 c, bool with_count = true) {
  const u32 p = __builtin_amdgcn_udot4(__builtin_amdgcn_perm(0u, 0x01000000u, c), 0x40201008u, (u32)LDS_LUT, false);
  if (with_count) return *LP(const u32x2, p);
  return u32x2{*LP(const u32, p), 0u};
}
__device__ __forceinline__ void put4(u32 o, u32 y) {
  const u64 w = (u64)y << ((o & 3u) * 8u);
  LAS u32* p = LP(u32, o & ~3u);
  __hip_atomic_fetch_or(p, (u32)w, __ATOMIC_RELAXED, __HIP_MEMORY_SCOPE_WAVEFRONT);
  __hip_atomic_fetch_or(p + 1, (u32)(w >> 32), __ATOMIC_RELAXED, __HIP_MEMORY_SCOPE_WAVEFRONT);
}
__device__ __forceinline__ u32x2 pair(u32 o) {
  const LAS u32* p = LP(const u32, o & ~3u);
  return u32x2{p[0], p[1]};
}
__device__ __forceinline__ u32 expand(u32x2 w, u32 o, u32 table, u32 sel) {
  return __builtin_amdgcn_perm(__builtin_amdgcn_alignbyte(w[1], w[0], o), table, sel);
}

__global__ void __launch_bounds__(NTHR, 3) ag_gemm(Args a) {
  const int tid = threadIdx.x, lane = tid & 63;
  const int wid = __builtin_amdgcn_readfirstlane(tid >> 6);
  const int b = blockIdx.x;
  char* const me = a.ws[0];
  const int jp = b & 7, idx = b >> 3, ct = idx & 3, pe = idx >> 3;
  const int ps = pe == 3 ? NPUSH + (idx & 7) : pe;
  const int brow = (wid & 3) * 16 + (lane >> 2);
  const unsigned char* const xlane = a.x + (size_t)(jp * BM) * KTOT + (wid < 4 ? ps : idx) * KS + (u32)(brow * KTOT + (((lane & 3) ^ ((brow >> 2) & 3)) << 5));
  const u32x4 v0 = *(const u32x4*)xlane, v1 = *(const u32x4*)(xlane + 16);
  u32* const cnt = (u32*)(me + WS_CNT + (size_t)b * 128);
  const u32 callv = __hip_atomic_load(cnt, __ATOMIC_RELAXED, __HIP_MEMORY_SCOPE_AGENT);
  cfence(); SCHED();
  const int rank = a.rank;
  int j, q, src;
  auto role_for = [&](u32 rot) {
    const int x = (b + rot) & 7;
    j = (x & 3) * 2 + ((idx >> 2) & 1); q = (x & 4) | (idx >> 3);
    src = (a.order >> (4 * q)) & (NR - 1);
  };
  auto role = [&]() {
    u32 r;
    while (!(r = __builtin_amdgcn_readfirstlane(*LP(volatile u32, L_ROT)))) __builtin_amdgcn_s_sleep(1);
    role_for(r - 1);
  };
  const u32 xcc = __builtin_amdgcn_s_getreg((3 << 11) | 20) & 15u;
  u32* const tmo = (u32*)(me + WS_TMO);
  const u32 dummy = L_DUMMY + (lane & 31) * 4;
  const u32 l16 = lane * 16u;
  char* const inv = me + WS_INV + (size_t)xcc * 128;

  if (wid < 4) {
    const bool early = pe == 3 && (idx & 7) < NE;
    auto nplane = [&](const u32x4 v0, const u32x4 v1) {
      u32x4 np;
#pragma unroll
      for (int k = 0; k < 4; ++k) np[k] = (v0[k] & MSM) | rotl4(v1[k] & MSM);
      return np;
    };
    char* nflag = a.ws[lane & (NR - 1)] + WS_NFLAG + (size_t)((rank * 8 + jp) * 8 + (idx & 7)) * 128;
    if (tid < 16) {
      u32 add = 0, cs = 0x0C0C0C0Cu, n = 0;
#pragma unroll
      for (int i = 0; i < 4; ++i)
        if ((tid >> i) & 1) { add |= (1 + n) << (8 * i); cs = (cs & ~(0xFFu << (8 * n))) | (i << (8 * n)); ++n; }
      *LP(u32x2, LDS_LUT + tid * 8) = u32x2{add, n};
      *LP(u32x2, LDS_CLUT + tid * 8) = u32x2{cs, n};
    }
    const int kr = lane >> 3, cp = lane & 7;
    u32 wvo[4];
#pragma unroll
    for (int i = 0; i < 4; ++i) wvo[i] = (kr + i * 8) * NL + ((cp >> 1) ^ ((kr >> 1) & 3)) * 32 + (cp & 1) * 16 - i * 1024;
    const unsigned char* const wsrc = a.w + (size_t)(wid * 32) * NL + ct * BN;
    auto stage_w = [&](int s) { glds4(LDS_W + (s & 3) * WSL + wid * 4096, wsrc + (size_t)s * KS * NL, wvo); };
    char* const pdst = a.ws[idx & (NR - 1)];
    char* pp = (pe == 3 ? me + (size_t)(jp * NSL + ps) * USTRIDE : pdst + WS_PUSH + (size_t)((rank * 8 + jp) * 8 + pe) * PL) + wid * 1024 + l16;
    const size_t podd = pe == 3 ? WS_SLOT : PUSH_PAR;
    asm volatile("" : "+v"(pp));
    bar();
    const u32x4 np0 = nplane(v0, v1);
    const u32 call = __builtin_amdgcn_readfirstlane(callv) + 1u;
    const size_t par = call & 1;
    if (pe == 1 || pe == 2) {
      asm volatile("" :: "v"(np0));
      const u64 tpush = __builtin_amdgcn_s_memrealtime();
      while (__builtin_amdgcn_s_memrealtime() - tpush < (u64)(pe * PACE)) __builtin_amdgcn_s_sleep(1);
    }
    if (pe < NPUSH || early) st16(pp + par * podd, np0);
    auto last = [&]() { waitv<0>(); return last_of_four(lane ? dummy : (u32)L_MC); };
    if (early) {
      asm volatile("" : "+v"(nflag));
      if (last() && lane < NR) st4(nflag, call);
    }
    float sc = a.sx[0] * a.sw[0];
    while (__builtin_amdgcn_readfirstlane(*LP(volatile u32, L_PUB)) < 4) __builtin_amdgcn_s_sleep(4);
    __builtin_amdgcn_s_sleep(32);
    if (pe < NPUSH && last() && lane == 0) st4(pdst + WS_PFLAG + (size_t)(rank * 8 + jp) * 128 + pe * 16, call);
    asm volatile("" : "+v"(sc));
    role();
    const char* const esrc = me + WS_E + (size_t)((q * 8 + j) * NSL) * PL + wid * 1024;
    const char* const psrc = me + WS_PUSH + par * PUSH_PAR + (size_t)((src * 8 + j) * 8) * PL + wid * 1024;
    auto stage_e = [&](int s, bool push) {
      if (push) glds<0, true>(LDS_N + (s & 7) * PL + wid * 1024, psrc + s * PL, l16);
      glds<0, true>(LDS_E + (s & 3) * PL + wid * 1024, esrc + s * PL, l16);
    };
    v16f acc[2] = {};
    const int r31 = lane & 31, h = lane >> 5, wm = wid >> 1, wn = wid & 1;
    const int q8 = (lane & 15) >> 1, p8 = lane & 1, g1 = (lane >> 4) & 1;
    u32 boff[2], aoff[2];
#pragma unroll
    for (int i = 0; i < 2; ++i) {
      boff[i] = LDS_W + (32 * h + q8) * 128 + (((wn * 2 + i) ^ ((q8 >> 1) & 3)) << 5) + g1 * 16 + p8 * 8;
      aoff[i] = (wm * 32 + r31) * 64 + (((i * 2 + h) ^ ((r31 >> 2) & 3)) << 4);
    }
    struct Frag { u32x4 ep[2]; v8i bf[2][2]; };
    auto ldfrag = [&](Frag& f, int kk, int s) {
      f.ep[kk] = *LP(const u32x4, aoff[kk] + (s & 3) * PL + LDS_E);
#pragma unroll
      for (int n = 0; n < 2; ++n) {
        v2i t[4];
#pragma unroll
        for (int r = 0; r < 4; ++r) t[r] = __builtin_amdgcn_ds_read_tr8_b64_v2i32(LP(v2i, boff[n] + (s & 3) * WSL) + (kk * 64 + r * 8) * 16);
        f.bf[kk][n] = v8i{t[0][0], t[0][1], t[1][0], t[1][1], t[2][0], t[2][1], t[3][0], t[3][1]};
      }
    };
    auto afrag = [&](const u32x4 np, const u32x4 ep) {
      v8i af;
#pragma unroll
      for (int k = 0; k < 4; ++k) {
        af[k] = (np[k] & MSM) | (ep[k] & MEX);
        af[4 + k] = rotr4((np[k] & MEX) | (ep[k] & MSM));
      }
      return af;
    };
#define MFMA(n, af, bf) acc[n] = __builtin_amdgcn_mfma_scale_f32_32x32x64_f8f6f4(af, bf, acc[n], 0, 0, 0, 0, 0, 0)
    auto step = [&](int s, const Frag& cur, Frag& nxt, bool more, bool stg) {
      const u32x4 np0 = *LP(const u32x4, aoff[0] + (s & 7) * PL + LDS_N);
      const u32x4 np1 = *LP(const u32x4, aoff[1] + (s & 7) * PL + LDS_N);
      SCHED();
      if (stg) { stage_e(s + 4, false); stage_w(s + 4); }
      SCHED();
      const v8i a0 = afrag(np0, cur.ep[0]);
      MFMA(0, a0, cur.bf[0][0]);
      SCHED();
      const v8i a1 = afrag(np1, cur.ep[1]);
      SCHED();
      if (!more) {
        MFMA(0, a1, cur.bf[1][0]); MFMA(1, a0, cur.bf[0][1]); MFMA(1, a1, cur.bf[1][1]);
        return;
      }
      MFMA(1, a0, cur.bf[0][1]);
      SCHED();
      ldfrag(nxt, 0, s + 1);
      SCHED();
      MFMA(0, a1, cur.bf[1][0]);
      SCHED();
      ldfrag(nxt, 1, s + 1);
      SCHED();
      MFMA(1, a1, cur.bf[1][1]);
      SCHED();
      asm volatile("" :: "v"(a0), "v"(cur.bf[0][0]), "v"(cur.bf[0][1]), "v"(cur.bf[1][0]));
    };
    const u32 arr = lane ? dummy : (u32)L_ARR;
    auto count = [&](u32 addr) { asm volatile("ds_add_u32 %0, %1" :: "v"(addr), "v"(1u) : "memory"); };
    auto spin = [&](int na, int nl, int nr, int nc = 0) {
      while (true) {
        u32x4 w;
        asm volatile("ds_read_b128 %0, %1\n\ts_waitcnt lgkmcnt(0)" : "=&v"(w) : "v"((u32)L_ARR) : "memory");
        if (__all((int)w[0] >= na && (int)w[1] >= nl && (int)w[2] >= nr && (int)w[3] >= nc)) break;
      }
    };
    auto meet = [&](int s) {
      SCHED();
      asm volatile("s_waitcnt lgkmcnt(0)\n\tds_add_u32 %0, %1" :: "v"(arr), "v"(1u) : "memory");
      spin(4 * s + 8, s + 1, min(s + 5, NSL));
      __builtin_amdgcn_s_waitcnt(0xC07F);
      SCHED();
    };
    {
      Frag fa, fb;
#pragma unroll 1
      for (int s = 0; s < 4; ++s) { stage_w(s); __builtin_amdgcn_s_sleep(8); }
      spin(0, 0, 4);
#pragma unroll 1
      for (int s = 0; s < 4; ++s) stage_e(s, s < NPUSH);
      waitv<0>(); meet(-1);
      ldfrag(fa, 0, 0); ldfrag(fa, 1, 0);
      meet(0); step(0, fa, fb, true, true);
#pragma unroll 1
      for (int s = 1; s < NSL - 5; s += 2) {
        waitv<10>(); meet(s);
        step(s, fb, fa, true, true);
        waitv<10>(); meet(s + 1);
        step(s + 1, fa, fb, true, true);
      }
      waitv<10>(); meet(NSL - 5); step(NSL - 5, fb, fa, true, true);
      static_assert(KT == 3, "");
      waitv<10>(); meet(NSL - 4); step(NSL - 4, fa, fb, true, false);
      const int hb = wm * KT;
      const u32 arr2 = lane ? dummy : (u32)L_CVL;
      waitv<5>(); count(arr);
      if (wm) { waitv<0>(); count(arr2); }
      SCHED(); spin(4 * (NSL - 3) + 8, NSL - 2 + hb, 0, 0); SCHED(); step(NSL - 3, fb, fa, true, false);
      if (!wm) { waitv<0>(); count(arr2); }
      SCHED(); spin(0, NSL - 1 + hb, 0, 12); SCHED(); step(NSL - 2, fa, fb, true, false);
      spin(0, 0, NSL + 1, 0);
      if (*LP(volatile u32, L_BAD)) sc = __builtin_nanf("");
      asm volatile("" : "+v"(sc));
      SCHED(); spin(0, NSL + hb, 0, 0); SCHED(); step(NSL - 1, fb, fa, false, false);
    }
    if (tid == 0) st4(cnt, call);
    float* o = a.out + (size_t)(src * MS + j * BM + wm * 32 + 4 * h) * NL + ct * BN + wn * 64 + r31;
#pragma unroll
    for (int n = 0; n < 2; ++n)
#pragma unroll
      for (int i = 0; i < 16; ++i)
        __hip_atomic_store(o + (size_t)((i & 3) + 8 * (i >> 2)) * NL + n * 32, acc[n][i] * sc, __ATOMIC_RELAXED, __HIP_MEMORY_SCOPE_SYSTEM);
  } else if (wid < 8) {
    const int lw = wid - 4;
    const u32 scr = LDS_SCR + lw * SCR;
    u32 call;
    {
      char* fdst = a.ws[lane & (NR - 1)] + WS_FLAG + (size_t)((rank * 8 + jp) * NSL + idx) * 128;
      *LP(u32x4, scr + l16) = u32x4{0, 0, 0, 0};
      *LP(u32x2, scr + 1024 + lane * 8) = u32x2{0, 0};
      bar();
      call = __builtin_amdgcn_readfirstlane(callv) + 1u;
      asm volatile("" : "+v"(fdst));
      char* const dst = me + (size_t)(call & 1) * WS_SLOT + (size_t)(jp * NSL + idx) * USTRIDE;
      const u32 vbuf = scr + VBO;
      const u32 v[8] = {v0[0], v0[1], v0[2], v0[3], v1[0], v1[1], v1[2], v1[3]};
      u32 z[8], sum = 0;
#pragma unroll
      for (int k = 0; k < 8; ++k) { z[k] = v[k] & MEX; if (k & 1) sum = __builtin_amdgcn_udot4(z[k], B1, sum, false); }
      sum = __builtin_amdgcn_readlane(wave_scan(sum), 63);
      const u32 bb = ((sum + 7946u) >> 13) & 15u;
      const u32 kz = ((7u - bb) & 15u) * 0x08080808u;
      u32x4 np;
#pragma unroll
      for (int k = 0; k < 4; ++k) np[k] = (v[k] & MSM) | rotl4(v[4 + k] & MSM);
      if (idx >= N0) st16(dst + lw * 1024 + l16, np);
      u32x2 e1[8], cp2 = {0, 0};
#pragma unroll
      for (int k = 0; k < 8; ++k) {
        z[k] = (((z[k] + kz) >> 3) & M0F) ^ 0x04040404u;
        const u32 p = __builtin_amdgcn_perm(0xFFFFFFFFu, 0x8201FFFFu, z[k]);
        cp2[k >> 2] |= (p & M03) << (2 * (k & 3));
        e1[k] = centry(p);
      }
      const u32 t1 = (e1[0][1] + e1[1][1] + e1[2][1] + e1[3][1]) | ((e1[4][1] + e1[5][1] + e1[6][1] + e1[7][1]) << 16);
      const u32 sc1 = wave_scan(t1);
      const u32 tot1 = __builtin_amdgcn_readlane(sc1, 63);
      const u32 n2 = (tot1 & 0xFFFFu) + (tot1 >> 16);
      bool raw = n2 > 1024u;
      u32 n2d = 0, n3d = 0, n4d = 0;
      if (!raw) {
        const u32 ex = sc1 - t1 + (scr | ((scr + (tot1 & 0xFFFFu)) << 16));
        u32 o = ex & 0xFFFFu;
#pragma unroll
        for (int k = 0; k < 8; ++k) {
          if (k == 4) o = ex >> 16;
          put4(o, __builtin_amdgcn_perm(0u, z[k], e1[k][0])); o += e1[k][1];
        }
        cfence();
        const u32x4 q2 = *LP(const u32x4, scr + l16);
        u32 l2 = 0; u32x2 e2[4];
#pragma unroll
        for (int k = 0; k < 4; ++k) {
          const u32 p = __builtin_amdgcn_perm(0xFFFFFFFFu, 0xFFFF8201u, q2[k] ^ B1);
          l2 |= (p & M03) << (2 * k); e2[k] = centry(p);
        }
        *LP(u32, vbuf + lane * 4) = l2;
        const u32 t2 = e2[0][1] + e2[1][1] + e2[2][1] + e2[3][1];
        const u32 sc2 = wave_scan(t2);
        const u32 n3 = __builtin_amdgcn_readlane(sc2, 63);
        raw = n3 > 256u;
        if (!raw) {
          n2d = (n2 + 15u) >> 4; n3d = (n3 + 15u) >> 4;
          if (t2) {
            o = scr + A3O + sc2 - t2;
#pragma unroll
            for (int k = 0; k < 4; ++k) { put4(o, __builtin_amdgcn_perm(0u, q2[k] ^ 0x05050505u, e2[k][0])); o += e2[k][1]; }
          }
          cfence();
          const u32 q3 = *LP(const u32, scr + A3O + lane * 4) ^ 0x05050505u;
          const u32 p3 = __builtin_amdgcn_perm(0x800182FFu, 0xFFFFFFFFu, q3);
          u32 l3 = (p3 & M03) << (2 * (lane & 3));
          l3 |= __builtin_amdgcn_update_dpp(0u, l3, 0xB1, 0xf, 0xf, false);
          l3 |= __builtin_amdgcn_update_dpp(0u, l3, 0x4E, 0xf, 0xf, false);
          *LP(u32, vbuf + (n2d + (lane >> 2)) * 4) = l3;
          if (__any(p3 & 0x04040404u)) {
            const u32x2 e3 = centry(p3);
            const u32 sc3 = wave_scan(e3[1]);
            const u32 n4 = __builtin_amdgcn_readlane(sc3, 63);
            raw = n4 > 128u;
            if (!raw) {
              if (e3[1]) put4(scr + A4O + sc3 - e3[1], __builtin_amdgcn_perm(0u, q3, e3[0]));
              cfence();
              const u32x2 q4 = *LP(const u32x2, scr + A4O + (lane & 15) * 8);
              n4d = (n4 + 7u) >> 3;
              if (lane < 16) *LP(u32, vbuf + (n2d + n3d + lane) * 4) = q4[0] | (q4[1] << 4);
            }
          }
        }
      }
      cfence();
      u32 vd = n2d + n3d + n4d;
      if (raw) {
        cp2 = u32x2{(v[0] & MEX) | rotl4(v[4] & MEX), (v[1] & MEX) | rotl4(v[5] & MEX)}; vd = 128;
        *LP(u32, vbuf + lane * 4) = (v[2] & MEX) | rotl4(v[6] & MEX);
        *LP(u32, vbuf + 256 + lane * 4) = (v[3] & MEX) | rotl4(v[7] & MEX);
      }
      cfence();
      u32 voff;
      asm volatile("ds_add_rtn_u32 %0, %1, %2\n\ts_waitcnt lgkmcnt(0)" : "=&v"(voff) : "v"(lane ? dummy : (u32)L_VAL), "v"(vd) : "memory");
      const u32 va = *LP(const u32, vbuf + lane * 4), vb = *LP(const u32, vbuf + 256 + lane * 4);
      st8(dst + U_C + lw * 512 + lane * 8, cp2);
      voff = __builtin_amdgcn_readfirstlane(voff);
      if ((u32)lane < vd) st4(dst + U_V + (voff + lane) * 4, va);
      if ((u32)lane + 64u < vd) st4(dst + U_V + (voff + 64 + lane) * 4, vb);
      if (lane == 0) st4(dst + U_H + lw * 4, voff | (n2d << 10) | (n3d << 17) | (n4d << 22) | (bb << 27) | (raw ? 0x80000000u : 0u));
      waitv<0>();
      if (last_of_four(lane ? dummy : (u32)L_PUB)) {
        const u32 vtot = *LP(volatile u32, L_VAL);
        if (lane < NR) st4(fdst + (size_t)(call & 1) * FLAG_PAR, (call << 6) | ((16u + vtot * 4u + 63u) >> 6));
      }
    }
    role();
    char* const edst = me + WS_E + (size_t)((q * 8 + j) * NSL + ct) * PL + lw * 1024 + l16;
    char* const eflag = me + WS_EFLAG + (size_t)((q * 8 + j) * 32 + ct) * 16;
    auto flag = [&](int t) {
      if (last_of_four(lane ? dummy : (u32)(L_EC + t * 4)) && lane == 0) st4(eflag + t * 64, call);
    };
    int pend = -1;
#pragma unroll 1
    for (int t = 0; t < 8; ++t) {
      if ((int)__builtin_amdgcn_readfirstlane(*LP(volatile u32, L_CVL)) <= t) {
        if (pend >= 0) { waitv<0>(); flag(pend); pend = -1; }
        while ((int)__builtin_amdgcn_readfirstlane(*LP(volatile u32, L_CVL)) <= t) __builtin_amdgcn_s_sleep(2);
      }
      cfence();
      const u32 S = LDS_CV + t * CVS;
      const u32 info = __builtin_amdgcn_readfirstlane(*LP(const u32, S + 2048 + lw * 4));
      const u32x2 cc = *LP(const u32x2, S + lw * 512 + lane * 8);
      const u32 V = S + 2064 + (info & 0x1FFu) * 4;
      const u32 n2d = (info >> 10) & 0x7Fu, n3d = (info >> 17) & 0x1Fu, bb = (info >> 27) & 15u;
      u32 l2 = *LP(const u32, V + lane * 4);
      u32x4 ep;
      if (info >> 31) {
        ep = u32x4{cc[0], cc[1], l2, *LP(const u32, V + 256 + lane * 4)};
      } else {
        const u32 l3 = *LP(const u32, V + n2d * 4 + (lane >> 2) * 4);
        u32 c1[8], c2[4]; u32x2 e1[8], e2[4];
#pragma unroll
        for (int k = 0; k < 8; ++k) { c1[k] = (cc[k >> 2] >> (2 * (k & 3))) & M03; e1[k] = dentry(c1[k], (k & 3) != 3); }
        SCHED();
        l2 = (u32)lane < n2d ? l2 : 0u;
        u32 w4 = 0, c3 = (l3 >> (2 * (lane & 3))) & M03;
#pragma unroll
        for (int k = 0; k < 4; ++k) { c2[k] = (l2 >> (2 * k)) & M03; e2[k] = dentry(c2[k], k != 3); }
        SCHED();
        const u32 bbv = bb * B1;
        if (info & (0x1Fu << 22)) {
          c3 = (u32)lane < 4 * n3d ? c3 : 0u;
          const u32x2 e3 = dentry(c3);
          const u32 l4 = *LP(const u32, V + (n2d + n3d + (lane & 15)) * 4) ^ 0x44444444u;
          const u32 K4 = ((bb + 9u) & 15u) * B1;
          *LP(u32x2, scr + A4O + (lane & 15) * 8) = u32x2{(((l4 & M0F) + K4) & M0F) << 3, ((((l4 >> 4) & M0F) + K4) & M0F) << 3};
          cfence();
          const u32 o4 = scr + A4O + min(wave_scan(e3[1]) - e3[1], 128u);
          const u32x2 ww = pair(o4);
          w4 = __builtin_amdgcn_alignbyte(ww[1], ww[0], o4);
          c3 += e3[0];
        }
        *LP(u32, scr + A3O + lane * 4) = __builtin_amdgcn_perm(w4, ((bbv + 0x000A0B0Cu) & M0F) << 3, c3);
        const u32 ta = __builtin_popcount(cc[0] & (cc[0] >> 1) & 0x55555555u), tb = __builtin_popcount(cc[1] & (cc[1] >> 1) & 0x55555555u);
        const u32 t2 = __builtin_popcount(l2 & (l2 >> 1) & 0x55555555u);
        const u32 tt = ta | (tb << 11) | (t2 << 22);
        const u32 sc = wave_scan(tt);
        const u32 na = __builtin_amdgcn_readlane(sc, 63) & 0x7FFu;
        const u32 ex = sc - tt;
        cfence();
        u32 o2[4], o1[8]; u32x2 w[8];
        o2[0] = (ex >> 22) + (scr + A3O);
#pragma unroll
        for (int k = 1; k < 4; ++k) o2[k] = o2[k - 1] + e2[k - 1][1];
#pragma unroll
        for (int k = 0; k < 4; ++k) w[k] = pair(o2[k]);
        SCHED();
        o1[0] = (ex & 0x7FFu) + scr; o1[4] = ((ex >> 11) & 0x7FFu) + (scr + na);
#pragma unroll
        for (int k = 1; k < 8; ++k) if (k != 4) o1[k] = o1[k - 1] + e1[k - 1][1];
#pragma unroll
        for (int k = 0; k < 4; ++k) c2[k] += e2[k][0];
        SCHED();
        u32x4 y;
#pragma unroll
        for (int k = 0; k < 4; ++k) y[k] = expand(w[k], o2[k], ((bbv + 0x000D0E02u) & M0F) << 3, c2[k]);
        *LP(u32x4, scr + l16) = y;
        cfence();
#pragma unroll
        for (int k = 0; k < 8; ++k) w[k] = pair(o1[k]);
        SCHED();
#pragma unroll
        for (int k = 0; k < 8; ++k) c1[k] += e1[k][0];
        SCHED();
        const u32 T1 = ((bbv + 0x00000F01u) & M0F) << 3;
#pragma unroll
        for (int k = 0; k < 4; ++k) ep[k] = expand(w[k], o1[k], T1, c1[k]) | rotl4(expand(w[4 + k], o1[4 + k], T1, c1[4 + k]));
      }
      st16(edst + (size_t)t * 4 * PL, ep);
      if (pend >= 0) { waitv<1>(); flag(pend); }
      pend = t;
    }
    waitv<0>();
    flag(7);
  } else if (wid == 9) {
    if (b < 8) asm volatile("buffer_inv sc0 sc1" ::: "memory");
    bar();
    const u32 call = __builtin_amdgcn_readfirstlane(callv) + 1u;
    waitv<0>();
    if (b < 8 && lane == 0) st4(inv, call);
    while (__builtin_amdgcn_readfirstlane(*LP(volatile u32, L_PUB)) < 4) __builtin_amdgcn_s_sleep(4);
    role();
    const u64 t0 = __builtin_amdgcn_s_memrealtime();
    {
      const char* f = me + WS_FLAG + (size_t)(call & 1) * FLAG_PAR + (size_t)((src * 8 + j) * NSL + (lane & 31)) * 128;
      u32 go = 1, fv;
      while (true) {
        fv = ld4(f);
        const bool ok = (fv >> 6) >= (call & 0x3FFFFFFu);
        if ((lane & 35) == ct && ok) *LP(volatile u32, L_LEN + (lane >> 2) * 4) = max(min(fv & 63u, 33u), 1u) * 64u + 2048u;
        if (__all(ok)) break;
        if (expired(t0, tmo, call)) { go = 2; *LP(volatile u32, L_BAD) = 1; break; }
        __builtin_amdgcn_s_sleep(2);
      }
      *LP(volatile u32, L_NGO) = go;
    }
    const char* const f = lane < 32 ? me + WS_EFLAG + (size_t)((q * 8 + j) * 32 + lane) * 16
                                    : me + WS_PFLAG + (size_t)(src * 8 + j) * 128 + (lane & 7) * 16;
    u64 known = ~0ull << (32 + NPUSH);
    int nready = 0;
    while (true) {
      const int upto = 4 * min((int)__builtin_amdgcn_readfirstlane(*LP(volatile u32, L_CVL)) + 1, 8);
      bool ok = false;
      if (!((known >> lane) & 1) && (lane >= 32 || lane < upto)) ok = ld4(f) >= call;
      known |= __ballot(ok);
      nready = __builtin_ctzll(~(u64)((u32)known & (u32)(known >> 32)));
      if (nready == NSL) break;
      *LP(volatile u32, L_RDY) = nready;
      if (expired(t0, tmo, call)) break;
      __builtin_amdgcn_s_sleep(32);
    }
    *LP(volatile u32, L_RDY) = NSL;
    if (__builtin_amdgcn_readfirstlane(ld4(tmo)) == call) *LP(volatile u32, L_BAD) = 1;
    *LP(volatile u32, L_RDY) = NSL + 1;
  } else {
    __builtin_amdgcn_s_setprio(3);
    *LP(u32x4, LDS_MISC) = u32x4{0, 0, 0, 0};
    *LP(u32x4, LDS_MISC + 16) = u32x4{0, NPUSH, 0, 0};
    *LP(u32x2, L_ROT) = u32x2{0, 0};
    *LP(u32x4, L_EC) = u32x4{0, 0, 0, 0}; *LP(u32x4, L_EC + 16) = u32x4{0, 0, 0, 0};
    *LP(u32x4, L_LEN) = u32x4{0, 0, 0, 0}; *LP(u32x4, L_LEN + 16) = u32x4{0, 0, 0, 0};
    bar();
    const u32 call = __builtin_amdgcn_readfirstlane(callv) + 1u;
    asm volatile("" :: "s"(call));
    const u32 tstart = (u32)__builtin_amdgcn_s_memrealtime();
    {
      if (b == 0 && lane < 32) st4(me + WS_ROT + lane * 128, (call << 4) | (xcc & 7u));
      const u64 t0 = __builtin_amdgcn_s_memrealtime();
      u32 rot = (xcc - b) & 7u;
      bool told = false;
      while (true) {
        role_for(rot);
        const u32 v = ld4(lane < NE ? me + WS_NFLAG + (size_t)((src * 8 + j) * 8 + lane) * 128 : lane < 63 ? inv : me + WS_ROT + (b & 31) * 128);
        const u64 ok = __ballot(v >= call) | (1ull << 63);
        const u32 r = __builtin_amdgcn_readlane(v, 63);
        if (!told && (r >> 4) >= (call & 0xFFFFFFFu)) {
          told = true;
          *LP(volatile u32, L_ROT) = (r & 7u) + 1u;
          if ((r & 7u) != rot) { rot = r & 7u; continue; }
        }
        if (told && ok == ~0ull) break;
        if (told && (~ok & ((1u << NE) - 1)) == 0 && __builtin_amdgcn_s_memrealtime() - t0 > 500) {
          asm volatile("buffer_inv sc0 sc1\n\ts_waitcnt vmcnt(0)" ::: "memory");
          break;
        }
        if (expired(t0, tmo, call)) {
          *LP(volatile u32, L_BAD) = 1; *LP(volatile u32, L_ROT) = rot + 1u;
          *LP(volatile u32, L_CVL) = 8; *LP(volatile u32, L_NL) = NSL + KT;
          return;
        }
        __builtin_amdgcn_s_sleep(2);
      }
      waitv<0>();
    }
    u32 lim = 0;
    bool dead = false;
    const char* const base = a.ws[(src - rank) & (NR - 1)] + (size_t)(call & 1) * WS_SLOT + (size_t)(j * NSL) * USTRIDE;
    const u32 tau = q ? TAU : TAU / 4;
    u32 due = (tstart + (u32)max((int)((u32)__builtin_amdgcn_s_memrealtime() - tstart) - CMAX, TREF)) << 4;
    auto pace = [&](u32 bytes) {
      while ((int)(((u32)__builtin_amdgcn_s_memrealtime() << 4) - due) < 0) {}
      due += bytes * tau >> 10;
    };
    int ct_ = 0, cp_ = 0;
    auto issue_c = [&]() {
      const char* sb = base + (size_t)(4 * ct_ + ct) * USTRIDE + U_C;
      const u32 la = LDS_CV + ct_ * CVS;
      if (cp_ == 0) {
        while (!(lim = __builtin_amdgcn_readfirstlane(*LP(volatile u32, L_LEN + ct_ * 4))))
          if (__builtin_amdgcn_readfirstlane(*LP(volatile u32, L_NGO)) == 2) { dead = true; break; }
        if (ct_ == 0) { const u32 now = ((u32)__builtin_amdgcn_s_memrealtime() - CMAX) << 4; if ((int)(now - due) > 0) due = now; }
      }
      if (dead) return;
      pace(cp_ == 2 ? 4u * (lim - 2048u) : 4096u);
      if (cp_ == 2 && lim > 3072u) {
        if (l16 + 3072u < lim) glds<0, true>(la + 3072, sb, l16 + 3072u);
        if (l16 + 4096u < lim) glds<0, true>(la + 4096, sb, l16 + 4096u);
      }
      if (l16 + cp_ * 1024u < lim) glds<0, true>(la + cp_ * 1024, sb, l16 + cp_ * 1024u);
      if (++cp_ == 3) { cp_ = 0; ++ct_; }
    };
    int arrived = 0;
    auto issue_n = [&](int s, int p) {
      const char* sb = base + (size_t)s * USTRIDE;
      const u32 la = LDS_N + (s & 7) * PL;
      pace(1024u);
      if (p == 0) {
        while (s >= 8 && arrived < 4 * (s - 5)) arrived = __builtin_amdgcn_readfirstlane(*LP(volatile u32, L_ARR));
        glds<0, false>(la, sb, l16);
      }
      if (p == 1) glds<1024, false>(la, sb, l16);
      if (p == 2) glds<2048, false>(la, sb, l16);
      if (p == 3) glds<3072, false>(la, sb, l16);
    };
    auto post = [&](u32 addr, u32 v) { *LP(volatile u32, addr) = v; };
    constexpr int NB = 24 - 3 * PRE;
    static_assert(PRE >= 2 && PRE <= 7 && W2 >= 4 && W2 <= 8 && N0 + NB + 4 <= NSL, "");
    int cdone = 0;
    auto landed_c = [&]() { ++cdone; if (cdone % 3 == 0) post(L_CVL, cdone / 3); };
#pragma unroll 1
    for (int s = NPUSH; s < N0; ++s) {
#pragma unroll
      for (int p = 0; p < 4; ++p) { waitv<W2 - 1>(); issue_n(s, p); }
    }
    waitv<3>(); issue_c();
    waitv<1>(); post(L_NL, N0); issue_c();
#pragma unroll 1
    for (int n = 0; n < 3 * PRE - 2; ++n) { waitv<1>(); issue_c(); landed_c(); }
    waitv<1>(); landed_c();
    u32 go;
    do go = __builtin_amdgcn_readfirstlane(*LP(volatile u32, L_NGO)); while (!go);
    if (go == 2) { waitv<0>(); post(L_CVL, 8); post(L_NL, NSL + KT); return; }
#pragma unroll
    for (int p = 0; p < 4; ++p) issue_n(N0, p);
    waitv<4>(); issue_c(); landed_c();
#pragma unroll 1
    for (int s = N0; s < N0 + NB; ++s) {
#pragma unroll
      for (int p = 0; p < 4; ++p) { waitv<4>(); if (p == 3) post(L_NL, s + 1); issue_n(s + 1, p); }
      waitv<4>();
      if (s + 1 < N0 + NB) issue_c();
      landed_c();
    }
#pragma unroll
    for (int n = 4; n < W2; ++n) issue_n(N0 + NB + n / 4, n % 4);
    constexpr int S1 = NSL - KT - (W2 + 3) / 4;
    static_assert(S1 >= N0 + NB, "");
#pragma unroll 1
    for (int s = N0 + NB; s < S1; ++s) {
#pragma unroll
      for (int p = 0; p < 4; ++p) { waitv<W2 - 1>(); if (p == 3) post(L_NL, s + 1); issue_n(s + (p + W2) / 4, (p + W2) % 4); }
    }
    constexpr int NA = 4 * (NSL - KT - S1), NZ = NA + 4 * KT;
#pragma unroll
    for (int n = 0; n < NZ; ++n) {
      waitn(NZ - 1 - n < W2 - 1 ? NZ - 1 - n : W2 - 1);
      if (n < NA ? n % 4 == 3 : n % 2 == 1) post(L_NL, n < NA ? S1 + n / 4 + 1 : NSL - KT + (n - NA) / 2 + 1);
      const int m = n + W2;
      if (m < NA) issue_n(S1 + m / 4, m % 4);
      else if (m < NZ) {
        const int hh = (m - NA) / 2, s = NSL - KT + hh % KT, p = (hh / KT) * 2 + (m - NA) % 2;
        issue_n(s, p);
      }
    }
  }
}

extern "C" void kernel_launch(int rank, int nranks, ncclComm_t comm,
                              hipStream_t stream,
                              void* const* d_in, const int* in_sizes, int n_in,
                              void* d_out, int out_size,
                              const int* in_dtypes, int out_dtype,
                              void* const* d_out_all,
                              void* const* d_ws_all, size_t ws_bytes) {
  if (nranks != NR || (unsigned)rank >= (unsigned)NR || ws_bytes < WS_END || n_in < 4) return;
  if (in_sizes[0] != MS * KTOT || in_sizes[1] != KTOT * NL || out_size != NR * MS * NL) return;
  static bool init = false;
  if (!init) {
    HIP_CHECK(hipFuncSetAttribute((const void*)ag_gemm, hipFuncAttributeMaxDynamicSharedMemorySize, LDS_TOT));
    init = true;
  }
  Args a;
  for (int p = 0; p < NR; ++p) a.ws[p] = (char*)d_ws_all[(rank + p) & (NR - 1)];
  a.x = (const unsigned char*)d_in[0]; a.w = (const unsigned char*)d_in[1];
  a.sx = (const float*)d_in[2]; a.sw = (const float*)d_in[3];
  a.out = (float*)d_out; a.rank = rank;
  {
    const unsigned cl = CLIQUE;
    unsigned o = 0; int n = 0;
    for (int pass = 0; pass < 2; ++pass)
      for (int i = 0; i < NR; ++i) { const int p = (rank + i) & 7; if ((int)(((cl >> p) ^ (cl >> rank)) & 1) == pass) o |= (unsigned)p << (4 * n++); }
    a.order = o;
  }
  hipLaunchKernelGGL(ag_gemm, dim3(NWG), dim3(NTHR), LDS_TOT, stream, a);
  HIP_CHECK(hipGetLastError());
}
```

```cpp
#include <hip/hip_runtime.h>
#include <rccl/rccl.h>
#include <cstdio>
#include <cstdint>

#pragma clang diagnostic ignored "-Wint-to-pointer-cast"

#define HIP_CHECK(expr) do { hipError_t _e = (expr); if (_e != hipSuccess) { \
    fprintf(stderr, "HIP error %s at %s:%d\n", hipGetErrorString(_e), __FILE__, __LINE__); \
    return; } } while (0)

typedef unsigned u32;
typedef unsigned u32x2 __attribute__((ext_vector_type(2)));
typedef unsigned u32x4 __attribute__((ext_vector_type(4)));
typedef int v2i __attribute__((ext_vector_type(2)));
typedef int v8i __attribute__((ext_vector_type(8)));
typedef float v16f __attribute__((ext_vector_type(16)));
typedef unsigned long long u64;
#define LAS __attribute__((address_space(3)))
#define LP(T, a) ((LAS T*)(u32)(a))

constexpr int NPUSH = 3;
constexpr int PACE = 60;
constexpr int NE = 3;
constexpr int PRE = 5;
constexpr int W2 = 6;
constexpr int KT = 3;
constexpr int TAU = 219;
constexpr int TREF = 185;
constexpr int CMAX = 90;
constexpr unsigned CLIQUE = 0x4D;

constexpr int NR = 8;
constexpr int KTOT = 4096, NL = 512, MS = 512;
constexpr int BM = 64, BN = 128, KS = 128, NSL = KTOT / KS;
constexpr int NWG = 256, NTHR = 640;
constexpr int PL = 4096;
constexpr int U_C = 4096, U_H = 6144, U_V = 6160, USTRIDE = 9216;
constexpr int N0 = NPUSH + NE;
static_assert(NPUSH >= 1 && NPUSH <= 3 && NE >= 1 && N0 <= 8, "");
constexpr size_t WS_SLOT  = (size_t)NWG * USTRIDE;
constexpr size_t WS_PUSH  = 2 * WS_SLOT;
constexpr size_t PUSH_PAR = (size_t)NR * 8 * 8 * PL;
constexpr size_t WS_E     = WS_PUSH + 2 * PUSH_PAR;
constexpr size_t WS_FLAG  = WS_E + (size_t)64 * NSL * PL;
constexpr size_t FLAG_PAR = (size_t)NR * 8 * NSL * 128;
constexpr size_t WS_PFLAG = WS_FLAG + 2 * FLAG_PAR;
constexpr size_t WS_NFLAG = WS_PFLAG + NR * 8 * 128;
constexpr size_t WS_EFLAG = WS_NFLAG + NR * 8 * 8 * 128;
constexpr size_t WS_CNT   = WS_EFLAG + 64 * 512;
constexpr size_t WS_INV   = WS_CNT + NWG * 128;
constexpr size_t WS_ROT   = WS_INV + 16 * 128;
constexpr size_t WS_TMO   = WS_ROT + 32 * 128;
constexpr size_t WS_END   = WS_TMO + 128;
constexpr u64 DEADLINE = 100000000ull;
constexpr u64 SLOW = 20000ull;
constexpr int WSL = KS * BN;
constexpr int LDS_LUT = 0;
constexpr int LDS_CLUT = 128;
constexpr int LDS_MISC = 256;
constexpr int L_BAD = LDS_MISC;
constexpr int L_VAL = LDS_MISC + 4;
constexpr int L_NGO = LDS_MISC + 8;
constexpr int L_PUB = LDS_MISC + 12;
constexpr int L_ARR = LDS_MISC + 16;
constexpr int L_NL = LDS_MISC + 20;
constexpr int L_RDY = LDS_MISC + 24;
constexpr int L_CVL = LDS_MISC + 28;
constexpr int L_LEN = LDS_MISC + 32;
constexpr int L_DUMMY = LDS_MISC + 64;
constexpr int L_ROT = LDS_MISC + 192;
constexpr int L_MC = LDS_MISC + 196;
constexpr int L_EC = LDS_MISC + 208;
constexpr int SCR = 2048;
constexpr int A3O = 1040, A4O = 1312, VBO = 1536;
constexpr int LDS_SCR = 512;
constexpr int LDS_E = 9 * 1024;
constexpr int LDS_N = LDS_E + 4 * PL;
constexpr int LDS_W = LDS_N + 8 * PL;
constexpr int CVS = 4352;
constexpr int LDS_CV = LDS_W + 4 * WSL;
constexpr int LDS_TOT = LDS_CV + 8 * CVS;
static_assert(LDS_SCR + 4 * SCR <= LDS_E && LDS_TOT <= 160 * 1024, "lds");

struct Args {
  const unsigned char* x;
  char* ws[NR];
  const unsigned char* w;
  const float* sx; const float* sw; float* out;
  int rank; u32 order;
};

#define SCHED() __builtin_amdgcn_sched_barrier(0)
template <int N> __device__ __forceinline__ void waitv() { asm volatile("s_waitcnt vmcnt(%0)" :: "n"(N) : "memory"); }
__device__ __forceinline__ void waitn(int n) {
  switch (n) {
    case 0: waitv<0>(); break; case 1: waitv<1>(); break; case 2: waitv<2>(); break; case 3: waitv<3>(); break;
    case 4: waitv<4>(); break; case 5: waitv<5>(); break; case 6: waitv<6>(); break; default: waitv<7>();
  }
}
__device__ __forceinline__ void cfence() { asm volatile("" ::: "memory"); }
__device__ __forceinline__ void bar() {
  cfence(); SCHED();
  __builtin_amdgcn_s_waitcnt(0xC07F);
  __builtin_amdgcn_s_barrier();
  SCHED(); cfence();
}
template <int OFF, bool SYS> __device__ __forceinline__ void glds(u32 lds, const void* sbase, u32 voff) {
  if (SYS) asm volatile("s_mov_b32 m0, %2\n\ts_nop 0\n\tglobal_load_lds_dwordx4 %0, %1 offset:%3 sc0 sc1" :: "v"(voff), "s"(sbase), "s"(lds), "n"(OFF) : "memory");
  else asm volatile("s_mov_b32 m0, %2\n\ts_nop 0\n\tglobal_load_lds_dwordx4 %0, %1 offset:%3" :: "v"(voff), "s"(sbase), "s"(lds), "n"(OFF) : "memory");
}
__device__ __forceinline__ void glds4(u32 lds, const void* sbase, const u32 (&voff)[4]) {
  asm volatile("s_mov_b32 m0, %5\n\ts_nop 0\n\tglobal_load_lds_dwordx4 %0, %4\n\tglobal_load_lds_dwordx4 %1, %4 offset:1024\n\t"
               "global_load_lds_dwordx4 %2, %4 offset:2048\n\tglobal_load_lds_dwordx4 %3, %4 offset:3072"
               :: "v"(voff[0]), "v"(voff[1]), "v"(voff[2]), "v"(voff[3]), "s"(sbase), "s"(lds) : "memory");
}
__device__ __forceinline__ void st16(void* p, u32x4 v) { asm volatile("global_store_dwordx4 %0, %1, off sc0 sc1\n\ts_nop 1" :: "v"(p), "v"(v) : "memory"); }
__device__ __forceinline__ void st8(void* p, u32x2 v) { asm volatile("global_store_dwordx2 %0, %1, off sc0 sc1\n\ts_nop 1" :: "v"(p), "v"(v) : "memory"); }
__device__ __forceinline__ void st4(void* p, u32 v) { asm volatile("global_store_dword %0, %1, off sc0 sc1\n\ts_nop 1" :: "v"(p), "v"(v) : "memory"); }
__device__ __forceinline__ u32 ld4(const void* p) {
  u32 v;
  asm volatile("global_load_dword %0, %1, off sc0 sc1\n\ts_waitcnt vmcnt(0)" : "=&v"(v) : "v"(p) : "memory");
  return v;
}

__device__ __forceinline__ bool last_of_four(u32 addr) {
  u32 nth;
  asm volatile("ds_add_rtn_u32 %0, %1, %2\n\ts_waitcnt lgkmcnt(0)" : "=&v"(nth) : "v"(addr), "v"(1u) : "memory");
  return __builtin_amdgcn_readfirstlane(nth) == 3;
}
__device__ __forceinline__ bool expired(u64 t0, u32* tmo, u32 call) {
  const u64 dt = __builtin_amdgcn_s_memrealtime() - t0;
  if (dt <= SLOW) return false;
  if (dt > DEADLINE) { st4(tmo, call); waitv<0>(); return true; }
  __builtin_amdgcn_s_sleep(64);
  return __builtin_amdgcn_readfirstlane(ld4(tmo)) == call;
}

__device__ __forceinline__ u32 wave_scan(u32 x) {
  x += __builtin_amdgcn_update_dpp(0u, x, 0x111, 0xf, 0xf, true);
  x += __builtin_amdgcn_update_dpp(0u, x, 0x112, 0xf, 0xf, true);
  x += __builtin_amdgcn_update_dpp(0u, x, 0x114, 0xf, 0xf, true);
  x += __builtin_amdgcn_update_dpp(0u, x, 0x118, 0xf, 0xf, true);
  x += __builtin_amdgcn_update_dpp(0u, x, 0x142, 0xa, 0xf, false);
  x += __builtin_amdgcn_update_dpp(0u, x, 0x143, 0xc, 0xf, false);
  return x;
}
constexpr u32 B1 = 0x01010101u, M0F = 0x0F0F0F0Fu, MSM = 0x87878787u, MEX = 0x78787878u, M03 = 0x03030303u;
__device__ __forceinline__ u32 rotl4(u32 x) { return __builtin_amdgcn_alignbit(x, x, 28); }
__device__ __forceinline__ u32 rotr4(u32 x) { return __builtin_amdgcn_alignbit(x, x, 4); }
__device__ __forceinline__ u32x2 centry(u32 p) {
  return *LP(const u32x2, __builtin_amdgcn_udot4(p & 0x04040404u, 0x10080402u, (u32)LDS_CLUT, false));
}
__device__ __forceinline__ u32x2 dentry(u32 c, bool with_count = true) {
  const u32 p = __builtin_amdgcn_udot4(__builtin_amdgcn_perm(0u, 0x01000000u, c), 0x40201008u, (u32)LDS_LUT, false);
  if (with_count) return *LP(const u32x2, p);
  return u32x2{*LP(const u32, p), 0u};
}
__device__ __forceinline__ void put4(u32 o, u32 y) {
  const u64 w = (u64)y << ((o & 3u) * 8u);
  LAS u32* p = LP(u32, o & ~3u);
  __hip_atomic_fetch_or(p, (u32)w, __ATOMIC_RELAXED, __HIP_MEMORY_SCOPE_WAVEFRONT);
  __hip_atomic_fetch_or(p + 1, (u32)(w >> 32), __ATOMIC_RELAXED, __HIP_MEMORY_SCOPE_WAVEFRONT);
}
__device__ __forceinline__ u32x2 pair(u32 o) {
  const LAS u32* p = LP(const u32, o & ~3u);
  return u32x2{p[0], p[1]};
}
__device__ __forceinline__ u32 expand(u32x2 w, u32 o, u32 table, u32 sel) {
  return __builtin_amdgcn_perm(__builtin_amdgcn_alignbyte(w[1], w[0], o), table, sel);
}

__global__ void __launch_bounds__(NTHR, 3) ag_gemm(Args a) {
  const int tid = threadIdx.x, lane = tid & 63;
  const int wid = __builtin_amdgcn_readfirstlane(tid >> 6);
  const int b = blockIdx.x;
  char* const me = a.ws[0];
  const int jp = b & 7, idx = b >> 3, ct = idx & 3, pe = idx >> 3;
  const int ps = pe == 3 ? NPUSH + (idx & 7) : pe;
  const int brow = (wid & 3) * 16 + (lane >> 2);
  const unsigned char* const xlane = a.x + (size_t)(jp * BM) * KTOT + (wid < 4 ? ps : idx) * KS + (u32)(brow * KTOT + (((lane & 3) ^ ((brow >> 2) & 3)) << 5));
  const u32x4 v0 = *(const u32x4*)xlane, v1 = *(const u32x4*)(xlane + 16);
  u32* const cnt = (u32*)(me + WS_CNT + (size_t)b * 128);
  const u32 callv = __hip_atomic_load(cnt, __ATOMIC_RELAXED, __HIP_MEMORY_SCOPE_AGENT);
  cfence(); SCHED();
  const int rank = a.rank;
  int j, q, src;
  auto role = [&]() {
    u32 r;
    while (!(r = __builtin_amdgcn_readfirstlane(*LP(volatile u32, L_ROT)))) __builtin_amdgcn_s_sleep(1);
    const int x = (b + r - 1) & 7;
    j = (x & 3) * 2 + ((idx >> 2) & 1); q = (x & 4) | (idx >> 3);
    src = (a.order >> (4 * q)) & (NR - 1);
  };
  const u32 xcc = __builtin_amdgcn_s_getreg((3 << 11) | 20) & 15u;
  u32* const tmo = (u32*)(me + WS_TMO);
  const u32 dummy = L_DUMMY + (lane & 31) * 4;
  const u32 l16 = lane * 16u;
  char* const inv = me + WS_INV + (size_t)xcc * 128;

  if (wid < 4) {
    const bool early = pe == 3 && (idx & 7) < NE;
    auto nplane = [&](const u32x4 v0, const u32x4 v1) {
      u32x4 np;
#pragma unroll
      for (int k = 0; k < 4; ++k) np[k] = (v0[k] & MSM) | rotl4(v1[k] & MSM);
      return np;
    };
    char* nflag = a.ws[lane & (NR - 1)] + WS_NFLAG + (size_t)((rank * 8 + jp) * 8 + (idx & 7)) * 128;
    if (tid < 16) {
      u32 add = 0, cs = 0x0C0C0C0Cu, n = 0;
#pragma unroll
      for (int i = 0; i < 4; ++i)
        if ((tid >> i) & 1) { add |= (1 + n) << (8 * i); cs = (cs & ~(0xFFu << (8 * n))) | (i << (8 * n)); ++n; }
      *LP(u32x2, LDS_LUT + tid * 8) = u32x2{add, n};
      *LP(u32x2, LDS_CLUT + tid * 8) = u32x2{cs, n};
    }
    const int kr = lane >> 3, cp = lane & 7;
    u32 wvo[4];
#pragma unroll
    for (int i = 0; i < 4; ++i) wvo[i] = (kr + i * 8) * NL + ((cp >> 1) ^ ((kr >> 1) & 3)) * 32 + (cp & 1) * 16 - i * 1024;
    const unsigned char* const wsrc = a.w + (size_t)(wid * 32) * NL + ct * BN;
    auto stage_w = [&](int s) { glds4(LDS_W + (s & 3) * WSL + wid * 4096, wsrc + (size_t)s * KS * NL, wvo); };
    char* const pdst = a.ws[idx & (NR - 1)];
    char* pp = (pe == 3 ? me + (size_t)(jp * NSL + ps) * USTRIDE : pdst + WS_PUSH + (size_t)((rank * 8 + jp) * 8 + pe) * PL) + wid * 1024 + l16;
    const size_t podd = pe == 3 ? WS_SLOT : PUSH_PAR;
    asm volatile("" : "+v"(pp));
    bar();
    const u32x4 np0 = nplane(v0, v1);
    const u32 call = __builtin_amdgcn_readfirstlane(callv) + 1u;
    const size_t par = call & 1;
    if (pe == 1 || pe == 2) {
      asm volatile("" :: "v"(np0));
      const u64 tpush = __builtin_amdgcn_s_memrealtime();
      while (__builtin_amdgcn_s_memrealtime() - tpush < (u64)(pe * PACE)) __builtin_amdgcn_s_sleep(1);
    }
    if (pe < NPUSH || early) st16(pp + par * podd, np0);
    auto last = [&]() { waitv<0>(); return last_of_four(lane ? dummy : (u32)L_MC); };
    if (early) {
      asm volatile("" : "+v"(nflag));
      if (last() && lane < NR) st4(nflag, call);
    }
    float sc = a.sx[0] * a.sw[0];
    while (__builtin_amdgcn_readfirstlane(*LP(volatile u32, L_PUB)) < 4) __builtin_amdgcn_s_sleep(4);
    __builtin_amdgcn_s_sleep(32);
    if (pe < NPUSH && last() && lane == 0) st4(pdst + WS_PFLAG + (size_t)(rank * 8 + jp) * 128 + pe * 16, call);
    asm volatile("" : "+v"(sc));
    role();
    const char* const esrc = me + WS_E + (size_t)((q * 8 + j) * NSL) * PL + wid * 1024;
    const char* const psrc = me + WS_PUSH + par * PUSH_PAR + (size_t)((src * 8 + j) * 8) * PL + wid * 1024;
    auto stage_e = [&](int s, bool push) {
      if (push) glds<0, true>(LDS_N + (s & 7) * PL + wid * 1024, psrc + s * PL, l16);
      glds<0, true>(LDS_E + (s & 3) * PL + wid * 1024, esrc + s * PL, l16);
    };
    v16f acc[2] = {};
    const int r31 = lane & 31, h = lane >> 5, wm = wid >> 1, wn = wid & 1;
    const int q8 = (lane & 15) >> 1, p8 = lane & 1, g1 = (lane >> 4) & 1;
    u32 boff[2], aoff[2];
#pragma unroll
    for (int i = 0; i < 2; ++i) {
      boff[i] = LDS_W + (32 * h + q8) * 128 + (((wn * 2 + i) ^ ((q8 >> 1) & 3)) << 5) + g1 * 16 + p8 * 8;
      aoff[i] = (wm * 32 + r31) * 64 + (((i * 2 + h) ^ ((r31 >> 2) & 3)) << 4);
    }
    struct Frag { u32x4 ep[2]; v8i bf[2][2]; };
    auto ldfrag = [&](Frag& f, int kk, int s) {
      f.ep[kk] = *LP(const u32x4, aoff[kk] + (s & 3) * PL + LDS_E);
#pragma unroll
      for (int n = 0; n < 2; ++n) {
        v2i t[4];
#pragma unroll
        for (int r = 0; r < 4; ++r) t[r] = __builtin_amdgcn_ds_read_tr8_b64_v2i32(LP(v2i, boff[n] + (s & 3) * WSL) + (kk * 64 + r * 8) * 16);
        f.bf[kk][n] = v8i{t[0][0], t[0][1], t[1][0], t[1][1], t[2][0], t[2][1], t[3][0], t[3][1]};
      }
    };
    auto afrag = [&](const u32x4 np, const u32x4 ep) {
      v8i af;
#pragma unroll
      for (int k = 0; k < 4; ++k) {
        af[k] = (np[k] & MSM) | (ep[k] & MEX);
        af[4 + k] = rotr4((np[k] & MEX) | (ep[k] & MSM));
      }
      return af;
    };
#define MFMA(n, af, bf) acc[n] = __builtin_amdgcn_mfma_scale_f32_32x32x64_f8f6f4(af, bf, acc[n], 0, 0, 0, 0, 0, 0)
    auto step = [&](int s, const Frag& cur, Frag& nxt, bool more, bool stg) {
      const u32x4 np0 = *LP(const u32x4, aoff[0] + (s & 7) * PL + LDS_N);
      const u32x4 np1 = *LP(const u32x4, aoff[1] + (s & 7) * PL + LDS_N);
      SCHED();
      if (stg) { stage_e(s + 4, false); stage_w(s + 4); }
      SCHED();
      const v8i a0 = afrag(np0, cur.ep[0]);
      MFMA(0, a0, cur.bf[0][0]);
      SCHED();
      const v8i a1 = afrag(np1, cur.ep[1]);
      SCHED();
      if (!more) {
        MFMA(0, a1, cur.bf[1][0]); MFMA(1, a0, cur.bf[0][1]); MFMA(1, a1, cur.bf[1][1]);
        return;
      }
      MFMA(1, a0, cur.bf[0][1]);
      SCHED();
      ldfrag(nxt, 0, s + 1);
      SCHED();
      MFMA(0, a1, cur.bf[1][0]);
      SCHED();
      ldfrag(nxt, 1, s + 1);
      SCHED();
      MFMA(1, a1, cur.bf[1][1]);
      SCHED();
      asm volatile("" :: "v"(a0), "v"(cur.bf[0][0]), "v"(cur.bf[0][1]), "v"(cur.bf[1][0]));
    };
    const u32 arr = lane ? dummy : (u32)L_ARR;
    auto count = [&](u32 addr) { asm volatile("ds_add_u32 %0, %1" :: "v"(addr), "v"(1u) : "memory"); };
    auto spin = [&](int na, int nl, int nr, int nc = 0) {
      while (true) {
        u32x4 w;
        asm volatile("ds_read_b128 %0, %1\n\ts_waitcnt lgkmcnt(0)" : "=&v"(w) : "v"((u32)L_ARR) : "memory");
        if (__all((int)w[0] >= na && (int)w[1] >= nl && (int)w[2] >= nr && (int)w[3] >= nc)) break;
      }
    };
    auto meet = [&](int s) {
      SCHED();
      asm volatile("s_waitcnt lgkmcnt(0)\n\tds_add_u32 %0, %1" :: "v"(arr), "v"(1u) : "memory");
      spin(4 * s + 8, s + 1, min(s + 5, NSL));
      __builtin_amdgcn_s_waitcnt(0xC07F);
      SCHED();
    };
    {
      Frag fa, fb;
#pragma unroll 1
      for (int s = 0; s < 4; ++s) { stage_w(s); __builtin_amdgcn_s_sleep(8); }
      spin(0, 0, 4);
#pragma unroll 1
      for (int s = 0; s < 4; ++s) stage_e(s, s < NPUSH);
      waitv<0>(); meet(-1);
      ldfrag(fa, 0, 0); ldfrag(fa, 1, 0);
      meet(0); step(0, fa, fb, true, true);
#pragma unroll 1
      for (int s = 1; s < NSL - 5; s += 2) {
        waitv<10>(); meet(s);
        step(s, fb, fa, true, true);
        waitv<10>(); meet(s + 1);
        step(s + 1, fa, fb, true, true);
      }
      waitv<10>(); meet(NSL - 5); step(NSL - 5, fb, fa, true, true);
      static_assert(KT == 3, "");
      waitv<10>(); meet(NSL - 4); step(NSL - 4, fa, fb, true, false);
      const int hb = wm * KT;
      const u32 arr2 = lane ? dummy : (u32)L_CVL;
      waitv<5>(); count(arr);
      if (wm) { waitv<0>(); count(arr2); }
      SCHED(); spin(4 * (NSL - 3) + 8, NSL - 2 + hb, 0, 0); SCHED(); step(NSL - 3, fb, fa, true, false);
      if (!wm) { waitv<0>(); count(arr2); }
      SCHED(); spin(0, NSL - 1 + hb, 0, 12); SCHED(); step(NSL - 2, fa, fb, true, false);
      spin(0, 0, NSL + 1, 0);
      if (*LP(volatile u32, L_BAD)) sc = __builtin_nanf("");
      asm volatile("" : "+v"(sc));
      SCHED(); spin(0, NSL + hb, 0, 0); SCHED(); step(NSL - 1, fb, fa, false, false);
    }
    if (tid == 0) st4(cnt, call);
    float* o = a.out + (size_t)(src * MS + j * BM + wm * 32 + 4 * h) * NL + ct * BN + wn * 64 + r31;
#pragma unroll
    for (int n = 0; n < 2; ++n)
#pragma unroll
      for (int i = 0; i < 16; ++i)
        __hip_atomic_store(o + (size_t)((i & 3) + 8 * (i >> 2)) * NL + n * 32, acc[n][i] * sc, __ATOMIC_RELAXED, __HIP_MEMORY_SCOPE_SYSTEM);
  } else if (wid < 8) {
    const int lw = wid - 4;
    const u32 scr = LDS_SCR + lw * SCR;
    u32 call;
    {
      char* fdst = a.ws[lane & (NR - 1)] + WS_FLAG + (size_t)((rank * 8 + jp) * NSL + idx) * 128;
      *LP(u32x4, scr + l16) = u32x4{0, 0, 0, 0};
      *LP(u32x2, scr + 1024 + lane * 8) = u32x2{0, 0};
      bar();
      call = __builtin_amdgcn_readfirstlane(callv) + 1u;
      asm volatile("" : "+v"(fdst));
      char* const dst = me + (size_t)(call & 1) * WS_SLOT + (size_t)(jp * NSL + idx) * USTRIDE;
      const u32 vbuf = scr + VBO;
      const u32 v[8] = {v0[0], v0[1], v0[2], v0[3], v1[0], v1[1], v1[2], v1[3]};
      u32 z[8], sum = 0;
#pragma unroll
      for (int k = 0; k < 8; ++k) { z[k] = v[k] & MEX; if (k & 1) sum = __builtin_amdgcn_udot4(z[k], B1, sum, false); }
      sum = __builtin_amdgcn_readlane(wave_scan(sum), 63);
      const u32 bb = ((sum + 7946u) >> 13) & 15u;
      const u32 kz = ((7u - bb) & 15u) * 0x08080808u;
      u32x4 np;
#pragma unroll
      for (int k = 0; k < 4; ++k) np[k] = (v[k] & MSM) | rotl4(v[4 + k] & MSM);
      if (idx >= N0) st16(dst + lw * 1024 + l16, np);
      u32x2 e1[8], cp2 = {0, 0};
#pragma unroll
      for (int k = 0; k < 8; ++k) {
        z[k] = (((z[k] + kz) >> 3) & M0F) ^ 0x04040404u;
        const u32 p = __builtin_amdgcn_perm(0xFFFFFFFFu, 0x8201FFFFu, z[k]);
        cp2[k >> 2] |= (p & M03) << (2 * (k & 3));
        e1[k] = centry(p);
      }
      const u32 t1 = (e1[0][1] + e1[1][1] + e1[2][1] + e1[3][1]) | ((e1[4][1] + e1[5][1] + e1[6][1] + e1[7][1]) << 16);
      const u32 sc1 = wave_scan(t1);
      const u32 tot1 = __builtin_amdgcn_readlane(sc1, 63);
      const u32 n2 = (tot1 & 0xFFFFu) + (tot1 >> 16);
      bool raw = n2 > 1024u;
      u32 n2d = 0, n3d = 0, n4d = 0;
      if (!raw) {
        const u32 ex = sc1 - t1 + (scr | ((scr + (tot1 & 0xFFFFu)) << 16));
        u32 o = ex & 0xFFFFu;
#pragma unroll
        for (int k = 0; k < 8; ++k) {
          if (k == 4) o = ex >> 16;
          put4(o, __builtin_amdgcn_perm(0u, z[k], e1[k][0])); o += e1[k][1];
        }
        cfence();
        const u32x4 q2 = *LP(const u32x4, scr + l16);
        u32 l2 = 0; u32x2 e2[4];
#pragma unroll
        for (int k = 0; k < 4; ++k) {
          const u32 p = __builtin_amdgcn_perm(0xFFFFFFFFu, 0xFFFF8201u, q2[k] ^ B1);
          l2 |= (p & M03) << (2 * k); e2[k] = centry(p);
        }
        *LP(u32, vbuf + lane * 4) = l2;
        const u32 t2 = e2[0][1] + e2[1][1] + e2[2][1] + e2[3][1];
        const u32 sc2 = wave_scan(t2);
        const u32 n3 = __builtin_amdgcn_readlane(sc2, 63);
        raw = n3 > 256u;
        if (!raw) {
          n2d = (n2 + 15u) >> 4; n3d = (n3 + 15u) >> 4;
          if (t2) {
            o = scr + A3O + sc2 - t2;
#pragma unroll
            for (int k = 0; k < 4; ++k) { put4(o, __builtin_amdgcn_perm(0u, q2[k] ^ 0x05050505u, e2[k][0])); o += e2[k][1]; }
          }
          cfence();
          const u32 q3 = *LP(const u32, scr + A3O + lane * 4) ^ 0x05050505u;
          const u32 p3 = __builtin_amdgcn_perm(0x800182FFu, 0xFFFFFFFFu, q3);
          u32 l3 = (p3 & M03) << (2 * (lane & 3));
          l3 |= __builtin_amdgcn_update_dpp(0u, l3, 0xB1, 0xf, 0xf, false);
          l3 |= __builtin_amdgcn_update_dpp(0u, l3, 0x4E, 0xf, 0xf, false);
          *LP(u32, vbuf + (n2d + (lane >> 2)) * 4) = l3;
          if (__any(p3 & 0x04040404u)) {
            const u32x2 e3 = centry(p3);
            const u32 sc3 = wave_scan(e3[1]);
            const u32 n4 = __builtin_amdgcn_readlane(sc3, 63);
            raw = n4 > 128u;
            if (!raw) {
              if (e3[1]) put4(scr + A4O + sc3 - e3[1], __builtin_amdgcn_perm(0u, q3, e3[0]));
              cfence();
              const u32x2 q4 = *LP(const u32x2, scr + A4O + (lane & 15) * 8);
              n4d = (n4 + 7u) >> 3;
              if (lane < 16) *LP(u32, vbuf + (n2d + n3d + lane) * 4) = q4[0] | (q4[1] << 4);
            }
          }
        }
      }
      cfence();
      u32 vd = n2d + n3d + n4d;
      if (raw) {
        cp2 = u32x2{(v[0] & MEX) | rotl4(v[4] & MEX), (v[1] & MEX) | rotl4(v[5] & MEX)}; vd = 128;
        *LP(u32, vbuf + lane * 4) = (v[2] & MEX) | rotl4(v[6] & MEX);
        *LP(u32, vbuf + 256 + lane * 4) = (v[3] & MEX) | rotl4(v[7] & MEX);
      }
      cfence();
      u32 voff;
      asm volatile("ds_add_rtn_u32 %0, %1, %2\n\ts_waitcnt lgkmcnt(0)" : "=&v"(voff) : "v"(lane ? dummy : (u32)L_VAL), "v"(vd) : "memory");
      const u32 va = *LP(const u32, vbuf + lane * 4), vb = *LP(const u32, vbuf + 256 + lane * 4);
      st8(dst + U_C + lw * 512 + lane * 8, cp2);
      voff = __builtin_amdgcn_readfirstlane(voff);
      if ((u32)lane < vd) st4(dst + U_V + (voff + lane) * 4, va);
      if ((u32)lane + 64u < vd) st4(dst + U_V + (voff + 64 + lane) * 4, vb);
      if (lane == 0) st4(dst + U_H + lw * 4, voff | (n2d << 10) | (n3d << 17) | (n4d << 22) | (bb << 27) | (raw ? 0x80000000u : 0u));
      waitv<0>();
      if (last_of_four(lane ? dummy : (u32)L_PUB)) {
        const u32 vtot = *LP(volatile u32, L_VAL);
        if (lane < NR) st4(fdst + (size_t)(call & 1) * FLAG_PAR, (call << 6) | ((16u + vtot * 4u + 63u) >> 6));
      }
    }
    role();
    char* const edst = me + WS_E + (size_t)((q * 8 + j) * NSL + ct) * PL + lw * 1024 + l16;
    char* const eflag = me + WS_EFLAG + (size_t)((q * 8 + j) * 32 + ct) * 16;
    auto flag = [&](int t) {
      if (last_of_four(lane ? dummy : (u32)(L_EC + t * 4)) && lane == 0) st4(eflag + t * 64, call);
    };
    int pend = -1;
#pragma unroll 1
    for (int t = 0; t < 8; ++t) {
      if ((int)__builtin_amdgcn_readfirstlane(*LP(volatile u32, L_CVL)) <= t) {
        if (pend >= 0) { waitv<0>(); flag(pend); pend = -1; }
        while ((int)__builtin_amdgcn_readfirstlane(*LP(volatile u32, L_CVL)) <= t) __builtin_amdgcn_s_sleep(2);
      }
      cfence();
      const u32 S = LDS_CV + t * CVS;
      const u32 info = __builtin_amdgcn_readfirstlane(*LP(const u32, S + 2048 + lw * 4));
      const u32x2 cc = *LP(const u32x2, S + lw * 512 + lane * 8);
      const u32 V = S + 2064 + (info & 0x1FFu) * 4;
      const u32 n2d = (info >> 10) & 0x7Fu, n3d = (info >> 17) & 0x1Fu, bb = (info >> 27) & 15u;
      u32 l2 = *LP(const u32, V + lane * 4);
      u32x4 ep;
      if (info >> 31) {
        ep = u32x4{cc[0], cc[1], l2, *LP(const u32, V + 256 + lane * 4)};
      } else {
        const u32 l3 = *LP(const u32, V + n2d * 4 + (lane >> 2) * 4);
        u32 c1[8], c2[4]; u32x2 e1[8], e2[4];
#pragma unroll
        for (int k = 0; k < 8; ++k) { c1[k] = (cc[k >> 2] >> (2 * (k & 3))) & M03; e1[k] = dentry(c1[k], (k & 3) != 3); }
        SCHED();
        l2 = (u32)lane < n2d ? l2 : 0u;
        u32 w4 = 0, c3 = (l3 >> (2 * (lane & 3))) & M03;
#pragma unroll
        for (int k = 0; k < 4; ++k) { c2[k] = (l2 >> (2 * k)) & M03; e2[k] = dentry(c2[k], k != 3); }
        SCHED();
        const u32 bbv = bb * B1;
        if (info & (0x1Fu << 22)) {
          c3 = (u32)lane < 4 * n3d ? c3 : 0u;
          const u32x2 e3 = dentry(c3);
          const u32 l4 = *LP(const u32, V + (n2d + n3d + (lane & 15)) * 4) ^ 0x44444444u;
          const u32 K4 = ((bb + 9u) & 15u) * B1;
          *LP(u32x2, scr + A4O + (lane & 15) * 8) = u32x2{(((l4 & M0F) + K4) & M0F) << 3, ((((l4 >> 4) & M0F) + K4) & M0F) << 3};
          cfence();
          const u32 o4 = scr + A4O + min(wave_scan(e3[1]) - e3[1], 128u);
          const u32x2 ww = pair(o4);
          w4 = __builtin_amdgcn_alignbyte(ww[1], ww[0], o4);
          c3 += e3[0];
        }
        *LP(u32, scr + A3O + lane * 4) = __builtin_amdgcn_perm(w4, ((bbv + 0x000A0B0Cu) & M0F) << 3, c3);
        const u32 ta = __builtin_popcount(cc[0] & (cc[0] >> 1) & 0x55555555u), tb = __builtin_popcount(cc[1] & (cc[1] >> 1) & 0x55555555u);
        const u32 t2 = __builtin_popcount(l2 & (l2 >> 1) & 0x55555555u);
        const u32 tt = ta | (tb << 11) | (t2 << 22);
        const u32 sc = wave_scan(tt);
        const u32 na = __builtin_amdgcn_readlane(sc, 63) & 0x7FFu;
        const u32 ex = sc - tt;
        cfence();
        u32 o2[4], o1[8]; u32x2 w[8];
        o2[0] = (ex >> 22) + (scr + A3O);
#pragma unroll
        for (int k = 1; k < 4; ++k) o2[k] = o2[k - 1] + e2[k - 1][1];
#pragma unroll
        for (int k = 0; k < 4; ++k) w[k] = pair(o2[k]);
        SCHED();
        o1[0] = (ex & 0x7FFu) + scr; o1[4] = ((ex >> 11) & 0x7FFu) + (scr + na);
#pragma unroll
        for (int k = 1; k < 8; ++k) if (k != 4) o1[k] = o1[k - 1] + e1[k - 1][1];
#pragma unroll
        for (int k = 0; k < 4; ++k) c2[k] += e2[k][0];
        SCHED();
        u32x4 y;
#pragma unroll
        for (int k = 0; k < 4; ++k) y[k] = expand(w[k], o2[k], ((bbv + 0x000D0E02u) & M0F) << 3, c2[k]);
        *LP(u32x4, scr + l16) = y;
        cfence();
#pragma unroll
        for (int k = 0; k < 8; ++k) w[k] = pair(o1[k]);
        SCHED();
#pragma unroll
        for (int k = 0; k < 8; ++k) c1[k] += e1[k][0];
        SCHED();
        const u32 T1 = ((bbv + 0x00000F01u) & M0F) << 3;
#pragma unroll
        for (int k = 0; k < 4; ++k) ep[k] = expand(w[k], o1[k], T1, c1[k]) | rotl4(expand(w[4 + k], o1[4 + k], T1, c1[4 + k]));
      }
      st16(edst + (size_t)t * 4 * PL, ep);
      if (pend >= 0) { waitv<1>(); flag(pend); }
      pend = t;
    }
    waitv<0>();
    flag(7);
  } else if (wid == 9) {
    if (b < 8) asm volatile("buffer_inv sc0 sc1" ::: "memory");
    bar();
    const u32 call = __builtin_amdgcn_readfirstlane(callv) + 1u;
    waitv<0>();
    if (b < 8 && lane == 0) st4(inv, call);
    while (__builtin_amdgcn_readfirstlane(*LP(volatile u32, L_PUB)) < 4) __builtin_amdgcn_s_sleep(4);
    role();
    const u64 t0 = __builtin_amdgcn_s_memrealtime();
    {
      const char* f = me + WS_FLAG + (size_t)(call & 1) * FLAG_PAR + (size_t)((src * 8 + j) * NSL + (lane & 31)) * 128;
      u32 go = 1, fv;
      while (true) {
        fv = ld4(f);
        const bool ok = (fv >> 6) >= (call & 0x3FFFFFFu);
        if ((lane & 35) == ct && ok) *LP(volatile u32, L_LEN + (lane >> 2) * 4) = max(min(fv & 63u, 33u), 1u) * 64u + 2048u;
        if (__all(ok)) break;
        if (expired(t0, tmo, call)) { go = 2; *LP(volatile u32, L_BAD) = 1; break; }
        __builtin_amdgcn_s_sleep(2);
      }
      *LP(volatile u32, L_NGO) = go;
    }
    const char* const f = lane < 32 ? me + WS_EFLAG + (size_t)((q * 8 + j) * 32 + lane) * 16
                                    : me + WS_PFLAG + (size_t)(src * 8 + j) * 128 + (lane & 7) * 16;
    u64 known = ~0ull << (32 + NPUSH);
    int nready = 0;
    while (true) {
      const int upto = 4 * min((int)__builtin_amdgcn_readfirstlane(*LP(volatile u32, L_CVL)) + 1, 8);
      bool ok = false;
      if (!((known >> lane) & 1) && (lane >= 32 || lane < upto)) ok = ld4(f) >= call;
      known |= __ballot(ok);
      nready = __builtin_ctzll(~(u64)((u32)known & (u32)(known >> 32)));
      if (nready == NSL) break;
      *LP(volatile u32, L_RDY) = nready;
      if (expired(t0, tmo, call)) break;
      __builtin_amdgcn_s_sleep(32);
    }
    *LP(volatile u32, L_RDY) = NSL;
    if (__builtin_amdgcn_readfirstlane(ld4(tmo)) == call) *LP(volatile u32, L_BAD) = 1;
    *LP(volatile u32, L_RDY) = NSL + 1;
  } else {
    __builtin_amdgcn_s_setprio(3);
    *LP(u32x4, LDS_MISC) = u32x4{0, 0, 0, 0};
    *LP(u32x4, LDS_MISC + 16) = u32x4{0, NPUSH, 0, 0};
    *LP(u32x2, L_ROT) = u32x2{0, 0};
    *LP(u32x4, L_EC) = u32x4{0, 0, 0, 0}; *LP(u32x4, L_EC + 16) = u32x4{0, 0, 0, 0};
    *LP(u32x4, L_LEN) = u32x4{0, 0, 0, 0}; *LP(u32x4, L_LEN + 16) = u32x4{0, 0, 0, 0};
    bar();
    const u32 call = __builtin_amdgcn_readfirstlane(callv) + 1u;
    asm volatile("" :: "s"(call));
    const u32 tstart = (u32)__builtin_amdgcn_s_memrealtime();
    {
      if (b == 0 && lane < 32) st4(me + WS_ROT + lane * 128, (call << 4) | (xcc & 7u));
      const u64 t0 = __builtin_amdgcn_s_memrealtime();
      u32 r;
      while (((r = __builtin_amdgcn_readfirstlane(ld4(me + WS_ROT + (b & 31) * 128))) >> 4) < (call & 0xFFFFFFFu)) {
        if (expired(t0, tmo, call)) { *LP(volatile u32, L_BAD) = 1; r = 0; break; }
        __builtin_amdgcn_s_sleep(1);
      }
      *LP(volatile u32, L_ROT) = (r & 7u) + 1u;
      role();
    }
    {
      const char* f = lane < NE ? me + WS_NFLAG + (size_t)((src * 8 + j) * 8 + lane) * 128 : inv;
      const u64 t0 = __builtin_amdgcn_s_memrealtime();
      while (true) {
        const u64 ok = __ballot(ld4(f) >= call);
        if (ok == ~0ull) break;
        if ((~ok & ((1u << NE) - 1)) == 0 && __builtin_amdgcn_s_memrealtime() - t0 > 500) {
          asm volatile("buffer_inv sc0 sc1\n\ts_waitcnt vmcnt(0)" ::: "memory");
          break;
        }
        if (expired(t0, tmo, call)) {
          *LP(volatile u32, L_BAD) = 1; *LP(volatile u32, L_CVL) = 8; *LP(volatile u32, L_NL) = NSL + KT;
          return;
        }
        __builtin_amdgcn_s_sleep(2);
      }
      waitv<0>();
    }
    u32 lim = 0;
    bool dead = false;
    const char* const base = a.ws[(src - rank) & (NR - 1)] + (size_t)(call & 1) * WS_SLOT + (size_t)(j * NSL) * USTRIDE;
    const u32 tau = q ? TAU : TAU / 4;
    u32 due = (tstart + (u32)max((int)((u32)__builtin_amdgcn_s_memrealtime() - tstart) - CMAX, TREF)) << 4;
    auto pace = [&](u32 bytes) {
      while ((int)(((u32)__builtin_amdgcn_s_memrealtime() << 4) - due) < 0) {}
      due += bytes * tau >> 10;
    };
    int ct_ = 0, cp_ = 0;
    auto issue_c = [&]() {
      const char* sb = base + (size_t)(4 * ct_ + ct) * USTRIDE + U_C;
      const u32 la = LDS_CV + ct_ * CVS;
      if (cp_ == 0) {
        while (!(lim = __builtin_amdgcn_readfirstlane(*LP(volatile u32, L_LEN + ct_ * 4))))
          if (__builtin_amdgcn_readfirstlane(*LP(volatile u32, L_NGO)) == 2) { dead = true; break; }
        if (ct_ == 0) { const u32 now = ((u32)__builtin_amdgcn_s_memrealtime() - CMAX) << 4; if ((int)(now - due) > 0) due = now; }
      }
      if (dead) return;
      pace(cp_ == 2 ? 4u * (lim - 2048u) : 4096u);
      if (cp_ == 2 && lim > 3072u) {
        if (l16 + 3072u < lim) glds<0, true>(la + 3072, sb, l16 + 3072u);
        if (l16 + 4096u < lim) glds<0, true>(la + 4096, sb, l16 + 4096u);
      }
      if (l16 + cp_ * 1024u < lim) glds<0, true>(la + cp_ * 1024, sb, l16 + cp_ * 1024u);
      if (++cp_ == 3) { cp_ = 0; ++ct_; }
    };
    int arrived = 0;
    auto issue_n = [&](int s, int p) {
      const char* sb = base + (size_t)s * USTRIDE;
      const u32 la = LDS_N + (s & 7) * PL;
      pace(1024u);
      if (p == 0) {
        while (s >= 8 && arrived < 4 * (s - 5)) arrived = __builtin_amdgcn_readfirstlane(*LP(volatile u32, L_ARR));
        glds<0, false>(la, sb, l16);
      }
      if (p == 1) glds<1024, false>(la, sb, l16);
      if (p == 2) glds<2048, false>(la, sb, l16);
      if (p == 3) glds<3072, false>(la, sb, l16);
    };
    auto post = [&](u32 addr, u32 v) { *LP(volatile u32, addr) = v; };
    constexpr int NB = 24 - 3 * PRE;
    static_assert(PRE >= 2 && PRE <= 7 && W2 >= 4 && W2 <= 8 && N0 + NB + 4 <= NSL, "");
    int cdone = 0;
    auto landed_c = [&]() { ++cdone; if (cdone % 3 == 0) post(L_CVL, cdone / 3); };
#pragma unroll 1
    for (int s = NPUSH; s < N0; ++s) {
#pragma unroll
      for (int p = 0; p < 4; ++p) { waitv<W2 - 1>(); issue_n(s, p); }
    }
    waitv<3>(); issue_c();
    waitv<1>(); post(L_NL, N0); issue_c();
#pragma unroll 1
    for (int n = 0; n < 3 * PRE - 2; ++n) { waitv<1>(); issue_c(); landed_c(); }
    waitv<1>(); landed_c();
    u32 go;
    do go = __builtin_amdgcn_readfirstlane(*LP(volatile u32, L_NGO)); while (!go);
    if (go == 2) { waitv<0>(); post(L_CVL, 8); post(L_NL, NSL + KT); return; }
#pragma unroll
    for (int p = 0; p < 4; ++p) issue_n(N0, p);
    waitv<4>(); issue_c(); landed_c();
#pragma unroll 1
    for (int s = N0; s < N0 + NB; ++s) {
#pragma unroll
      for (int p = 0; p < 4; ++p) { waitv<4>(); if (p == 3) post(L_NL, s + 1); issue_n(s + 1, p); }
      waitv<4>();
      if (s + 1 < N0 + NB) issue_c();
      landed_c();
    }
#pragma unroll
    for (int n = 4; n < W2; ++n) issue_n(N0 + NB + n / 4, n % 4);
    constexpr int S1 = NSL - KT - (W2 + 3) / 4;
    static_assert(S1 >= N0 + NB, "");
#pragma unroll 1
    for (int s = N0 + NB; s < S1; ++s) {
#pragma unroll
      for (int p = 0; p < 4; ++p) { waitv<W2 - 1>(); if (p == 3) post(L_NL, s + 1); issue_n(s + (p + W2) / 4, (p + W2) % 4); }
    }
    constexpr int NA = 4 * (NSL - KT - S1), NZ = NA + 4 * KT;
#pragma unroll
    for (int n = 0; n < NZ; ++n) {
      waitn(NZ - 1 - n < W2 - 1 ? NZ - 1 - n : W2 - 1);
      if (n < NA ? n % 4 == 3 : n % 2 == 1) post(L_NL, n < NA ? S1 + n / 4 + 1 : NSL - KT + (n - NA) / 2 + 1);
      const int m = n + W2;
      if (m < NA) issue_n(S1 + m / 4, m % 4);
      else if (m < NZ) {
        const int hh = (m - NA) / 2, s = NSL - KT + hh % KT, p = (hh / KT) * 2 + (m - NA) % 2;
        issue_n(s, p);
      }
    }
  }
}

extern "C" void kernel_launch(int rank, int nranks, ncclComm_t comm,
                              hipStream_t stream,
                              void* const* d_in, const int* in_sizes, int n_in,
                              void* d_out, int out_size,
                              const int* in_dtypes, int out_dtype,
                              void* const* d_out_all,
                              void* const* d_ws_all, size_t ws_bytes) {
  if (nranks != NR || (unsigned)rank >= (unsigned)NR || ws_bytes < WS_END || n_in < 4) return;
  if (in_sizes[0] != MS * KTOT || in_sizes[1] != KTOT * NL || out_size != NR * MS * NL) return;
  static bool init = false;
  if (!init) {
    HIP_CHECK(hipFuncSetAttribute((const void*)ag_gemm, hipFuncAttributeMaxDynamicSharedMemorySize, LDS_TOT));
    init = true;
  }
  Args a;
  for (int p = 0; p < NR; ++p) a.ws[p] = (char*)d_ws_all[(rank + p) & (NR - 1)];
  a.x = (const unsigned char*)d_in[0]; a.w = (const unsigned char*)d_in[1];
  a.sx = (const float*)d_in[2]; a.sw = (const float*)d_in[3];
  a.out = (float*)d_out; a.rank = rank;
  {
    const unsigned cl = CLIQUE;
    unsigned o = 0; int n = 0;
    for (int pass = 0; pass < 2; ++pass)
      for (int i = 0; i < NR; ++i) { const int p = (rank + i) & 7; if ((int)(((cl >> p) ^ (cl >> rank)) & 1) == pass) o |= (unsigned)p << (4 * n++); }
    a.order = o;
  }
  hipLaunchKernelGGL(ag_gemm, dim3(NWG), dim3(NTHR), LDS_TOT, stream, a);
  HIP_CHECK(hipGetLastError());
}
```
